```python
import numpy as np

kernel_name = "gpu_gemm_a2a_m4096_k4096_n16384_bf16_gelu"


ABSMAX_RTOL = 0.008

COMPUTE_DTYPE = "bfloat16"
EPILOGUE = "gelu"
IN_QUANT = None
OUT_QUANT = None
IN_DTYPES = {"x": "bfloat16", "w_mat": "bfloat16"}
WORLD_SIZE = 8

def _to_bf16(x):
    u = np.ascontiguousarray(x, dtype=np.float32).view(np.uint32)
    return ((u + 0x7FFF + ((u >> 16) & 1)) & 0xFFFF0000).view(np.float32)

def setup_inputs():

    x = _to_bf16(np.random.randn(4096, 4096).astype(np.float32))
    w_mat = _to_bf16((np.random.randn(4096, 16384) / np.sqrt(4096)).astype(np.float32))
    return {"x": x, "w_mat": w_mat}

def _epilogue(y):
    c = 0.7978845608028654
    return (0.5 * y * (1.0 + np.tanh(c * (y + 0.044715 * y ** 3)))).astype(np.float32)

def reference(x, w_mat):
    y = (x @ w_mat).astype(np.float32)
    return _epilogue(y)
```

```cpp
#include <hip/hip_runtime.h>
#include <rccl/rccl.h>
#include <cstdio>

#define HIP_CHECK(expr) do { hipError_t _e = (expr); if (_e != hipSuccess) { \
    fprintf(stderr, "HIP error %s at %s:%d\n", hipGetErrorString(_e), __FILE__, __LINE__); \
    return; } } while (0)

namespace {
#define LAS __attribute__((address_space(3)))
typedef short s16x4 __attribute__((ext_vector_type(4)));
typedef short s16x8 __attribute__((ext_vector_type(8)));
typedef __bf16 bf16x8 __attribute__((ext_vector_type(8)));
typedef float f32x4 __attribute__((ext_vector_type(4)));
typedef unsigned u32x4 __attribute__((ext_vector_type(4)));

constexpr int NR = 8, MLOC = 512, KD = 4096, NG = 16384, NLOC = 2048;
constexpr int MFR = 5, MFL = 3;
constexpr int BMR = 32 * MFR, BML = 32 * MFL;
constexpr int RROWS = 2 * BMR, FROWS = 2 * BML;
static_assert(RROWS + FROWS == MLOC, "row split");
constexpr int AMAX = BMR * 128, SLOT = AMAX + 16384, NS = 4, RING = NS * SLOT, LDS_BYTES = RING + 256;
constexpr unsigned XPITCH = KD * 2, WPITCH = NG * 2, OPITCH = NLOC * 2;
constexpr unsigned OUT_BYTES = (unsigned)NR * MLOC * OPITCH;
constexpr unsigned XBLK = FROWS * XPITCH;
constexpr unsigned WS_CTR = 0;
constexpr unsigned WS_TMO = 8192;
constexpr unsigned WS_XFLAG = 16384;
constexpr unsigned WS_DONE = 32768;
constexpr unsigned WS_X = 1u << 20;
constexpr unsigned WS_NEED = WS_X + 2 * NR * XBLK;
constexpr unsigned long long DEADLINE = 40000000ull;
constexpr int AUX_SYS = 17;

#define XPER2 9
#define OPER 7
constexpr int NGRP = 63 / OPER < 2 * MFR ? 63 / OPER : 2 * MFR;

struct Params { const char* x; const char* w; char* out[NR]; char* ws[NR]; int rank; };

__device__ __forceinline__ unsigned pk_bf16(float lo, float hi) {
  typedef __bf16 bf16x2 __attribute__((ext_vector_type(2)));
  bf16x2 v = {(__bf16)lo, (__bf16)hi};
  return __builtin_bit_cast(unsigned, v);
}
__device__ __forceinline__ float gelu(float y) {
  const float a = -2.0f * 0.7978845608028654f * 1.4426950408889634f, b = a * 0.044715f;
  const float e = __builtin_amdgcn_exp2f(y * __builtin_fmaf(b, y * y, a));
  return y * __builtin_amdgcn_rcpf(1.0f + e);
}
__device__ __forceinline__ u32x4 epi8(const f32x4 u, const f32x4 v, bool odd) {
  const unsigned a0 = pk_bf16(gelu(u[0]), gelu(u[1])), a1 = pk_bf16(gelu(u[2]), gelu(u[3]));
  const unsigned b0 = pk_bf16(gelu(v[0]), gelu(v[1])), b1 = pk_bf16(gelu(v[2]), gelu(v[3]));
  u32x4 w; w.x = odd ? b0 : a0; w.y = odd ? b1 : a1; w.z = odd ? a0 : b0; w.w = odd ? a1 : b1;
  return w;
}
template <int N_> __device__ __forceinline__ void waitv() { asm volatile("s_waitcnt vmcnt(%0)" :: "i"(N_) : "memory"); }
#define WAIT_L0() __builtin_amdgcn_s_waitcnt(0xC07F)
#define BAR() __builtin_amdgcn_s_barrier()
#define SCHED() __builtin_amdgcn_sched_barrier(0)
#define SGB(mask, n) __builtin_amdgcn_sched_group_barrier(mask, n, 0)

struct Frag { s16x8 a[MFR]; s16x8 b[4]; };
template <int MF> __device__ __forceinline__ void rd(Frag& f, LAS char* lds, unsigned aaddr, const unsigned (&baddr)[4], unsigned so, int ks) {
#pragma unroll
  for (int gn = 0; gn < 4; ++gn) {
    const s16x4 lo = __builtin_amdgcn_ds_read_tr16_b64_v4i16((LAS s16x4*)(lds + (baddr[gn] + so) + ks * 8192));
    const s16x4 hi = __builtin_amdgcn_ds_read_tr16_b64_v4i16((LAS s16x4*)(lds + (baddr[gn] + so) + ks * 8192 + 1024));
    f.b[gn] = __builtin_shufflevector(lo, hi, 0, 1, 2, 3, 4, 5, 6, 7);
    if (gn < MF) f.a[gn] = *(LAS s16x8*)(lds + (aaddr + so) + gn * 2048);
  }
#pragma unroll
  for (int m = 4; m < MF; ++m) f.a[m] = *(LAS s16x8*)(lds + (aaddr + so) + m * 2048);
}
template <int MF> __device__ __forceinline__ void mm(f32x4 (&acc)[MF][4], const Frag& f) {
#pragma unroll
  for (int m = 0; m < MF; ++m)
#pragma unroll
    for (int gn = 0; gn < 4; ++gn)
      acc[m][gn] = __builtin_amdgcn_mfma_f32_16x16x32_bf16(__builtin_bit_cast(bf16x8, f.b[gn]), __builtin_bit_cast(bf16x8, f.a[m]), acc[m][gn], 0, 0, 0);
}
template <int MF, int MFN, int NV> __device__ __forceinline__ void ilv() {
  constexpr int NM = 4 * MF, NRD = 8 + MFN;
  constexpr int DBL = NRD + 3 > NM ? NRD + 3 - NM : 0;
#pragma unroll
  for (int z = 0; z < NM; ++z) {
    SGB(0x8, 1);
    if (z < DBL) SGB(0x100, 2); else if (z + DBL < NRD) SGB(0x100, 1);
    if (NV) SGB(0x402, NV);
  }
}
}

#define STAMP(i)

__global__ __launch_bounds__(512, 2) void gemm_a2a(Params p) {
  extern __shared__ __attribute__((aligned(16))) char smem[];
  LAS char* lds = (LAS char*)smem;
  const int tid = threadIdx.x, lane = tid & 63, wid = __builtin_amdgcn_readfirstlane(tid >> 6);
  const int c = blockIdx.x, xcd = c & 7, j = c >> 3, r = p.rank;
  char* const myws = p.ws[r];
  STAMP(0);
  const bool own = j >= 28;
  const int e1 = j >> 2, hm = j & 1, pb = (j >> 1) & 1;
  const int dest = own ? r : (r + 1 + e1) & 7;
  const int pl1 = xcd + 8 * pb;
  const int row1 = own ? 256 * hm : BMR * hm;
  const int h2 = xcd & 1, pl2 = 4 * (xcd >> 1) + (j & 3);
  const int src = own ? r : (r + 1 + e1) & 7;
  const int row2 = own ? 256 * h2 + BMR : RROWS + BML * h2;

  if (wid >= 4) {
    const int pw = wid - 4;
    unsigned vA1[MFR], vA2[MFL], vB[4];
#pragma unroll
    for (int i = 0; i < MFR; ++i) { const int row = 8 * (MFR * pw + i) + (lane >> 3); vA1[i] = row * XPITCH + (((lane & 7) ^ ((row >> 1) & 7)) << 4); }
#pragma unroll
    for (int i = 0; i < 4; ++i) { const int kr = 16 * pw + 4 * i + (lane >> 4), f = (kr & 1) | ((kr & 2) << 1) | (kr & 8); vB[i] = kr * WPITCH + (((lane & 15) ^ f) << 4); }
    const char* a1 = p.x + (size_t)row1 * XPITCH;
    const char* b1 = p.w + (size_t)(dest * 16 + pl1) * 256;
    const int rot1 = (2 * hm + (j >> 1)) & 3, rot2 = (e1 + j) & 3;
    auto issue1 = [&](int s, unsigned so) {
      const int kt = (s & 60) | ((s + rot1) & 3);
      const char* a = a1 + kt * 128; const char* b = b1 + (size_t)kt * 64 * WPITCH;
#pragma unroll
      for (int i = 0; i < MFR; ++i) __builtin_amdgcn_global_load_lds((const unsigned*)(a + vA1[i]), (LAS unsigned*)(lds + so + (pw * MFR + i) * 1024), 16, 0, 0);
#pragma unroll
      for (int i = 0; i < 4; ++i) __builtin_amdgcn_global_load_lds((const unsigned*)(b + vB[i]), (LAS unsigned*)(lds + so + AMAX + (pw * 4 + i) * 1024), 16, 0, 0);
    };
    issue1(0, 0); issue1(1, SLOT); issue1(2, 2 * SLOT); issue1(3, 3 * SLOT);
    unsigned callv;
    { const unsigned one = 1; char* ctr = myws + WS_CTR + (c * 8 + wid) * 4;
      asm volatile("global_atomic_add %0, %1, %2, off sc0" : "=v"(callv) : "v"(ctr), "v"(one) : "memory"); }
#pragma unroll
    for (int i = 0; i < MFL; ++i) { const int row = 8 * (MFL * pw + i) + (lane >> 3); vA2[i] = row * XPITCH + (((lane & 7) ^ ((row >> 1) & 7)) << 4); }
    constexpr int L1 = MFR + 4, L2 = MFL + 4;
    waitv<3 * L1 + 1>(); BAR();
    unsigned so = 0;
    auto nxt = [&]() { so = so + SLOT == RING ? 0 : so + SLOT; };
    waitv<2 * L1 + 1>(); BAR(); issue1(4, so); nxt();
#pragma unroll 1
    for (int i = 1; i < 56; ++i) { waitv<2 * L1>(); BAR(); issue1(i + 4, so); nxt(); }
    asm volatile("" : "+v"(callv));
    const unsigned call = (__builtin_amdgcn_readfirstlane(callv) >> 6) + 1;
    const char* a2 = own ? p.x + (size_t)row2 * XPITCH : myws + WS_X + ((call & 1) * NR + src) * XBLK + (size_t)(BML * h2) * XPITCH;
    const char* b2 = p.w + (size_t)(r * 16 + pl2) * 256;
    auto issue2 = [&](int s, unsigned so) {
      const int kt = (s & 60) | ((s + rot2) & 3);
      const char* a = a2 + kt * 128; const char* b = b2 + (size_t)kt * 64 * WPITCH;
#pragma unroll
      for (int i = 0; i < MFL; ++i) __builtin_amdgcn_global_load_lds((const unsigned*)(a + vA2[i]), (LAS unsigned*)(lds + so + (pw * MFL + i) * 1024), 16, 0, 0);
#pragma unroll
      for (int i = 0; i < 4; ++i) __builtin_amdgcn_global_load_lds((const unsigned*)(b + vB[i]), (LAS unsigned*)(lds + so + AMAX + (pw * 4 + i) * 1024), 16, 0, 0);
    };
    const bool poller = (pw == 0) && !own;
    unsigned* const tmo = (unsigned*)(myws + WS_TMO);
    const unsigned need = 128u * call;
    unsigned fv = 0;
    auto poll_issue = [&](int q) {
      if (poller) { const char* fp = myws + WS_XFLAG + ((src * 2 + q) * 2 + h2) * 128;
        asm volatile("global_load_dword %0, %1, off sc0 sc1" : "=v"(fv) : "v"(fp) : "memory"); }
    };
    auto poll_check = [&](int q) {
      if (poller) {
        asm volatile("" : "+v"(fv));
        if ((int)(__builtin_amdgcn_readfirstlane(fv) - need) < 0) {
          unsigned* fp = (unsigned*)(myws + WS_XFLAG + ((src * 2 + q) * 2 + h2) * 128);
          const unsigned long long t0 = __builtin_amdgcn_s_memrealtime();
          while ((int)(__builtin_amdgcn_readfirstlane(__hip_atomic_load(fp, __ATOMIC_RELAXED, __HIP_MEMORY_SCOPE_SYSTEM)) - need) < 0) {
            if (__builtin_amdgcn_readfirstlane(__hip_atomic_load(tmo, __ATOMIC_RELAXED, __HIP_MEMORY_SCOPE_AGENT)) == call
                || __builtin_amdgcn_s_memrealtime() - t0 > DEADLINE) {
              __hip_atomic_store(tmo, call, __ATOMIC_RELAXED, __HIP_MEMORY_SCOPE_AGENT);
              const unsigned la = RING, one = 1;
              asm volatile("ds_write_b32 %0, %1\n\ts_waitcnt lgkmcnt(0)" :: "v"(la), "v"(one) : "memory");
              break;
            }
            __builtin_amdgcn_s_sleep(2);
          }
        }
        asm volatile("buffer_inv sc0 sc1" ::: "memory");
      }
    };
    waitv<2 * L1>(); BAR(); poll_issue(0); issue1(60, so); nxt();
    waitv<2 * L1>(); BAR(); issue1(61, so); nxt();
    waitv<2 * L1>(); poll_check(0); BAR(); issue1(62, so); nxt();
    waitv<2 * L1>(); BAR(); issue1(63, so); nxt();
    waitv<2 * L1>(); BAR(); issue2(0, so); nxt();
    waitv<L1 + L2>(); BAR(); issue2(1, so); nxt();
#pragma unroll 1
    for (int i = 62; i < 88; ++i) { waitv<2 * L2>(); BAR(); issue2(i - 60, so); nxt(); }
    waitv<2 * L2>(); BAR(); poll_issue(1); issue2(28, so); nxt();
    waitv<2 * L2>(); BAR(); issue2(29, so); nxt();
    waitv<2 * L2>(); poll_check(1); BAR(); issue2(30, so); nxt();
    waitv<2 * L2>(); BAR(); issue2(31, so); nxt();
    waitv<2 * L2>(); BAR(); issue2(32, so); nxt();
#pragma unroll 1
    for (int i = 93; i < 124; ++i) { waitv<2 * L2>(); BAR(); issue2(i - 60, so); nxt(); }
    waitv<2 * L2>(); BAR();
    waitv<L2>(); BAR();
    waitv<0>(); BAR();
    BAR();
    return;
  }

  __builtin_amdgcn_s_setprio(2);
  const int wr = wid >> 1, wc = wid & 1, fr = lane & 15, fq = lane >> 4;
  const bool odd = fq & 1;
  const unsigned call = __builtin_amdgcn_readfirstlane(lane == 0 ? __hip_atomic_fetch_add((unsigned*)(myws + WS_CTR) + c * 8 + wid, 1u, __ATOMIC_RELAXED, __HIP_MEMORY_SCOPE_AGENT) : 0u) + 1;
  const bool sender = wid < 3;
  const int pi = 3 * c + (sender ? wid : 0);
  const unsigned poff = (pi >> 2) * XPITCH + (pi & 3) * 1024 + lane * 16;
  const u32x4 xp0 = *(const u32x4*)(p.x + (size_t)RROWS * XPITCH + poff);
  const u32x4 xp1 = *(const u32x4*)(p.x + (size_t)RROWS * XPITCH + poff + 4096);
  const unsigned long long wsl = (unsigned long long)p.ws[(r + 1 + (lane < 7 ? lane : 0)) & 7];
  const unsigned wslo = (unsigned)wsl, wshi = (unsigned)(wsl >> 32);
  const unsigned xdst = WS_X + ((call & 1) * NR + r) * XBLK;
  const int ph = (c >> 3) + 3 * (c & 7) + 5 * wid;
  int xn = 0, xnext = sender ? ph % ((XPER2 + 1) / 2) : 1 << 20;
  const int xph = xnext;
  auto xfwd = [&](int s) {
    if (s == xnext) {
      const int q = xn >= 7, e = (xn + ph) % 7;
      char* base = (char*)(((unsigned long long)(unsigned)__builtin_amdgcn_readlane(wshi, e) << 32) | (unsigned)__builtin_amdgcn_readlane(wslo, e));
      const auto rs = __builtin_amdgcn_make_buffer_rsrc(base + xdst, 0, (int)XBLK, 0x00020000);
      if (xn == 8) waitv<1>();
      __builtin_amdgcn_raw_buffer_store_b128(q ? xp1 : xp0, rs, poff + q * 4096, 0, AUX_SYS);
      ++xn; xnext = xn < 14 ? (xn * XPER2) / 2 + xph : 1 << 20;
    }
  };
  auto xflag = [&](int q) {
    if (wid == 3 && lane < 7)
      __hip_atomic_fetch_add((unsigned*)((char*)wsl + WS_XFLAG + ((r * 2 + q) * 2 + (c >> 7)) * 128), 1u, __ATOMIC_RELAXED, __HIP_MEMORY_SCOPE_SYSTEM);
  };
  constexpr int XF0 = (8 * XPER2) / 2 + (XPER2 + 1) / 2;
  constexpr int XLAST = (13 * XPER2) / 2 + (XPER2 + 1) / 2 - 1;
  static_assert(XLAST < 64 && XF0 < 62, "x pacing");

  unsigned aR[2], aL[2], baddr[4];
#pragma unroll
  for (int ks = 0; ks < 2; ++ks) { const unsigned t = fr * 128 + 16 * ((4 * ks + fq) ^ ((fr >> 1) & 7)); aR[ks] = t + 16 * MFR * wr * 128; aL[ks] = t + 16 * MFL * wr * 128; }
  { const int q = (lane & 15) >> 2, pp = lane & 3, kr = 8 * fq + q, f = (kr & 1) | ((kr & 2) << 1) | (kr & 8);
#pragma unroll
    for (int gn = 0; gn < 4; ++gn) baddr[gn] = AMAX + kr * 256 + 16 * ((8 * wc + 4 * (gn >> 1) + pp) ^ f) + 8 * ((gn & 1) ^ (pp & 1)); }
  const auto rs1 = __builtin_amdgcn_make_buffer_rsrc(p.out[dest], 0, (int)OUT_BYTES, 0x00020000);
  const unsigned o1 = (r * MLOC + row1 + 16 * MFR * wr + fr) * OPITCH + (pl1 * 128 + 64 * wc + 8 * fq) * 2;
  const auto rs2 = __builtin_amdgcn_make_buffer_rsrc(p.out[r], 0, (int)OUT_BYTES, 0x00020000);
  const unsigned o2 = (src * MLOC + row2 + 16 * MFL * wr + fr) * OPITCH + (pl2 * 128 + 64 * wc + 8 * fq) * 2;

  Frag F0, F1;
  f32x4 accR[MFR][4], accL[MFL][4];
#pragma unroll
  for (int m = 0; m < MFR; ++m)
#pragma unroll
    for (int gn = 0; gn < 4; ++gn) accR[m][gn] = (f32x4){0.f, 0.f, 0.f, 0.f};
  unsigned so = 0;
  auto nxt = [&]() { so = so + SLOT == RING ? 0 : so + SLOT; };
  if (tid == 0) *(volatile LAS unsigned*)(lds + RING) = 0;
  WAIT_L0();
  BAR();
  STAMP(1);
  rd<MFR>(F0, lds, aR[0], baddr, 0, 0);
#pragma unroll 1
  for (int s = 0; s < 63; ++s) {
    SCHED(); rd<MFR>(F1, lds, aR[1], baddr, so, 1); mm<MFR>(accR, F0); ilv<MFR, MFR, 0>(); SCHED(); WAIT_L0(); BAR();
    nxt(); SCHED();
    rd<MFR>(F0, lds, aR[0], baddr, so, 0); mm<MFR>(accR, F1); ilv<MFR, MFR, 0>(); SCHED();
    xfwd(s);
    if (s == XF0 + 1) xflag(0);
  }
  SCHED(); rd<MFR>(F1, lds, aR[1], baddr, so, 1); mm<MFR>(accR, F0); ilv<MFR, MFR, 0>(); SCHED(); WAIT_L0(); BAR();
  nxt(); SCHED();
  rd<MFL>(F0, lds, aL[0], baddr, so, 0); mm<MFR>(accR, F1); ilv<MFR, MFL, 0>(); SCHED();
  xfwd(63);
  STAMP(2);
#pragma unroll
  for (int m = 0; m < MFL; ++m)
#pragma unroll
    for (int gn = 0; gn < 4; ++gn) accL[m][gn] = (f32x4){0.f, 0.f, 0.f, 0.f};
  const int oph = ph % OPER;
#define STEP_L(NV, PRE) do { \
    SCHED(); rd<MFL>(F1, lds, aL[1], baddr, so, 1); mm<MFL>(accL, F0); PRE; ilv<MFL, MFL, NV>(); SCHED(); WAIT_L0(); BAR(); \
    nxt(); SCHED(); \
    rd<MFL>(F0, lds, aL[0], baddr, so, 0); mm<MFL>(accL, F1); ilv<MFL, MFL, 0>(); SCHED(); } while (0)
#pragma unroll
  for (int g = 0; g < NGRP; ++g) {
    u32x4 ov;
    STEP_L(6, ov = epi8(accR[g >> 1][2 * (g & 1)], accR[g >> 1][2 * (g & 1) + 1], odd));
    if (g == 1 && sender) waitv<1>();
    if (oph == 0) __builtin_amdgcn_raw_buffer_store_b128(ov, rs1, o1 + (g >> 1) * 16 * OPITCH + (g & 1) * 64, 0, AUX_SYS);
#pragma unroll 1
    for (int t = 1; t < OPER; ++t) {
      STEP_L(0, );
      if (g == 1 && t == 1) xflag(1);
      if (oph == t) __builtin_amdgcn_raw_buffer_store_b128(ov, rs1, o1 + (g >> 1) * 16 * OPITCH + (g & 1) * 64, 0, AUX_SYS);
    }
  }
#pragma unroll 1
  for (int t = NGRP * OPER; t < 64; ++t) STEP_L(0, );
  STAMP(3);
#pragma unroll
  for (int g = NGRP; g < 2 * MFR; ++g)
    __builtin_amdgcn_raw_buffer_store_b128(epi8(accR[g >> 1][2 * (g & 1)], accR[g >> 1][2 * (g & 1) + 1], odd), rs1, o1 + (g >> 1) * 16 * OPITCH + (g & 1) * 64, 0, AUX_SYS);
  const bool bad = *(volatile LAS unsigned*)(lds + RING) != 0;
#pragma unroll
  for (int g = 0; g < 2 * MFL; ++g) {
    u32x4 ov = epi8(accL[g >> 1][2 * (g & 1)], accL[g >> 1][2 * (g & 1) + 1], odd);
    if (bad) ov = (u32x4){0x7fc07fc0u, 0x7fc07fc0u, 0x7fc07fc0u, 0x7fc07fc0u};
    __builtin_amdgcn_raw_buffer_store_b128(ov, rs2, o2 + (g >> 1) * 16 * OPITCH + (g & 1) * 64, 0, AUX_SYS);
  }
  STAMP(4);
  waitv<0>();
  BAR();
  STAMP(5);
  if (wid == 0) {
    if (!own && lane == 0) {
      __builtin_amdgcn_fence(__ATOMIC_RELEASE, "");
      waitv<0>();
      __hip_atomic_fetch_add((unsigned*)(p.ws[dest] + WS_DONE + r * 128), 1u, __ATOMIC_RELAXED, __HIP_MEMORY_SCOPE_SYSTEM);
    }
    if (c == 255) {
      unsigned* const tmo = (unsigned*)(myws + WS_TMO);
      unsigned* const fp = (unsigned*)(myws + WS_DONE + ((r + 1 + (lane < 7 ? lane : 0)) & 7) * 128);
      const unsigned need = 32u * call;
      const unsigned long long t0 = __builtin_amdgcn_s_memrealtime();
      while (!__all((int)(__hip_atomic_load(fp, __ATOMIC_RELAXED, __HIP_MEMORY_SCOPE_SYSTEM) - need) >= 0)) {
        if (__hip_atomic_load(tmo, __ATOMIC_RELAXED, __HIP_MEMORY_SCOPE_AGENT) == call || __builtin_amdgcn_s_memrealtime() - t0 > DEADLINE) {
          __hip_atomic_store(tmo, call, __ATOMIC_RELAXED, __HIP_MEMORY_SCOPE_AGENT);
          if (lane == 0) __builtin_amdgcn_raw_buffer_store_b32(0x7fc07fc0u, rs2, o2, 0, AUX_SYS);
          break;
        }
        __builtin_amdgcn_s_sleep(1);
      }
      __builtin_amdgcn_fence(__ATOMIC_ACQUIRE, "");
    }
  }
  STAMP(6);
}

extern "C" void kernel_launch(int rank, int nranks, ncclComm_t comm,
                              hipStream_t stream,
                              void* const* d_in, const int* in_sizes, int n_in,
                              void* d_out, int out_size,
                              const int* in_dtypes, int out_dtype,
                              void* const* d_out_all,
                              void* const* d_ws_all, size_t ws_bytes) {
  if (nranks != NR || (unsigned)rank >= (unsigned)NR || n_in != 2 || in_sizes[0] != MLOC * KD || in_sizes[1] != KD * NG ||
      out_size != NR * MLOC * NLOC || ws_bytes < WS_NEED) { fprintf(stderr, "gemm_a2a: unexpected problem shape\n"); return; }
  static bool attr_set[NR] = {};
  if (!attr_set[rank]) { HIP_CHECK(hipFuncSetAttribute((const void*)gemm_a2a, hipFuncAttributeMaxDynamicSharedMemorySize, LDS_BYTES)); attr_set[rank] = true; }
  Params p;
  p.x = (const char*)d_in[0]; p.w = (const char*)d_in[1]; p.rank = rank;
  for (int i = 0; i < NR; ++i) { p.out[i] = (char*)d_out_all[i]; p.ws[i] = (char*)d_ws_all[i]; }
  hipLaunchKernelGGL(gemm_a2a, dim3(256), dim3(512), LDS_BYTES, stream, p);
  HIP_CHECK(hipGetLastError());
}
```

```cpp
#include <hip/hip_runtime.h>
#include <rccl/rccl.h>
#include <cstdio>

#define HIP_CHECK(expr) do { hipError_t _e = (expr); if (_e != hipSuccess) { \
    fprintf(stderr, "HIP error %s at %s:%d\n", hipGetErrorString(_e), __FILE__, __LINE__); \
    return; } } while (0)

namespace {
#define LAS __attribute__((address_space(3)))
typedef short s16x4 __attribute__((ext_vector_type(4)));
typedef short s16x8 __attribute__((ext_vector_type(8)));
typedef __bf16 bf16x8 __attribute__((ext_vector_type(8)));
typedef float f32x4 __attribute__((ext_vector_type(4)));
typedef unsigned u32x4 __attribute__((ext_vector_type(4)));

constexpr int NR = 8, MLOC = 512, KD = 4096, NG = 16384, NLOC = 2048;
constexpr int MFR = 5, MFL = 3;
constexpr int BMR = 32 * MFR, BML = 32 * MFL;
constexpr int RROWS = 2 * BMR, FROWS = 2 * BML;
static_assert(RROWS + FROWS == MLOC, "row split");
constexpr int AMAX = BMR * 128, SLOT = AMAX + 16384, NS = 4, RING = NS * SLOT, LDS_BYTES = RING + 512;
constexpr unsigned XPITCH = KD * 2, WPITCH = NG * 2, OPITCH = NLOC * 2;
constexpr unsigned OUT_BYTES = (unsigned)NR * MLOC * OPITCH;
constexpr unsigned XBLK = FROWS * XPITCH;
constexpr unsigned WS_CTR = 0;
constexpr unsigned WS_TMO = 8192;
constexpr unsigned WS_XFLAG = 16384;
constexpr unsigned WS_DONE = 32768;
constexpr unsigned WS_X = 1u << 20;
constexpr unsigned WS_NEED = WS_X + 2 * NR * XBLK;
constexpr unsigned long long DEADLINE = 40000000ull;
constexpr int AUX_SYS = 17;

#define XPER2 9
#define OPER 7
constexpr int NGRP = 63 / OPER < 2 * MFR ? 63 / OPER : 2 * MFR;

struct Params { const char* x; const char* w; char* out[NR]; char* ws[NR]; int rank; };

__device__ __forceinline__ unsigned pk_bf16(float lo, float hi) {
  typedef __bf16 bf16x2 __attribute__((ext_vector_type(2)));
  bf16x2 v = {(__bf16)lo, (__bf16)hi};
  return __builtin_bit_cast(unsigned, v);
}
__device__ __forceinline__ float gelu(float y) {
  const float a = -2.0f * 0.7978845608028654f * 1.4426950408889634f, b = a * 0.044715f;
  const float e = __builtin_amdgcn_exp2f(y * __builtin_fmaf(b, y * y, a));
  return y * __builtin_amdgcn_rcpf(1.0f + e);
}
__device__ __forceinline__ u32x4 epi8(const f32x4 u, const f32x4 v, bool odd) {
  const unsigned a0 = pk_bf16(gelu(u[0]), gelu(u[1])), a1 = pk_bf16(gelu(u[2]), gelu(u[3]));
  const unsigned b0 = pk_bf16(gelu(v[0]), gelu(v[1])), b1 = pk_bf16(gelu(v[2]), gelu(v[3]));
  u32x4 w; w.x = odd ? b0 : a0; w.y = odd ? b1 : a1; w.z = odd ? a0 : b0; w.w = odd ? a1 : b1;
  return w;
}
template <int N_> __device__ __forceinline__ void waitv() { asm volatile("s_waitcnt vmcnt(%0)" :: "i"(N_) : "memory"); }
#define WAIT_L0() __builtin_amdgcn_s_waitcnt(0xC07F)
#define BAR() __builtin_amdgcn_s_barrier()
#define SCHED() __builtin_amdgcn_sched_barrier(0)
#define SGB(mask, n) __builtin_amdgcn_sched_group_barrier(mask, n, 0)

struct Frag { s16x8 a[MFR]; s16x8 b[4]; };
template <int MF> __device__ __forceinline__ void rd(Frag& f, LAS char* lds, unsigned aaddr, const unsigned (&baddr)[4], unsigned so, int ks) {
#pragma unroll
  for (int gn = 0; gn < 4; ++gn) {
    const s16x4 lo = __builtin_amdgcn_ds_read_tr16_b64_v4i16((LAS s16x4*)(lds + (baddr[gn] + so) + ks * 8192));
    const s16x4 hi = __builtin_amdgcn_ds_read_tr16_b64_v4i16((LAS s16x4*)(lds + (baddr[gn] + so) + ks * 8192 + 1024));
    f.b[gn] = __builtin_shufflevector(lo, hi, 0, 1, 2, 3, 4, 5, 6, 7);
    if (gn < MF) f.a[gn] = *(LAS s16x8*)(lds + (aaddr + so) + gn * 2048);
  }
#pragma unroll
  for (int m = 4; m < MF; ++m) f.a[m] = *(LAS s16x8*)(lds + (aaddr + so) + m * 2048);
}
template <int MF> __device__ __forceinline__ void mm(f32x4 (&acc)[MF][4], const Frag& f) {
#pragma unroll
  for (int m = 0; m < MF; ++m)
#pragma unroll
    for (int gn = 0; gn < 4; ++gn)
      acc[m][gn] = __builtin_amdgcn_mfma_f32_16x16x32_bf16(__builtin_bit_cast(bf16x8, f.b[gn]), __builtin_bit_cast(bf16x8, f.a[m]), acc[m][gn], 0, 0, 0);
}
template <int MF, int MFN, int NV> __device__ __forceinline__ void ilv() {
  constexpr int NM = 4 * MF, NRD = 8 + MFN;
  constexpr int DBL = NRD + 3 > NM ? NRD + 3 - NM : 0;
#pragma unroll
  for (int z = 0; z < NM; ++z) {
    SGB(0x8, 1);
    if (z < DBL) SGB(0x100, 2); else if (z + DBL < NRD) SGB(0x100, 1);
    if (NV) SGB(0x402, NV);
  }
}
}

#define STAMP(i)

__global__ __launch_bounds__(512, 2) void gemm_a2a(Params p) {
  extern __shared__ __attribute__((aligned(16))) char smem[];
  LAS char* lds = (LAS char*)smem;
  const int tid = threadIdx.x, lane = tid & 63, wid = __builtin_amdgcn_readfirstlane(tid >> 6);
  const int c = blockIdx.x, xcd = c & 7, j = c >> 3, r = p.rank;
  char* const myws = p.ws[r];
  STAMP(0);
  const bool own = j >= 28;
  const int e1 = j >> 2, hm = j & 1, pb = (j >> 1) & 1;
  const int dest = own ? r : (r + 1 + e1) & 7;
  const int pl1 = xcd + 8 * pb;
  const int row1 = own ? 256 * hm : BMR * hm;
  const int h2 = xcd & 1, pl2 = 4 * (xcd >> 1) + (j & 3);
  const int src = own ? r : (r + 1 + e1) & 7;
  const int row2 = own ? 256 * h2 + BMR : RROWS + BML * h2;

  if (wid >= 4) {
    const int pw = wid - 4;
    unsigned vA1[MFR], vA2[MFL], vB[4];
#pragma unroll
    for (int i = 0; i < MFR; ++i) { const int row = 8 * (MFR * pw + i) + (lane >> 3); vA1[i] = row * XPITCH + (((lane & 7) ^ ((row >> 1) & 7)) << 4); }
#pragma unroll
    for (int i = 0; i < 4; ++i) { const int kr = 16 * pw + 4 * i + (lane >> 4), f = (kr & 1) | ((kr & 2) << 1) | (kr & 8); vB[i] = kr * WPITCH + (((lane & 15) ^ f) << 4); }
    const char* a1 = p.x + (size_t)row1 * XPITCH;
    const char* b1 = p.w + (size_t)(dest * 16 + pl1) * 256;
#define ROT1 1
    const int rot1 = ROT1 * ((2 * hm + (j >> 1)) & 3), rot2 = (e1 + j) & 3;
    auto issue1 = [&](int s, unsigned so) {
      const int kt = (s & 60) | ((s + rot1) & 3);
      const char* a = a1 + kt * 128; const char* b = b1 + (size_t)kt * 64 * WPITCH;
#pragma unroll
      for (int i = 0; i < MFR; ++i) __builtin_amdgcn_global_load_lds((const unsigned*)(a + vA1[i]), (LAS unsigned*)(lds + so + (pw * MFR + i) * 1024), 16, 0, 0);
#pragma unroll
      for (int i = 0; i < 4; ++i) __builtin_amdgcn_global_load_lds((const unsigned*)(b + vB[i]), (LAS unsigned*)(lds + so + AMAX + (pw * 4 + i) * 1024), 16, 0, 0);
    };
    issue1(0, 0); issue1(1, SLOT); issue1(2, 2 * SLOT); issue1(3, 3 * SLOT);
    unsigned callv = 0;
    if (lane == 0) { const unsigned one = 1; char* ctr = myws + WS_CTR + (c * 8 + wid) * 4;
      asm volatile("global_atomic_add %0, %1, %2, off sc0" : "+v"(callv) : "v"(ctr), "v"(one) : "memory"); }
#pragma unroll
    for (int i = 0; i < MFL; ++i) { const int row = 8 * (MFL * pw + i) + (lane >> 3); vA2[i] = row * XPITCH + (((lane & 7) ^ ((row >> 1) & 7)) << 4); }
    constexpr int L1 = MFR + 4, L2 = MFL + 4;
    waitv<3 * L1 + 1>(); BAR();
    unsigned so = 0;
    auto nxt = [&]() { so = so + SLOT == RING ? 0 : so + SLOT; };
    waitv<2 * L1 + 1>(); BAR(); issue1(4, so); nxt();
#pragma unroll 1
    for (int i = 1; i < 56; ++i) { waitv<2 * L1>(); BAR(); issue1(i + 4, so); nxt(); }
    asm volatile("" : "+v"(callv));
    const unsigned call = __builtin_amdgcn_readfirstlane(callv) + 1;
    const char* a2 = own ? p.x + (size_t)row2 * XPITCH : myws + WS_X + ((call & 1) * NR + src) * XBLK + (size_t)(BML * h2) * XPITCH;
    const char* b2 = p.w + (size_t)(r * 16 + pl2) * 256;
    auto issue2 = [&](int s, unsigned so) {
      const int kt = (s & 60) | ((s + rot2) & 3);
      const char* a = a2 + kt * 128; const char* b = b2 + (size_t)kt * 64 * WPITCH;
#pragma unroll
      for (int i = 0; i < MFL; ++i) __builtin_amdgcn_global_load_lds((const unsigned*)(a + vA2[i]), (LAS unsigned*)(lds + so + (pw * MFL + i) * 1024), 16, 0, 0);
#pragma unroll
      for (int i = 0; i < 4; ++i) __builtin_amdgcn_global_load_lds((const unsigned*)(b + vB[i]), (LAS unsigned*)(lds + so + AMAX + (pw * 4 + i) * 1024), 16, 0, 0);
    };
    const bool poller = (pw == 0) && !own;
    unsigned* const tmo = (unsigned*)(myws + WS_TMO);
    const unsigned need = 128u * call;
    unsigned fv = 0;
    auto poll_issue = [&](int q) {
      if (poller) { const char* fp = myws + WS_XFLAG + ((src * 2 + q) * 2 + h2) * 128;
        asm volatile("global_load_dword %0, %1, off sc0 sc1" : "=v"(fv) : "v"(fp) : "memory"); }
    };
    auto poll_check = [&](int q) {
      if (poller) {
        asm volatile("" : "+v"(fv));
        if ((int)(__builtin_amdgcn_readfirstlane(fv) - need) < 0) {
          unsigned* fp = (unsigned*)(myws + WS_XFLAG + ((src * 2 + q) * 2 + h2) * 128);
          const unsigned long long t0 = __builtin_amdgcn_s_memrealtime();
          while ((int)(__builtin_amdgcn_readfirstlane(__hip_atomic_load(fp, __ATOMIC_RELAXED, __HIP_MEMORY_SCOPE_SYSTEM)) - need) < 0) {
            if (__builtin_amdgcn_readfirstlane(__hip_atomic_load(tmo, __ATOMIC_RELAXED, __HIP_MEMORY_SCOPE_AGENT)) == call
                || __builtin_amdgcn_s_memrealtime() - t0 > DEADLINE) {
              __hip_atomic_store(tmo, call, __ATOMIC_RELAXED, __HIP_MEMORY_SCOPE_AGENT);
              const unsigned la = RING, one = 1;
              asm volatile("ds_write_b32 %0, %1\n\ts_waitcnt lgkmcnt(0)" :: "v"(la), "v"(one) : "memory");
              break;
            }
            __builtin_amdgcn_s_sleep(2);
          }
        }
        asm volatile("buffer_inv sc0 sc1" ::: "memory");
      }
    };
    waitv<2 * L1>(); BAR(); poll_issue(0); issue1(60, so); nxt();
    waitv<2 * L1>(); BAR(); issue1(61, so); nxt();
    waitv<2 * L1>(); poll_check(0); BAR(); issue1(62, so); nxt();
    waitv<2 * L1>(); BAR(); issue1(63, so); nxt();
    waitv<2 * L1>(); BAR(); issue2(0, so); nxt();
    waitv<L1 + L2>(); BAR(); issue2(1, so); nxt();
#pragma unroll 1
    for (int i = 62; i < 88; ++i) { waitv<2 * L2>(); BAR(); issue2(i - 60, so); nxt(); }
    waitv<2 * L2>(); BAR(); poll_issue(1); issue2(28, so); nxt();
    waitv<2 * L2>(); BAR(); issue2(29, so); nxt();
    waitv<2 * L2>(); poll_check(1); BAR(); issue2(30, so); nxt();
    waitv<2 * L2>(); BAR(); issue2(31, so); nxt();
    waitv<2 * L2>(); BAR(); issue2(32, so); nxt();
#pragma unroll 1
    for (int i = 93; i < 124; ++i) { waitv<2 * L2>(); BAR(); issue2(i - 60, so); nxt(); }
    waitv<2 * L2>(); BAR();
    waitv<L2>(); BAR();
    waitv<0>(); BAR();
    BAR();
    return;
  }

  __builtin_amdgcn_s_setprio(2);
  const int wr = wid >> 1, wc = wid & 1, fr = lane & 15, fq = lane >> 4;
  const bool odd = fq & 1;
  const unsigned call = __builtin_amdgcn_readfirstlane(lane == 0 ? __hip_atomic_fetch_add((unsigned*)(myws + WS_CTR) + c * 8 + wid, 1u, __ATOMIC_RELAXED, __HIP_MEMORY_SCOPE_AGENT) : 0u) + 1;
  const bool sender = wid < 3;
  const int pi = 3 * c + (sender ? wid : 0);
  const unsigned poff = (pi >> 2) * XPITCH + (pi & 3) * 1024 + lane * 16;
  const u32x4 xp0 = *(const u32x4*)(p.x + (size_t)RROWS * XPITCH + poff);
  const u32x4 xp1 = *(const u32x4*)(p.x + (size_t)RROWS * XPITCH + poff + 4096);
  const unsigned long long wsl = (unsigned long long)p.ws[(r + 1 + (lane < 7 ? lane : 0)) & 7];
  const unsigned wslo = (unsigned)wsl, wshi = (unsigned)(wsl >> 32);
  const unsigned xdst = WS_X + ((call & 1) * NR + r) * XBLK;
  const int ph = (c >> 3) + 3 * (c & 7) + 5 * wid;
  int xn = 0, xnext = sender ? ph % ((XPER2 + 1) / 2) : 1 << 20;
  const int xph = xnext;
  auto xfwd = [&](int s) {
    if (s == xnext) {
      const int q = xn >= 7, e = (xn + ph) % 7;
      char* base = (char*)(((unsigned long long)(unsigned)__builtin_amdgcn_readlane(wshi, e) << 32) | (unsigned)__builtin_amdgcn_readlane(wslo, e));
      const auto rs = __builtin_amdgcn_make_buffer_rsrc(base + xdst, 0, (int)XBLK, 0x00020000);
      if (xn == 8) waitv<1>();
      __builtin_amdgcn_raw_buffer_store_b128(q ? xp1 : xp0, rs, poff + q * 4096, 0, AUX_SYS);
      ++xn; xnext = xn < 14 ? (xn * XPER2) / 2 + xph : 1 << 20;
    }
  };
  auto xflag = [&](int q) {
    if (wid == 3 && lane < 7)
      __hip_atomic_fetch_add((unsigned*)((char*)wsl + WS_XFLAG + ((r * 2 + q) * 2 + (c >> 7)) * 128), 1u, __ATOMIC_RELAXED, __HIP_MEMORY_SCOPE_SYSTEM);
  };
  constexpr int XF0 = (8 * XPER2) / 2 + (XPER2 + 1) / 2;
  constexpr int XLAST = (13 * XPER2) / 2 + (XPER2 + 1) / 2 - 1;
  static_assert(XLAST < 64 && XF0 < 62, "x pacing");

  unsigned aR[2], aL[2], baddr[4];
#pragma unroll
  for (int ks = 0; ks < 2; ++ks) { const unsigned t = fr * 128 + 16 * ((4 * ks + fq) ^ ((fr >> 1) & 7)); aR[ks] = t + 16 * MFR * wr * 128; aL[ks] = t + 16 * MFL * wr * 128; }
  { const int q = (lane & 15) >> 2, pp = lane & 3, kr = 8 * fq + q, f = (kr & 1) | ((kr & 2) << 1) | (kr & 8);
#pragma unroll
    for (int gn = 0; gn < 4; ++gn) baddr[gn] = AMAX + kr * 256 + 16 * ((8 * wc + 4 * (gn >> 1) + pp) ^ f) + 8 * ((gn & 1) ^ (pp & 1)); }
  const auto rs1 = __builtin_amdgcn_make_buffer_rsrc(p.out[dest], 0, (int)OUT_BYTES, 0x00020000);
  const unsigned o1 = (r * MLOC + row1 + 16 * MFR * wr + fr) * OPITCH + (pl1 * 128 + 64 * wc + 8 * fq) * 2;
  const auto rs2 = __builtin_amdgcn_make_buffer_rsrc(p.out[r], 0, (int)OUT_BYTES, 0x00020000);
  const unsigned o2 = (src * MLOC + row2 + 16 * MFL * wr + fr) * OPITCH + (pl2 * 128 + 64 * wc + 8 * fq) * 2;

#define PFD 0
  auto pf = [&](const char* base, unsigned voff) {
    unsigned keep; const unsigned dump = RING + 256;
    asm volatile("s_mov_b32 %0, m0\n\ts_mov_b32 m0, %3\n\ts_nop 0\n\tglobal_load_lds_dword %1, %2\n\ts_mov_b32 m0, %0" : "=&s"(keep) : "v"(voff), "s"(base), "s"(dump) : "memory");
  };
  const char* const pw1 = p.w + (size_t)(dest * 16 + pl1) * 256;
  const unsigned pvW1 = (32 * hm + (lane >> 1)) * WPITCH + (lane & 1) * 128;
  const unsigned pvX1 = min(16 * j + (lane >> 2), 415) * XPITCH + (lane & 3) * 128;
  auto pf1 = [&](int s) {
    const int t = s + PFD;
    if (PFD && t < 64) {
      if (wid == (s & 3)) pf(pw1 + (size_t)t * 64 * WPITCH, pvW1);
      if ((t & 3) == 0 && wid == ((s + 2) & 3) && j < 26) pf(p.x + t * 128, pvX1);
    }
  };
  Frag F0, F1;
  f32x4 accR[MFR][4], accL[MFL][4];
#pragma unroll
  for (int m = 0; m < MFR; ++m)
#pragma unroll
    for (int gn = 0; gn < 4; ++gn) accR[m][gn] = (f32x4){0.f, 0.f, 0.f, 0.f};
  unsigned so = 0;
  auto nxt = [&]() { so = so + SLOT == RING ? 0 : so + SLOT; };
  for (int s = 4 - PFD; s < 0; ++s) pf1(s);
  if (tid == 0) *(volatile LAS unsigned*)(lds + RING) = 0;
  WAIT_L0();
  BAR();
  STAMP(1);
  rd<MFR>(F0, lds, aR[0], baddr, 0, 0);
#pragma unroll 1
  for (int s = 0; s < 63; ++s) {
    SCHED(); rd<MFR>(F1, lds, aR[1], baddr, so, 1); mm<MFR>(accR, F0); ilv<MFR, MFR, 0>(); SCHED(); WAIT_L0(); BAR();
    nxt(); SCHED();
    rd<MFR>(F0, lds, aR[0], baddr, so, 0); mm<MFR>(accR, F1); ilv<MFR, MFR, 0>(); SCHED();
    xfwd(s); pf1(s);
    if (s == XF0 + 1) xflag(0);
  }
  SCHED(); rd<MFR>(F1, lds, aR[1], baddr, so, 1); mm<MFR>(accR, F0); ilv<MFR, MFR, 0>(); SCHED(); WAIT_L0(); BAR();
  nxt(); SCHED();
  rd<MFL>(F0, lds, aL[0], baddr, so, 0); mm<MFR>(accR, F1); ilv<MFR, MFL, 0>(); SCHED();
  xfwd(63);
  STAMP(2);
#pragma unroll
  for (int m = 0; m < MFL; ++m)
#pragma unroll
    for (int gn = 0; gn < 4; ++gn) accL[m][gn] = (f32x4){0.f, 0.f, 0.f, 0.f};
  const int oph = ph % OPER;
#define STEP_L(NV, PRE) do { \
    SCHED(); rd<MFL>(F1, lds, aL[1], baddr, so, 1); mm<MFL>(accL, F0); PRE; ilv<MFL, MFL, NV>(); SCHED(); WAIT_L0(); BAR(); \
    nxt(); SCHED(); \
    rd<MFL>(F0, lds, aL[0], baddr, so, 0); mm<MFL>(accL, F1); ilv<MFL, MFL, 0>(); SCHED(); } while (0)
#pragma unroll
  for (int g = 0; g < NGRP; ++g) {
    u32x4 ov;
    STEP_L(6, ov = epi8(accR[g >> 1][2 * (g & 1)], accR[g >> 1][2 * (g & 1) + 1], odd));
    if (g == 1 && sender) waitv<1>();
    if (oph == 0) __builtin_amdgcn_raw_buffer_store_b128(ov, rs1, o1 + (g >> 1) * 16 * OPITCH + (g & 1) * 64, 0, AUX_SYS);
#pragma unroll 1
    for (int t = 1; t < OPER; ++t) {
      STEP_L(0, );
      if (g == 1 && t == 1) xflag(1);
      if (oph == t) __builtin_amdgcn_raw_buffer_store_b128(ov, rs1, o1 + (g >> 1) * 16 * OPITCH + (g & 1) * 64, 0, AUX_SYS);
    }
  }
#pragma unroll 1
  for (int t = NGRP * OPER; t < 64; ++t) STEP_L(0, );
  STAMP(3);
#pragma unroll
  for (int g = NGRP; g < 2 * MFR; ++g)
    __builtin_amdgcn_raw_buffer_store_b128(epi8(accR[g >> 1][2 * (g & 1)], accR[g >> 1][2 * (g & 1) + 1], odd), rs1, o1 + (g >> 1) * 16 * OPITCH + (g & 1) * 64, 0, AUX_SYS);
  const bool bad = *(volatile LAS unsigned*)(lds + RING) != 0;
#pragma unroll
  for (int g = 0; g < 2 * MFL; ++g) {
    u32x4 ov = epi8(accL[g >> 1][2 * (g & 1)], accL[g >> 1][2 * (g & 1) + 1], odd);
    if (bad) ov = (u32x4){0x7fc07fc0u, 0x7fc07fc0u, 0x7fc07fc0u, 0x7fc07fc0u};
    __builtin_amdgcn_raw_buffer_store_b128(ov, rs2, o2 + (g >> 1) * 16 * OPITCH + (g & 1) * 64, 0, AUX_SYS);
  }
  STAMP(4);
  waitv<0>();
  BAR();
  STAMP(5);
  if (wid == 0) {
    if (!own && lane == 0) {
      __builtin_amdgcn_fence(__ATOMIC_RELEASE, "");
      waitv<0>();
      __hip_atomic_fetch_add((unsigned*)(p.ws[dest] + WS_DONE + r * 128), 1u, __ATOMIC_RELAXED, __HIP_MEMORY_SCOPE_SYSTEM);
    }
    if (c == 255) {
      unsigned* const tmo = (unsigned*)(myws + WS_TMO);
      unsigned* const fp = (unsigned*)(myws + WS_DONE + ((r + 1 + (lane < 7 ? lane : 0)) & 7) * 128);
      const unsigned need = 32u * call;
      const unsigned long long t0 = __builtin_amdgcn_s_memrealtime();
      while (!__all((int)(__hip_atomic_load(fp, __ATOMIC_RELAXED, __HIP_MEMORY_SCOPE_SYSTEM) - need) >= 0)) {
        if (__hip_atomic_load(tmo, __ATOMIC_RELAXED, __HIP_MEMORY_SCOPE_AGENT) == call || __builtin_amdgcn_s_memrealtime() - t0 > DEADLINE) {
          __hip_atomic_store(tmo, call, __ATOMIC_RELAXED, __HIP_MEMORY_SCOPE_AGENT);
          if (lane == 0) __builtin_amdgcn_raw_buffer_store_b32(0x7fc07fc0u, rs2, o2, 0, AUX_SYS);
          break;
        }
        __builtin_amdgcn_s_sleep(1);
      }
      __builtin_amdgcn_fence(__ATOMIC_ACQUIRE, "");
    }
  }
  STAMP(6);
}

extern "C" void kernel_launch(int rank, int nranks, ncclComm_t comm,
                              hipStream_t stream,
                              void* const* d_in, const int* in_sizes, int n_in,
                              void* d_out, int out_size,
                              const int* in_dtypes, int out_dtype,
                              void* const* d_out_all,
                              void* const* d_ws_all, size_t ws_bytes) {
  if (nranks != NR || (unsigned)rank >= (unsigned)NR || n_in != 2 || in_sizes[0] != MLOC * KD || in_sizes[1] != KD * NG ||
      out_size != NR * MLOC * NLOC || ws_bytes < WS_NEED) { fprintf(stderr, "gemm_a2a: unexpected problem shape\n"); return; }
  static bool attr_set[NR] = {};
  if (!attr_set[rank]) { HIP_CHECK(hipFuncSetAttribute((const void*)gemm_a2a, hipFuncAttributeMaxDynamicSharedMemorySize, LDS_BYTES)); attr_set[rank] = true; }
  Params p;
  p.x = (const char*)d_in[0]; p.w = (const char*)d_in[1]; p.rank = rank;
  for (int i = 0; i < NR; ++i) { p.out[i] = (char*)d_out_all[i]; p.ws[i] = (char*)d_ws_all[i]; }
  hipLaunchKernelGGL(gemm_a2a, dim3(256), dim3(512), LDS_BYTES, stream, p);
  HIP_CHECK(hipGetLastError());
}
```

```cpp
#include <hip/hip_runtime.h>
#include <rccl/rccl.h>
#include <cstdio>
#include <type_traits>

#define HIP_CHECK(expr) do { hipError_t _e = (expr); if (_e != hipSuccess) { \
    fprintf(stderr, "HIP error %s at %s:%d\n", hipGetErrorString(_e), __FILE__, __LINE__); \
    return; } } while (0)

#define USE_BAR
namespace {
#define LAS __attribute__((address_space(3)))
typedef short s16x4 __attribute__((ext_vector_type(4)));
typedef short s16x8 __attribute__((ext_vector_type(8)));
typedef __bf16 bf16x8 __attribute__((ext_vector_type(8)));
typedef float f32x4 __attribute__((ext_vector_type(4)));
typedef unsigned u32x4 __attribute__((ext_vector_type(4)));

constexpr int NR = 8, MLOC = 512, KD = 4096, NG = 16384, NLOC = 2048;
#define MFR_ 5
constexpr int MFR = MFR_, MFL = 8 - MFR_;
constexpr int BMR = 32 * MFR, BML = 32 * MFL;
constexpr int RROWS = 2 * BMR, FROWS = 2 * BML;
static_assert(RROWS + FROWS == MLOC, "row split");
constexpr int AMAX = BMR * 128, SLOT = AMAX + 16384, NS = 4, RING = NS * SLOT, LDS_BYTES = RING + 64 + 1024;
constexpr unsigned C_RDY = RING, C_XOK = RING + 16, C_BAD = RING + 20, C_FRE = RING + 32;
constexpr unsigned XPITCH = KD * 2, WPITCH = NG * 2, OPITCH = NLOC * 2;
constexpr unsigned OUT_BYTES = (unsigned)NR * MLOC * OPITCH;
constexpr unsigned XBLK = FROWS * XPITCH;
constexpr unsigned WS_CTR = 0;
constexpr unsigned WS_TMO = 32768;
constexpr unsigned WS_XFLAG = 36864;
constexpr unsigned WS_DONE = 40960;
constexpr unsigned WS_X = 1u << 20;
constexpr unsigned WS_NEED = WS_X + 2 * NR * XBLK;
constexpr unsigned long long DEADLINE = 40000000ull;
constexpr unsigned SPIN_CAP = 1u << 25;
constexpr int AUX_SYS = 17;

#define XPER2 9
#define OPER 7
#define OBEG 0
constexpr int NGRP = (64 - OBEG) / OPER < 2 * MFR ? (64 - OBEG) / OPER : 2 * MFR;

struct Seq { unsigned v[129]; constexpr Seq() : v() { for (int i = 0; i < 128; ++i) v[i] = i; v[128] = 1000; } };
__device__ const Seq k_seq = Seq();

struct Params { const char* x; const char* w; char* out[NR]; char* ws[NR]; int rank; };

__device__ __forceinline__ unsigned pk_bf16(float lo, float hi) {
  typedef __bf16 bf16x2 __attribute__((ext_vector_type(2)));
  bf16x2 v = {(__bf16)lo, (__bf16)hi};
  return __builtin_bit_cast(unsigned, v);
}
__device__ __forceinline__ float gelu(float y) {
  const float a = -2.0f * 0.7978845608028654f * 1.4426950408889634f, b = a * 0.044715f;
  const float e = __builtin_amdgcn_exp2f(y * __builtin_fmaf(b, y * y, a));
  return y * __builtin_amdgcn_rcpf(1.0f + e);
}
__device__ __forceinline__ u32x4 epi8(const f32x4 u, const f32x4 v, bool odd) {
  const unsigned a0 = pk_bf16(gelu(u[0]), gelu(u[1])), a1 = pk_bf16(gelu(u[2]), gelu(u[3]));
  const unsigned b0 = pk_bf16(gelu(v[0]), gelu(v[1])), b1 = pk_bf16(gelu(v[2]), gelu(v[3]));
  u32x4 w; w.x = odd ? b0 : a0; w.y = odd ? b1 : a1; w.z = odd ? a0 : b0; w.w = odd ? a1 : b1;
  return w;
}
template <int N_> __device__ __forceinline__ void waitv() { asm volatile("s_waitcnt vmcnt(%0)" :: "i"(N_) : "memory"); }
#define WAIT_L0() __builtin_amdgcn_s_waitcnt(0xC07F)
#define BAR() __builtin_amdgcn_s_barrier()
#define SCHED() __builtin_amdgcn_sched_barrier(0)
#define SGB(mask, n) __builtin_amdgcn_sched_group_barrier(mask, n, 0)

struct Frag { s16x8 a[MFR]; s16x4 blo[4], bhi[4]; };
template <int I, int N, class F> __device__ __forceinline__ void static_for(F&& f) { if constexpr (I < N) { f(std::integral_constant<int, I>{}); static_for<I + 1, N>(f); } }
__device__ __forceinline__ LAS char* lp(unsigned a) { return (LAS char*)(size_t)a; }
template <int K, int KS> __device__ __forceinline__ void rd1(Frag& f, unsigned aaddr, const unsigned (&baddr)[4], bool first = false) {
  if constexpr (K == 2) f.a[0] = *(LAS s16x8*)lp(aaddr);
  else if constexpr (K >= 9) f.a[K - 8] = *(LAS s16x8*)(lp(aaddr) + (K - 8) * 2048);
  else {
    constexpr int q = K < 2 ? K : K - 1, gn = q >> 1;
    if constexpr (q & 1) f.bhi[gn] = __builtin_amdgcn_ds_read_tr16_b64_v4i16((LAS s16x4*)(lp(baddr[gn]) + KS * 8192 + 1024));
    else f.blo[gn] = __builtin_amdgcn_ds_read_tr16_b64_v4i16((LAS s16x4*)(lp(baddr[gn]) + KS * 8192));
  }
}
template <int Z, int MF> __device__ __forceinline__ void mm1(f32x4 (&acc)[MF][4], const Frag& f) {
  constexpr int m = Z >> 2, gn = Z & 3;
  const s16x8 b = __builtin_shufflevector(f.blo[gn], f.bhi[gn], 0, 1, 2, 3, 4, 5, 6, 7);
  acc[m][gn] = __builtin_amdgcn_mfma_f32_16x16x32_bf16(__builtin_bit_cast(bf16x8, b), __builtin_bit_cast(bf16x8, f.a[m]), acc[m][gn], 0, 0, 0);
}
__device__ __forceinline__ unsigned min4(const u32x4 v) { return min(min(v.x, v.y), min(v.z, v.w)); }
__device__ __forceinline__ unsigned lds_min4(unsigned addr) {
  u32x4 v; asm volatile("ds_read_b128 %0, %1\n\ts_waitcnt lgkmcnt(0)" : "=&v"(v) : "v"(addr) : "memory");
  return __builtin_amdgcn_readfirstlane(min4(v));
}
__device__ __forceinline__ unsigned lds_get(unsigned addr) {
  unsigned v; asm volatile("ds_read_b32 %0, %1\n\ts_waitcnt lgkmcnt(0)" : "=&v"(v) : "v"(addr) : "memory");
  return __builtin_amdgcn_readfirstlane(v);
}
__device__ __forceinline__ void lds_put(unsigned addr, unsigned val) { asm volatile("ds_write_b32 %0, %1" :: "v"(addr), "v"(val) : "memory"); }
}

#define DBGW
#define STAMP(i)

__global__ __launch_bounds__(512, 2) void gemm_a2a(Params p) {
  extern __shared__ __attribute__((aligned(16))) char smem[];
  LAS char* lds = (LAS char*)smem;
  const int tid = threadIdx.x, lane = tid & 63, wid = __builtin_amdgcn_readfirstlane(tid >> 6);
  const int c = blockIdx.x, xcd = c & 7, j = c >> 3, r = p.rank;
  char* const myws = p.ws[r];
  STAMP(0);
  if (tid < 16) *(volatile LAS unsigned*)(lds + RING + tid * 4) = 0;
  WAIT_L0();
  BAR();
  const bool own = j >= 28;
  const int e1 = j >> 2, hm = j & 1, pb = (j >> 1) & 1;
  const int dest = own ? r : (r + 1 + e1) & 7;
  const int pl1 = xcd + 8 * pb;
  const int row1 = own ? 256 * hm : BMR * hm;
  const int h2 = xcd & 1, pl2 = 4 * (xcd >> 1) + (j & 3);
  const int src = own ? r : (r + 1 + e1) & 7;
  const int row2 = own ? 256 * h2 + BMR : RROWS + BML * h2;

  if (wid >= 4) {
    const int pw = wid - 4;
    unsigned vA1[MFR], vA2[MFL], vB[4];
#pragma unroll
    for (int i = 0; i < MFR; ++i) { const int row = 8 * (MFR * pw + i) + (lane >> 3); vA1[i] = row * XPITCH + (((lane & 7) ^ ((row >> 1) & 7)) << 4); }
#pragma unroll
    for (int i = 0; i < 4; ++i) { const int kr = 16 * pw + 4 * i + (lane >> 4), f = (kr & 1) | ((kr & 2) << 1) | (kr & 8); vB[i] = kr * WPITCH + (((lane & 15) ^ f) << 4); }
    const auto ra1 = __builtin_amdgcn_make_buffer_rsrc((void*)(p.x + (size_t)row1 * XPITCH), 0, BMR * XPITCH, 0x00020000);
    const auto rb1 = __builtin_amdgcn_make_buffer_rsrc((void*)(p.w + (size_t)(dest * 16 + pl1) * 256), 0, (KD - 1) * WPITCH + 256, 0x00020000);
#define PPRIO 0
    __builtin_amdgcn_s_setprio(PPRIO);
#define ROT1 1
#define WOFF 0
#define RW1 3
#define RW2 3
#define ROT2 1
    const int rot1 = ROT1 * (RW1 == 3 ? 2 * hm + (j >> 1) : 4 * hm + (j >> 1)), rot2 = ROT2 * (RW2 == 3 ? e1 + j : e1 + 2 * (j & 3)), wo = WOFF * (xcd & 3);
#define AUXA 0
#define AUXB 0
#define DSZ 16
#define PSLEEP 1
    u32x4 fr4 = {0, 0, 0, 0}; const unsigned frea = C_FRE;
    auto issue1 = [&](int s, unsigned so) {
      const int u = (s + wo) & 63;
      int kt = (u & (63 - RW1)) | ((u + rot1) & RW1);
#pragma unroll
      for (int i = 0; i < MFR; ++i) __builtin_amdgcn_raw_ptr_buffer_load_lds(ra1, lds + so + (pw * MFR + i) * 1024, DSZ, vA1[i], kt * 128, 0, AUXA);
#pragma unroll
      for (int i = 0; i < 4; ++i) __builtin_amdgcn_raw_ptr_buffer_load_lds(rb1, lds + so + AMAX + (pw * 4 + i) * 1024, DSZ, vB[i], kt * (64 * WPITCH), 0, AUXB);
    };
    issue1(0, 0); issue1(1, SLOT); issue1(2, 2 * SLOT); issue1(3, 3 * SLOT);
    unsigned callv = 0;
    { const char* ctr = myws + WS_CTR + c * 128; asm volatile("global_load_dword %0, %1, off sc1" : "+v"(callv) : "v"(ctr) : "memory"); }
#pragma unroll
    for (int i = 0; i < MFL; ++i) { const int row = 8 * (MFL * pw + i) + (lane >> 3); vA2[i] = row * XPITCH + (((lane & 7) ^ ((row >> 1) & 7)) << 4); }
    constexpr int L1 = MFR + 4, L2 = MFL + 4;
#define PT(a, b)
    waitv<3 * L1 + 1>(); BAR();
    unsigned so = 0;
    auto nxt = [&]() { so = so + SLOT == RING ? 0 : so + SLOT; };
    waitv<2 * L1 + 1>(); BAR(); issue1(4, so); nxt();
#pragma unroll 1
    for (int i = 1; i < 56; ++i) { PT(0, 2); waitv<2 * L1>(); PT(0, 0); BAR(); PT(0, 1); issue1(i + 4, so); nxt(); }
    DBGW;
    asm volatile("" : "+v"(callv));
    const unsigned call = __builtin_amdgcn_readfirstlane(callv) + 1;
    const char* a2 = own ? p.x + (size_t)row2 * XPITCH : myws + WS_X + ((call & 1) * NR + src) * XBLK + (size_t)(BML * h2) * XPITCH;
    const auto ra2 = __builtin_amdgcn_make_buffer_rsrc((void*)a2, 0, BML * XPITCH, 0x00020000);
    const auto rb2 = __builtin_amdgcn_make_buffer_rsrc((void*)(p.w + (size_t)(r * 16 + pl2) * 256), 0, (KD - 1) * WPITCH + 256, 0x00020000);
    auto issue2 = [&](int s, unsigned so) {
      const int u = (s + wo) & 63;
      int kt = (u & (63 - RW2)) | ((u + rot2) & RW2);
#pragma unroll
      for (int i = 0; i < MFL; ++i) __builtin_amdgcn_raw_ptr_buffer_load_lds(ra2, lds + so + (pw * MFL + i) * 1024, DSZ, vA2[i], kt * 128, 0, AUXA);
#pragma unroll
      for (int i = 0; i < 4; ++i) __builtin_amdgcn_raw_ptr_buffer_load_lds(rb2, lds + so + AMAX + (pw * 4 + i) * 1024, DSZ, vB[i], kt * (64 * WPITCH), 0, AUXB);
    };
    const bool poller = (pw == 0) && !own;
    unsigned* const tmo = (unsigned*)(myws + WS_TMO);
    const unsigned need = 384u * call;
    unsigned fv = 0;
    auto poll_issue = [&](int q) {
      if (poller) { const char* fp = myws + WS_XFLAG + ((src * 2 + q) * 2 + h2) * 128;
        asm volatile("global_load_dword %0, %1, off sc0 sc1" : "+v"(fv) : "v"(fp) : "memory"); }
    };
    auto poll_check = [&](int q) {
      if (poller) {
        asm volatile("" : "+v"(fv));
        if ((int)(__builtin_amdgcn_readfirstlane(fv) - need) < 0) {
          unsigned* fp = (unsigned*)(myws + WS_XFLAG + ((src * 2 + q) * 2 + h2) * 128);
          const unsigned long long t0 = __builtin_amdgcn_s_memrealtime();
          while ((int)(__builtin_amdgcn_readfirstlane(__hip_atomic_load(fp, __ATOMIC_RELAXED, __HIP_MEMORY_SCOPE_SYSTEM)) - need) < 0) {
            if (__builtin_amdgcn_readfirstlane(__hip_atomic_load(tmo, __ATOMIC_RELAXED, __HIP_MEMORY_SCOPE_AGENT)) == call
                || __builtin_amdgcn_s_memrealtime() - t0 > DEADLINE) {
              __hip_atomic_store(tmo, call, __ATOMIC_RELAXED, __HIP_MEMORY_SCOPE_AGENT);
              lds_put(C_BAD, 1);
              break;
            }
            __builtin_amdgcn_s_sleep(2);
          }
        }
        asm volatile("buffer_inv sc0 sc1" ::: "memory");
      }
    };
    waitv<2 * L1>(); BAR(); poll_issue(0); issue1(60, so); nxt();
    waitv<2 * L1>(); BAR(); issue1(61, so); nxt();
    waitv<2 * L1>(); DBGW; poll_check(0); BAR(); issue1(62, so); nxt();
    waitv<2 * L1>(); BAR(); issue1(63, so); nxt();
    waitv<2 * L1>(); BAR(); issue2(0, so); nxt();
    waitv<L1 + L2>(); BAR(); issue2(1, so); nxt();
    PT(2, 0);
#pragma unroll 1
    for (int i = 62; i < 88; ++i) { PT(1, 2); waitv<2 * L2>(); PT(1, 0); BAR(); PT(1, 1); issue2(i - 60, so); nxt(); }
    waitv<2 * L2>(); BAR(); poll_issue(1); issue2(28, so); nxt();
    waitv<2 * L2>(); BAR(); issue2(29, so); nxt();
    waitv<2 * L2>(); DBGW; poll_check(1); BAR(); issue2(30, so); nxt();
    waitv<2 * L2>(); BAR(); issue2(31, so); nxt();
    waitv<2 * L2>(); BAR(); issue2(32, so); nxt();
#pragma unroll 1
    for (int i = 93; i < 124; ++i) { waitv<2 * L2>(); BAR(); issue2(i - 60, so); nxt(); }
    waitv<2 * L2>(); BAR();
    waitv<L2>(); BAR();
    waitv<0>(); BAR();
    BAR();
    return;
  }

#define CPRIO 2
#define CSLEEP 1
  __builtin_amdgcn_s_setprio(CPRIO);
  const int wr = wid >> 1, wc = wid & 1, fr = lane & 15, fq = lane >> 4;
  const bool odd = fq & 1;
  const unsigned call = __builtin_amdgcn_readfirstlane(__hip_atomic_load((unsigned*)(myws + WS_CTR + c * 128), __ATOMIC_RELAXED, __HIP_MEMORY_SCOPE_AGENT)) + 1;
  const bool sender = wid < 3;
  const int pi = 3 * c + (sender ? wid : 0);
  const unsigned poff = (pi >> 2) * XPITCH + (pi & 3) * 1024 + lane * 16;
  const u32x4 xp0 = *(const u32x4*)(p.x + (size_t)RROWS * XPITCH + poff);
  const u32x4 xp1 = *(const u32x4*)(p.x + (size_t)RROWS * XPITCH + poff + 4096);
  const unsigned long long wsl = (unsigned long long)p.ws[(r + 1 + (lane < 7 ? lane : 0)) & 7];
  const unsigned wslo = (unsigned)wsl, wshi = (unsigned)(wsl >> 32);
  const unsigned xdst = WS_X + ((call & 1) * NR + r) * XBLK;
  const int ph = (c >> 3) + 3 * (c & 7);
  unsigned long long ev1 = 0, ev2 = 0;
  if (sender) for (int n = 0; n < 14; ++n) ev1 |= 1ull << ((n * XPER2) / 2 + ph % ((XPER2 + 1) / 2));
  for (int g = 0; g < NGRP; ++g) ev2 |= 1ull << (OBEG + g * OPER + ph % OPER);
  static_assert((13 * XPER2) / 2 + (XPER2 + 1) / 2 - 1 < 64, "x pacing");
  int xn = 0, xe = (ph + 5 * wid) % 7;
  auto xflag = [&](int q) {
    if (lane < 7) __hip_atomic_fetch_add((unsigned*)((char*)wsl + WS_XFLAG + ((r * 2 + q) * 2 + (c >> 7)) * 128), 1u, __ATOMIC_RELAXED, __HIP_MEMORY_SCOPE_SYSTEM);
  };
  auto xfwd = [&]() {
    const int q = xn >= 7;
    char* base = (char*)(((unsigned long long)(unsigned)__builtin_amdgcn_readlane(wshi, xe) << 32) | (unsigned)__builtin_amdgcn_readlane(wslo, xe));
    const auto rs = __builtin_amdgcn_make_buffer_rsrc(base + xdst, 0, (int)XBLK, 0x00020000);
    if (xn == 8) { waitv<1>(); xflag(0); }
    __builtin_amdgcn_raw_buffer_store_b128(q ? xp1 : xp0, rs, poff + q * 4096, 0, AUX_SYS);
    ++xn; xe = xe == 6 ? 0 : xe + 1;
  };

  const unsigned lb = (unsigned)(size_t)lds;
  unsigned aR[2], aL[2], baddr[4];
#pragma unroll
  for (int ks = 0; ks < 2; ++ks) { const unsigned t = lb + fr * 128 + 16 * ((4 * ks + fq) ^ ((fr >> 1) & 7)); aR[ks] = t + 16 * MFR * wr * 128; aL[ks] = t + 16 * MFL * wr * 128; }
  { const int q = (lane & 15) >> 2, pp = lane & 3, kr = 8 * fq + q, f = (kr & 1) | ((kr & 2) << 1) | (kr & 8);
#pragma unroll
    for (int gn = 0; gn < 4; ++gn) baddr[gn] = lb + AMAX + kr * 256 + 16 * ((8 * wc + 4 * (gn >> 1) + pp) ^ f) + 8 * ((gn & 1) ^ (pp & 1)); }
  const auto rs1 = __builtin_amdgcn_make_buffer_rsrc(p.out[dest], 0, (int)OUT_BYTES, 0x00020000);
  const unsigned o1 = (r * MLOC + row1 + 16 * MFR * wr + fr) * OPITCH + (pl1 * 128 + 64 * wc + 8 * fq) * 2;
  const auto rs2 = __builtin_amdgcn_make_buffer_rsrc(p.out[r], 0, (int)OUT_BYTES, 0x00020000);
  const unsigned o2 = (src * MLOC + row2 + 16 * MFL * wr + fr) * OPITCH + (pl2 * 128 + 64 * wc + 8 * fq) * 2;

  Frag F0, F1;
  f32x4 accR[MFR][4], accL[MFL][4];
#pragma unroll
  for (int m = 0; m < MFR; ++m)
#pragma unroll
    for (int gn = 0; gn < 4; ++gn) accR[m][gn] = (f32x4){0.f, 0.f, 0.f, 0.f};
  unsigned so = 0;
  auto nxt = [&]() { so = so + SLOT == RING ? 0 : so + SLOT; };
  unsigned ctl = lb + C_RDY, fre = lb + C_FRE + lane * 16 + wid * 4, sv = 0;
  asm volatile("" : "+v"(ctl), "+v"(fre), "+v"(sv));
#define rdyp ((volatile LAS u32x4*)lp(ctl))
#define frep ((volatile LAS unsigned*)lp(fre))
  auto wready = [&](unsigned n) {
    for (unsigned it = 0; __builtin_amdgcn_readfirstlane(min4(*rdyp)) < n && it < SPIN_CAP; ++it) __builtin_amdgcn_s_sleep(CSLEEP); };
  BAR();
  STAMP(1);
  unsigned bc[4], ac[2];
#pragma unroll
  for (int gn = 0; gn < 4; ++gn) bc[gn] = baddr[gn];
  ac[0] = aR[0]; ac[1] = aR[1];
  static_for<0, 8 + MFR>([&](auto K) { rd1<K, 0>(F0, ac[0], bc, true); });
#define DBL5 0
#define DBL4 1
#define DBL3 4
  unsigned rdy, thr;
  auto half_a = [&](auto MF_, auto& acc, const unsigned (&an)[2], unsigned long long& ev, unsigned want) {
    constexpr int MF = MF_, NM = 4 * MF, NRD = 8 + MF, DBL = MF == 2 ? 5 : MF == 3 ? DBL3 : MF == 4 ? DBL4 : DBL5, LAST = NRD - DBL - 1;
    u32x4 rv; unsigned t = 0;
#define CHKOFF 1
#define RVOFF 1
    constexpr int RVGAP = LAST - RVOFF < 0 ? 0 : LAST - RVOFF;
    static_for<0, NM>([&](auto Z) {
      constexpr int z = Z;
      SCHED(); mm1<z, MF>(acc, F0); SCHED();
      if constexpr (z < DBL) { rd1<2 * z, 1>(F1, ac[1], bc); rd1<2 * z + 1, 1>(F1, ac[1], bc); }
      else if constexpr (z <= LAST) rd1<z + DBL, 1>(F1, ac[1], bc);
      if constexpr (z == DBL) { nxt(); asm volatile("" : "+s"(so)); }
      if constexpr (z == DBL + 1) { thr = (ev & 1) ? 0xffffffffu : want; asm volatile("" : "+s"(thr)); }
      if constexpr (z == DBL + 2) { ev >>= 1; ++sv; asm volatile("" : "+s"(ev), "+v"(sv)); }
      if constexpr (z == LAST + 1) { bc[0] = baddr[0] + so; bc[1] = baddr[1] + so; asm volatile("" : "+v"(bc[0]), "+v"(bc[1])); }
      if constexpr (z == LAST + 2) { bc[2] = baddr[2] + so; bc[3] = baddr[3] + so; asm volatile("" : "+v"(bc[2]), "+v"(bc[3])); }
      if constexpr (z == LAST + 3) { ac[0] = an[0] + so; ac[1] = an[1] + so; asm volatile("" : "+v"(ac[0]), "+v"(ac[1])); }
      if constexpr (z == NM - 1) rdy = 1000;
    });
    static_assert(LAST + 3 < NM - 1 && DBL + 2 < LAST + 1, "half step too short");
    SCHED();
  };
  auto half_b = [&](auto MF_, auto MFN_, auto& acc) {
    constexpr int MF = MF_, MFN = MFN_, NM = 4 * MF, NRD = 8 + MFN, DBL = MF == 2 ? 5 : MF == 3 ? DBL3 : MF == 4 ? DBL4 : DBL5;
    SCHED(); WAIT_L0(); BAR();
    static_for<0, NM>([&](auto Z) {
      constexpr int z = Z;
      SCHED(); mm1<z, MF>(acc, F1); SCHED();
      if constexpr (z < DBL) { rd1<2 * z, 0>(F0, ac[0], bc); rd1<2 * z + 1, 0>(F0, ac[0], bc); }
      else if constexpr (z + DBL < NRD) rd1<z + DBL, 0>(F0, ac[0], bc);
    });
    SCHED();
  };
  constexpr std::integral_constant<int, MFR> cR{}; constexpr std::integral_constant<int, MFL> cL{};
  unsigned want = 2;
#pragma unroll 1
  do {
    half_a(cR, accR, aR, ev1, want);
    if (__builtin_expect(rdy < thr, 0)) { if (thr == 0xffffffffu) xfwd(); if (rdy < want) wready(want); }
    half_b(cR, cR, accR);
  } while (++want != 65);
  half_a(cR, accR, aL, ev1, want);
  if (__builtin_expect(rdy < thr, 0)) { if (thr == 0xffffffffu) xfwd(); if (rdy < want) wready(want); }
  half_b(cR, cL, accR);
  ++want;
  STAMP(2);
#pragma unroll
  for (int m = 0; m < MFL; ++m)
#pragma unroll
    for (int gn = 0; gn < 4; ++gn) accL[m][gn] = (f32x4){0.f, 0.f, 0.f, 0.f};
#define OUT1(g) __builtin_amdgcn_raw_buffer_store_b128(epi8(accR[(g) >> 1][2 * ((g) & 1)], accR[(g) >> 1][2 * ((g) & 1) + 1], odd), rs1, o1 + ((g) >> 1) * 16 * OPITCH + ((g) & 1) * 64, 0, AUX_SYS)
  int gi = 0;
#pragma unroll 1
  do {
    half_a(cL, accL, aL, ev2, want);
    if (__builtin_expect(rdy < thr, 0)) {
      if (thr == 0xffffffffu) {
        if (gi == 1 && sender) { waitv<1>(); xflag(1); }
        switch (gi) {
          case 0: OUT1(0); break;
          case 1: if (NGRP > 1) OUT1(1); break;
          case 2: if (NGRP > 2) OUT1(2); break;
          case 3: if (NGRP > 3) OUT1(3); break;
          case 4: if (NGRP > 4) OUT1(4); break;
          case 5: if (NGRP > 5) OUT1(5); break;
          case 6: if (NGRP > 6) OUT1(6); break;
          case 7: if (NGRP > 7) OUT1(7); break;
          case 8: if (NGRP > 8) OUT1(8); break;
          default: if (NGRP > 9) OUT1(9); break;
        }
        ++gi;
      }
      if (rdy < want) wready(want);
    }
    half_b(cL, cL, accL);
  } while (++want != 130);
  STAMP(3);
#pragma unroll
  for (int g = NGRP; g < 2 * MFR; ++g) OUT1(g);
  const bool bad = *(volatile LAS unsigned*)(lds + C_BAD) != 0;
#pragma unroll
  for (int g = 0; g < 2 * MFL; ++g) {
    u32x4 ov = epi8(accL[g >> 1][2 * (g & 1)], accL[g >> 1][2 * (g & 1) + 1], odd);
    if (bad) ov = (u32x4){0x7fc07fc0u, 0x7fc07fc0u, 0x7fc07fc0u, 0x7fc07fc0u};
    __builtin_amdgcn_raw_buffer_store_b128(ov, rs2, o2 + (g >> 1) * 16 * OPITCH + (g & 1) * 64, 0, AUX_SYS);
  }
  STAMP(4);
  waitv<2 * MFL>();
  STAMP(5);
  if (!own && lane == 0) __hip_atomic_fetch_add((unsigned*)(p.ws[dest] + WS_DONE + r * 128), 1u, __ATOMIC_RELAXED, __HIP_MEMORY_SCOPE_SYSTEM);
  if (tid == 0) __hip_atomic_store((unsigned*)(myws + WS_CTR + c * 128), call, __ATOMIC_RELAXED, __HIP_MEMORY_SCOPE_AGENT);
  if (c == 255 && wid == 0) {
    unsigned* const tmo = (unsigned*)(myws + WS_TMO);
    unsigned* const fp = (unsigned*)(myws + WS_DONE + ((r + 1 + (lane < 7 ? lane : 0)) & 7) * 128);
    const unsigned need = 128u * call;
    const unsigned long long t0 = __builtin_amdgcn_s_memrealtime();
    for (unsigned it = 1; !__all((int)(__hip_atomic_load(fp, __ATOMIC_RELAXED, __HIP_MEMORY_SCOPE_SYSTEM) - need) >= 0); ++it) {
      if ((it & 15) == 0 && (__hip_atomic_load(tmo, __ATOMIC_RELAXED, __HIP_MEMORY_SCOPE_AGENT) == call || __builtin_amdgcn_s_memrealtime() - t0 > DEADLINE)) {
        __hip_atomic_store(tmo, call, __ATOMIC_RELAXED, __HIP_MEMORY_SCOPE_AGENT);
        waitv<0>();
        if (lane == 0) __builtin_amdgcn_raw_buffer_store_b32(0x7fc07fc0u, rs2, o2, 0, AUX_SYS);
        break;
      }
      __builtin_amdgcn_s_sleep(1);
    }
  }
  STAMP(6);
}

extern "C" void kernel_launch(int rank, int nranks, ncclComm_t comm,
                              hipStream_t stream,
                              void* const* d_in, const int* in_sizes, int n_in,
                              void* d_out, int out_size,
                              const int* in_dtypes, int out_dtype,
                              void* const* d_out_all,
                              void* const* d_ws_all, size_t ws_bytes) {
  if (nranks != NR || (unsigned)rank >= (unsigned)NR || n_in != 2 || in_sizes[0] != MLOC * KD || in_sizes[1] != KD * NG ||
      out_size != NR * MLOC * NLOC || ws_bytes < WS_NEED) { fprintf(stderr, "gemm_a2a: unexpected problem shape\n"); return; }
  static bool attr_set[NR] = {};
  if (!attr_set[rank]) { HIP_CHECK(hipFuncSetAttribute((const void*)gemm_a2a, hipFuncAttributeMaxDynamicSharedMemorySize, LDS_BYTES)); attr_set[rank] = true; }
  Params p;
  p.x = (const char*)d_in[0]; p.w = (const char*)d_in[1]; p.rank = rank;
  for (int i = 0; i < NR; ++i) { p.out[i] = (char*)d_out_all[i]; p.ws[i] = (char*)d_ws_all[i]; }
  hipLaunchKernelGGL(gemm_a2a, dim3(256), dim3(512), LDS_BYTES, stream, p);
  HIP_CHECK(hipGetLastError());
}
```

```cpp
#include <hip/hip_runtime.h>
#include <rccl/rccl.h>
#include <cstdio>
#include <type_traits>

#define HIP_CHECK(expr) do { hipError_t _e = (expr); if (_e != hipSuccess) { \
    fprintf(stderr, "HIP error %s at %s:%d\n", hipGetErrorString(_e), __FILE__, __LINE__); \
    return; } } while (0)

namespace {
#define LAS __attribute__((address_space(3)))
typedef short s16x4 __attribute__((ext_vector_type(4)));
typedef short s16x8 __attribute__((ext_vector_type(8)));
typedef __bf16 bf16x8 __attribute__((ext_vector_type(8)));
typedef float f32x4 __attribute__((ext_vector_type(4)));
typedef unsigned u32x4 __attribute__((ext_vector_type(4)));
typedef unsigned long long u64;

constexpr int NR = 8, MLOC = 512, KD = 4096, NG = 16384, NLOC = 2048;
constexpr int MF1 = 5, MF2 = 4;
constexpr int RROWS = 64 * MF1, FROWS = MLOC - RROWS;
constexpr int AMAX = 32 * MF1 * 128, SLOT = AMAX + 16384, RING = 4 * SLOT, LDS_BYTES = RING;
static_assert(FROWS == 192 && LDS_BYTES <= 163840, "tiles");
constexpr unsigned XPITCH = KD * 2, WPITCH = NG * 2, OPITCH = NLOC * 2;
constexpr unsigned OUT_BYTES = (unsigned)NR * MLOC * OPITCH;
constexpr unsigned XBLK = FROWS * XPITCH;
constexpr unsigned WS_CTR = 0;
constexpr unsigned WS_TMO = 32768;
constexpr unsigned WS_XFLAG = 36864;
constexpr unsigned WS_DONE = 49152;
constexpr unsigned WS_X = 1u << 20;
constexpr unsigned WS_NEED = WS_X + NR * XBLK;
constexpr u64 DEADLINE = 40000000ull;
#define AUXO 17
constexpr int AUX_SYS = AUXO;

#define GL 7
constexpr int NIN = 64 / GL < 2 * MF1 ? 64 / GL : 2 * MF1;

struct Params { const char* x; const char* w; char* out[NR]; char* ws[NR]; int rank; };
struct Tile { int c, r, peer, own, pl1, pl2, row1, src2, row2, rot1, rot2; };

__device__ __forceinline__ unsigned pk_bf16(float lo, float hi) {
  typedef __bf16 bf16x2 __attribute__((ext_vector_type(2)));
  bf16x2 v = {(__bf16)lo, (__bf16)hi};
  return __builtin_bit_cast(unsigned, v);
}
constexpr float GELU_A = -2.0f * 0.7978845608028654f * 1.4426950408889634f, GELU_B = GELU_A * 0.044715f;
__device__ __forceinline__ float gelu(float y) {
  const float e = __builtin_amdgcn_exp2f(y * __builtin_fmaf(GELU_B, y * y, GELU_A));
  return y * __builtin_amdgcn_rcpf(1.0f + e);
}
__device__ __forceinline__ u32x4 epi8(const f32x4 u, const f32x4 v, bool odd) {
  const unsigned a0 = pk_bf16(gelu(u[0]), gelu(u[1])), a1 = pk_bf16(gelu(u[2]), gelu(u[3]));
  const unsigned b0 = pk_bf16(gelu(v[0]), gelu(v[1])), b1 = pk_bf16(gelu(v[2]), gelu(v[3]));
  u32x4 w; w.x = odd ? b0 : a0; w.y = odd ? b1 : a1; w.z = odd ? a0 : b0; w.w = odd ? a1 : b1;
  return w;
}
template <int N_> __device__ __forceinline__ void waitv() { asm volatile("s_waitcnt vmcnt(%0)" :: "i"(N_) : "memory"); }
#define WAIT_L0() __builtin_amdgcn_s_waitcnt(0xC07F)
#define BAR() __builtin_amdgcn_s_barrier()
#define SCHED() __builtin_amdgcn_sched_barrier(0)
template <int I, int N, class F> __device__ __forceinline__ void static_for(F&& f) { if constexpr (I < N) { f(std::integral_constant<int, I>{}); static_for<I + 1, N>(f); } }
__device__ __forceinline__ LAS char* lp(unsigned a) { return (LAS char*)(size_t)a; }
__device__ __forceinline__ auto rsrc(const void* base, unsigned bytes) { return __builtin_amdgcn_make_buffer_rsrc((void*)base, 0, (int)bytes, 0x00020000); }

struct Frag { s16x8 a[MF1]; s16x4 blo[4], bhi[4]; };
template <int K, int KS> __device__ __forceinline__ void rd1(Frag& f, unsigned aaddr, const unsigned (&baddr)[4]) {
  if constexpr (K == 2) f.a[0] = *(LAS s16x8*)lp(aaddr);
  else if constexpr (K >= 9) f.a[K - 8] = *(LAS s16x8*)(lp(aaddr) + (K - 8) * 2048);
  else {
    constexpr int q = K < 2 ? K : K - 1, gn = q >> 1;
    if constexpr (q & 1) f.bhi[gn] = __builtin_amdgcn_ds_read_tr16_b64_v4i16((LAS s16x4*)(lp(baddr[gn]) + KS * 8192 + 1024));
    else f.blo[gn] = __builtin_amdgcn_ds_read_tr16_b64_v4i16((LAS s16x4*)(lp(baddr[gn]) + KS * 8192));
  }
}
template <int Z, int MF> __device__ __forceinline__ void mm1(f32x4 (&acc)[MF][4], const Frag& f) {
  constexpr int m = Z >> 2, gn = Z & 3;
  const s16x8 b = __builtin_shufflevector(f.blo[gn], f.bhi[gn], 0, 1, 2, 3, 4, 5, 6, 7);
  asm volatile("v_mfma_f32_16x16x32_bf16 %0, %1, %2, %0" : "+v"(acc[m][gn]) : "v"(b), "v"(f.a[m]));
}
struct GeluState { f32x4 cu, cv; float ga, gb, ka; unsigned pk0, pk1, pk2, pk3, ov0, ov1, ov2, ov3; };
template <int F> __device__ __forceinline__ void gelu_piece(GeluState& g) {
  constexpr int pr = F / 10, st = F % 10;
  constexpr u64 ODDM = 0xffff0000ffff0000ull;
  if constexpr (F < 40) {
    const float ya = pr < 2 ? g.cu[2 * (pr & 1)] : g.cv[2 * (pr & 1)], yb = pr < 2 ? g.cu[2 * (pr & 1) + 1] : g.cv[2 * (pr & 1) + 1];
    if constexpr (st == 0) asm volatile("v_mul_f32 %0, %2, %2\n\tv_mul_f32 %1, %3, %3" : "=&v"(g.ga), "=v"(g.gb) : "v"(ya), "v"(yb));
    if constexpr (st == 1) asm volatile("v_fmamk_f32 %0, %0, %3, %2\n\tv_fmamk_f32 %1, %1, %3, %2" : "+v"(g.ga), "+v"(g.gb) : "v"(g.ka), "i"(__builtin_bit_cast(unsigned, GELU_B)));
    if constexpr (st == 2 || st == 8) asm volatile("v_mul_f32 %0, %2, %0\n\tv_mul_f32 %1, %3, %1" : "+v"(g.ga), "+v"(g.gb) : "v"(ya), "v"(yb));
    if constexpr (st == 3) asm volatile("v_exp_f32 %0, %0" : "+v"(g.ga));
    if constexpr (st == 4) asm volatile("v_exp_f32 %0, %0" : "+v"(g.gb));
    if constexpr (st == 5) asm volatile("v_add_f32 %0, 1.0, %0\n\tv_add_f32 %1, 1.0, %1" : "+v"(g.ga), "+v"(g.gb));
    if constexpr (st == 6) asm volatile("v_rcp_f32 %0, %0" : "+v"(g.ga));
    if constexpr (st == 7) asm volatile("v_rcp_f32 %0, %0" : "+v"(g.gb));
    if constexpr (st == 9) { unsigned& d = pr == 0 ? g.pk0 : pr == 1 ? g.pk1 : pr == 2 ? g.pk2 : g.pk3; asm volatile("v_cvt_pk_bf16_f32 %0, %1, %2" : "=v"(d) : "v"(g.ga), "v"(g.gb)); }
  } else if constexpr (F == 40) asm volatile("v_cndmask_b32_e64 %0, %2, %4, %6\n\tv_cndmask_b32_e64 %1, %3, %5, %6" : "=&v"(g.ov0), "=v"(g.ov1) : "v"(g.pk0), "v"(g.pk1), "v"(g.pk2), "v"(g.pk3), "s"(ODDM));
  else if constexpr (F == 41) asm volatile("v_cndmask_b32_e64 %0, %4, %2, %6\n\tv_cndmask_b32_e64 %1, %5, %3, %6" : "=&v"(g.ov2), "=v"(g.ov3) : "v"(g.pk0), "v"(g.pk1), "v"(g.pk2), "v"(g.pk3), "s"(ODDM));
}
template <int G> __device__ __forceinline__ void pick(GeluState& s, const f32x4 (&acc)[MF1][4]) {
  constexpr int m = G >> 1, h = G & 1;
#pragma unroll
  for (int i = 0; i < 4; ++i) {
    const float u0 = acc[m][2 * h][i], u1 = acc[m][2 - 2 * h][i], v0 = acc[m][2 * h + 1][i], v1 = acc[m][3 - 2 * h][i];
    s.cu[i] = __int_as_float(__builtin_amdgcn_update_dpp(__float_as_int(u0), __float_as_int(u1), 0x128, 0xf, h ? 0x3 : 0xc, false));
    s.cv[i] = __int_as_float(__builtin_amdgcn_update_dpp(__float_as_int(v0), __float_as_int(v1), 0x128, 0xf, h ? 0x3 : 0xc, false));
  }
}
constexpr int dbl_gaps(int mf) { return mf == 2 ? 5 : mf == 3 ? 4 : mf == 4 ? 1 : 0; }

#define STAMP(i)
#define NEED(n) (n)

#define XFLY 3
__device__ __forceinline__ void xpush(const Params& p, const Tile& t, int wid, int lane) {
  const u64 wsl = (u64)p.ws[(t.r + 1 + (lane < 7 ? lane : 0)) & 7];
  const int w = ((t.c >> 3) - 28) * 64 + (t.c & 7) * 8 + wid;
  u32x4 v[2][3]; unsigned po[3];
#pragma unroll
  for (int i = 0; i < 3; ++i) { const int q = 3 * w + i; po[i] = (q >> 2) * XPITCH + (q & 3) * 1024 + lane * 16;
    v[0][i] = *(const u32x4*)(p.x + (size_t)RROWS * XPITCH + po[i]); v[1][i] = *(const u32x4*)(p.x + (size_t)RROWS * XPITCH + po[i] + 4096); }
#pragma unroll
  for (int h = 0; h < 2; ++h) {
    int e = w % 7;
#pragma unroll 1
    for (int n = 0; n < 7; ++n) {
      char* base = (char*)(((u64)(unsigned)__builtin_amdgcn_readlane((unsigned)(wsl >> 32), e) << 32) | (unsigned)__builtin_amdgcn_readlane((unsigned)wsl, e));
      const auto rs = rsrc(base + WS_X + t.r * XBLK, XBLK);
#pragma unroll
      for (int i = 0; i < 3; ++i) { __builtin_amdgcn_raw_buffer_store_b128(v[h][i], rs, po[i], h * 4096, 17); waitv<XFLY>(); }
      e = e == 6 ? 0 : e + 1;
    }
    waitv<0>(); BAR();
    if (wid == 0 && lane < 7) __hip_atomic_fetch_add((unsigned*)((char*)wsl + WS_XFLAG + (t.r * 2 + h) * 128), 1u, __ATOMIC_RELAXED, __HIP_MEMORY_SCOPE_SYSTEM);
  }
}

__device__ __forceinline__ void producer(const Params& p, const Tile& t, LAS char* lds, int pw, int lane) {
  char* const myws = p.ws[t.r];
  const unsigned va0 = (lane >> 3) * XPITCH + (((lane & 7) ^ (lane >> 4)) << 4), va1 = va0 ^ 64;
  const unsigned vb0 = (lane >> 4) * WPITCH + (((lane & 15) ^ (((lane >> 4) & 1) | ((lane >> 5) << 2))) << 4), vb1 = vb0 ^ 128;
  const unsigned ve0 = va0 ^ ((pw & 1) << 6), ve1 = ve0 ^ 64;
  const auto ra1 = rsrc(p.x + (size_t)(t.row1 + 40 * pw) * XPITCH, 40 * XPITCH);
  const auto rb1 = rsrc(p.w + (size_t)(t.peer * 16 + t.pl1) * 256, (KD - 1) * WPITCH + 256);
  const int s2 = t.src2, r2 = t.row2 + 32 * (pw & 1);
  const auto ra2 = rsrc(s2 == t.r ? p.x + (size_t)r2 * XPITCH : myws + WS_X + s2 * XBLK + (size_t)(r2 - RROWS) * XPITCH, t.row2 < 0 ? 0 : 32 * XPITCH);
  const auto rb2 = rsrc(p.w + (size_t)(t.r * 16 + t.pl2) * 256, (KD - 1) * WPITCH + 256);
  unsigned so = 0;
  auto issue1 = [&](int s) {
    const int kt = (s & 60) | ((s + t.rot1) & 3);
#pragma unroll
    for (int i = 0; i < MF1; ++i) __builtin_amdgcn_raw_ptr_buffer_load_lds(ra1, lds + so + (pw * MF1 + i) * 1024, 16, (i & 1) ? ve1 : ve0, kt * 128 + i * 8 * XPITCH, 0, 0);
#pragma unroll
    for (int i = 0; i < 4; ++i) __builtin_amdgcn_raw_ptr_buffer_load_lds(rb1, lds + so + AMAX + (pw * 4 + i) * 1024, 16, (i & 2) ? vb1 : vb0, (kt * 64 + pw * 16 + i * 4) * WPITCH, 0, 0);
    so = so + SLOT == RING ? 0 : so + SLOT;
  };
  auto issue2 = [&](int s) {
    const int kt = (s & 60) | ((s + t.rot2) & 3);
#pragma unroll
    for (int i = 0; i < MF2; ++i) __builtin_amdgcn_raw_ptr_buffer_load_lds(ra2, lds + so + (pw * MF2 + i) * 1024, 16, (i & 1) ? va1 : va0, kt * 128 + i * 8 * XPITCH, 0, 0);
#pragma unroll
    for (int i = 0; i < 4; ++i) __builtin_amdgcn_raw_ptr_buffer_load_lds(rb2, lds + so + AMAX + (pw * 4 + i) * 1024, 16, (i & 2) ? vb1 : vb0, (kt * 64 + pw * 16 + i * 4) * WPITCH, 0, 0);
    so = so + SLOT == RING ? 0 : so + SLOT;
  };
  constexpr int L1 = MF1 + 4, L2 = MF2 + 4;
  unsigned callv = 0, fv = 0;
  const bool poller = (pw == 0) && !t.own;
  auto poll = [&](int q) {
    if (poller) { const char* fp = myws + WS_XFLAG + (t.peer * 2 + q) * 128; asm volatile("global_load_dword %0, %1, off sc0 sc1" : "+v"(fv) : "v"(fp) : "memory"); }
  };
  auto gate = [&](int q) {
    if (poller) {
      asm volatile("" : "+v"(fv), "+v"(callv));
      const unsigned call = __builtin_amdgcn_readfirstlane(callv) + 1, need = NEED(32u * call);
      if (__any((int)(fv - need) < 0)) {
        unsigned* const tmo = (unsigned*)(myws + WS_TMO);
        const u64 t0 = __builtin_amdgcn_s_memrealtime();
        while (__any((int)(__hip_atomic_load((unsigned*)(myws + WS_XFLAG + (t.peer * 2 + q) * 128), __ATOMIC_RELAXED, __HIP_MEMORY_SCOPE_SYSTEM) - need) < 0)) {
          if (__hip_atomic_load(tmo, __ATOMIC_RELAXED, __HIP_MEMORY_SCOPE_AGENT) == call || __builtin_amdgcn_s_memrealtime() - t0 > DEADLINE) {
            __hip_atomic_store(tmo, call, __ATOMIC_RELAXED, __HIP_MEMORY_SCOPE_AGENT);
            waitv<0>();
            break;
          }
          __builtin_amdgcn_s_sleep(2);
        }
      }
      asm volatile("buffer_inv sc0 sc1" ::: "memory");
    }
  };
  if (!t.own) {
    issue1(0); issue1(1); issue1(2); issue1(3);
    { const char* ctr = myws + WS_CTR + t.c * 128; asm volatile("global_load_dword %0, %1, off sc1" : "+v"(callv) : "v"(ctr) : "memory"); }
    waitv<3 * L1 + 1>(); BAR();
    waitv<2 * L1 + 1>(); BAR(); issue1(4);
#pragma unroll 1
    for (int i = 1; i < 60; ++i) {
      waitv<2 * L1>();
      if (i == 58) gate(0);
      BAR();
      if (i == 56) poll(0);
      issue1(i + 4);
    }
    waitv<2 * L1>(); BAR(); issue2(0);
    waitv<L1 + L2>(); BAR(); issue2(1);
    waitv<2 * L2>(); BAR(); issue2(2);
    waitv<2 * L2>(); BAR(); issue2(3);
  } else {
    waitv<0>();
    issue2(0); issue2(1); issue2(2); issue2(3);
    waitv<3 * L2>(); BAR();
  }
#pragma unroll 1
  for (int i = 0; i < 60; ++i) {
    waitv<2 * L2>();
    if (i == 26) gate(1);
    BAR();
    if (i == 24) poll(1);
    issue2(i + 4);
  }
  waitv<2 * L2>(); BAR(); waitv<L2>(); BAR(); waitv<0>(); BAR(); BAR();
  asm volatile("" :: "v"(callv), "v"(fv));
}

__device__ __forceinline__ void consumer(const Params& p, const Tile& t, LAS char* lds, int wid, int lane) {
  char* const myws = p.ws[t.r];
  __builtin_amdgcn_s_setprio(2);
  const int wr = wid >> 1, wc = wid & 1, fr = lane & 15, fq = lane >> 4;
  const bool odd = fq & 1;
  const unsigned callv = __hip_atomic_load((unsigned*)(myws + WS_CTR + t.c * 128), __ATOMIC_RELAXED, __HIP_MEMORY_SCOPE_AGENT);

  const unsigned lb = (unsigned)(size_t)lds;
  unsigned a1[2], a2[2], baddr[4];
#pragma unroll
  for (int ks = 0; ks < 2; ++ks) { const unsigned u = lb + fr * 128 + 16 * ((4 * ks + fq) ^ ((fr >> 1) & 7)); a1[ks] = u + 16 * MF1 * wr * 128; a2[ks] = u + 16 * MF2 * wr * 128; }
  { const int q = (lane & 15) >> 2, pp = lane & 3, kr = 8 * fq + q, f = (kr & 1) | ((kr & 2) << 1) | (kr & 8);
#pragma unroll
    for (int gn = 0; gn < 4; ++gn) baddr[gn] = lb + AMAX + kr * 256 + 16 * ((8 * wc + 4 * (gn >> 1) + pp) ^ f) + 8 * ((gn & 1) ^ (pp & 1)); }
  const auto rs1 = rsrc(p.out[t.peer], t.own ? 0 : OUT_BYTES);
  const unsigned o1 = (t.r * MLOC + t.row1 + 16 * MF1 * wr + (fr & 7)) * OPITCH + (t.pl1 * 128 + 64 * wc + 32 * (fr >> 3) + 8 * fq) * 2;
#define OSTEP(g) (8 * OPITCH)
#define OADDR(g) ((g) * 8 * OPITCH)
  const auto rs2 = rsrc(p.out[t.r], t.row2 < 0 ? 0 : OUT_BYTES);
  const unsigned o2 = (t.src2 * MLOC + t.row2 + fr) * OPITCH + (t.pl2 * 128 + 64 * wc + 8 * fq) * 2;

  Frag F0, F1;
  f32x4 acc1[MF1][4], acc2[MF2][4];
#pragma unroll
  for (int m = 0; m < MF1; ++m)
#pragma unroll
    for (int gn = 0; gn < 4; ++gn) { acc1[m][gn] = (f32x4){0.f, 0.f, 0.f, 0.f}; asm volatile("" : "+v"(acc1[m][gn])); }
  unsigned so = 0, bc[4], ac[2];
#pragma unroll
  for (int gn = 0; gn < 4; ++gn) bc[gn] = baddr[gn];
  ac[0] = t.own ? a2[0] : a1[0]; ac[1] = t.own ? a2[1] : a1[1];
#pragma unroll
  for (int m = 0; m < MF2; ++m)
#pragma unroll
    for (int gn = 0; gn < 4; ++gn) { acc2[m][gn] = (f32x4){0.f, 0.f, 0.f, 0.f}; asm volatile("" : "+v"(acc2[m][gn])); }
  asm volatile("s_nop 1");
  BAR();
  STAMP(1);
  static_for<0, 8 + MF1>([&](auto K) { rd1<K, 0>(F0, ac[0], bc); });
  GeluState gs; gs.ka = GELU_A; asm volatile("" : "+v"(gs.ka));
  auto half_a = [&](auto MF_, auto& acc, const unsigned (&an)[2], auto EV, auto G0_) {
    constexpr int MF = MF_, NM = 4 * MF, NRD = 8 + MF, DBL = dbl_gaps(MF), LAST = NRD - DBL - 1, G0 = G0_;
    static_for<0, NM>([&](auto Z) {
      constexpr int z = Z;
      SCHED(); mm1<z, MF>(acc, F0); SCHED();
      if constexpr (z < DBL) { rd1<2 * z, 1>(F1, ac[1], bc); rd1<2 * z + 1, 1>(F1, ac[1], bc); }
      else if constexpr (z <= LAST) rd1<z + DBL, 1>(F1, ac[1], bc);
      if constexpr (z == LAST - 1) { so = so + SLOT == RING ? 0 : so + SLOT; asm volatile("" : "+s"(so)); }
      if constexpr (z == LAST + 1) { bc[0] = baddr[0] + so; bc[1] = baddr[1] + so; asm volatile("" : "+v"(bc[0]), "+v"(bc[1])); }
      if constexpr (z == LAST + 2) { bc[2] = baddr[2] + so; bc[3] = baddr[3] + so; asm volatile("" : "+v"(bc[2]), "+v"(bc[3])); }
      if constexpr (z == LAST + 3) { ac[0] = an[0] + so; ac[1] = an[1] + so; asm volatile("" : "+v"(ac[0]), "+v"(ac[1])); }
      if constexpr (G0 >= 0 && z > LAST + 3) gelu_piece<G0 + z - LAST - 4>(gs);
    });
    static_assert(LAST + 3 < NM, "half step too short");
    SCHED();
  };
  auto half_b = [&](auto MF_, auto MFN_, auto& acc, auto ZERO, auto G0_) {
    constexpr int MF = MF_, MFN = MFN_, NM = 4 * MF, NRD = 8 + MFN, DBL = dbl_gaps(MF), G0 = G0_;
    SCHED(); WAIT_L0(); BAR();
    static_for<0, NM>([&](auto Z) {
      constexpr int z = Z;
      SCHED(); mm1<z, MF>(acc, F1); SCHED();
      if constexpr (z < DBL) { rd1<2 * z, 0>(F0, ac[0], bc); rd1<2 * z + 1, 0>(F0, ac[0], bc); }
      else if constexpr (z + DBL < NRD) rd1<z + DBL, 0>(F0, ac[0], bc);
      else if constexpr (G0 >= 0) gelu_piece<G0 + z + DBL - NRD>(gs);
    });
    SCHED();
  };
  constexpr std::integral_constant<int, MF1> c1{}; constexpr std::integral_constant<int, MF2> c2{};
  constexpr std::integral_constant<int, -1> nog{}; constexpr std::true_type yes{}; constexpr std::false_type no{};
  int it = 63;
  if (!t.own) {
#pragma unroll 1
    do {
      half_a(c1, acc1, a1, no, nog);
      half_b(c1, c1, acc1, no, nog);
      --it; asm volatile("" : "+s"(it));
    } while (it != 0);
    half_a(c1, acc1, a2, no, nog);
    half_b(c1, c2, acc1, no, nog);
    asm volatile("s_nop 15" ::: "memory"); SCHED();
  }
  so = __builtin_amdgcn_readfirstlane(so);
  STAMP(2);
  constexpr int FA = 4 * MF2 - (8 + MF2 - dbl_gaps(MF2)) - 3, FB = 4 * MF2 - (8 + MF2 - dbl_gaps(MF2));
  static_assert(GL * FB + (GL - 1) * FA >= 42, "GELU pieces do not fit");
  unsigned oo = o1;
  it = 0;
#pragma unroll 1
  do {
    half_a(c2, acc2, a2, no, nog);
    if (it) { __builtin_amdgcn_raw_buffer_store_b128((u32x4){gs.ov0, gs.ov1, gs.ov2, gs.ov3}, rs1, oo, 0, AUX_SYS); oo += OSTEP(it); }
#define PICK(g) case g: pick<(g < NIN ? g : 0)>(gs, acc1); break
    switch (it) { PICK(0); PICK(1); PICK(2); PICK(3); PICK(4); PICK(5); PICK(6); PICK(7); PICK(8); PICK(9); default: __builtin_unreachable(); }
    asm volatile("" : "+v"(gs.cu), "+v"(gs.cv));
    half_b(c2, c2, acc2, no, std::integral_constant<int, 0>{});
    static_for<1, GL>([&](auto K) { constexpr int k = K;
      half_a(c2, acc2, a2, no, std::integral_constant<int, FB + (k - 1) * (FA + FB)>{});
      half_b(c2, c2, acc2, no, std::integral_constant<int, k * (FA + FB)>{}); });
    ++it; asm volatile("" : "+s"(it));
  } while (it != NIN);
  __builtin_amdgcn_raw_buffer_store_b128((u32x4){gs.ov0, gs.ov1, gs.ov2, gs.ov3}, rs1, oo, 0, AUX_SYS);
  static_for<NIN * GL, 64>([&](auto) { half_a(c2, acc2, a2, no, nog); half_b(c2, c2, acc2, no, nog); });
  asm volatile("s_nop 15" ::: "memory"); SCHED();
  STAMP(3);
  static_for<NIN, 2 * MF1>([&](auto G) { constexpr int g = G;
    pick<g>(gs, acc1); __builtin_amdgcn_raw_buffer_store_b128(epi8(gs.cu, gs.cv, odd), rs1, o1 + OADDR(g), 0, AUX_SYS); });
  const unsigned tmov = __hip_atomic_load((unsigned*)(myws + WS_TMO), __ATOMIC_RELAXED, __HIP_MEMORY_SCOPE_AGENT);
  SCHED();
#pragma unroll
  for (int g = 0; g < 2 * MF2; ++g)
    __builtin_amdgcn_raw_buffer_store_b128(epi8(acc2[g >> 1][2 * (g & 1)], acc2[g >> 1][2 * (g & 1) + 1], odd), rs2, o2 + (g >> 1) * 16 * OPITCH + (g & 1) * 64, 0, AUX_SYS);
  STAMP(4);
  waitv<2 * MF2>();
  STAMP(5);
  const unsigned call = __builtin_amdgcn_readfirstlane(callv) + 1;
  if (!t.own && lane == 0) __hip_atomic_fetch_add((unsigned*)(p.ws[t.peer] + WS_DONE + (t.r * 4 + wid) * 128), 1u, __ATOMIC_RELAXED, __HIP_MEMORY_SCOPE_SYSTEM);
  if (wid == 0 && lane == 0) __hip_atomic_store((unsigned*)(myws + WS_CTR + t.c * 128), call, __ATOMIC_RELAXED, __HIP_MEMORY_SCOPE_AGENT);
  bool bad = tmov == call;
  if (t.c == 255 && wid == 0) {
    unsigned* const tmo = (unsigned*)(myws + WS_TMO);
    unsigned* const fp = (unsigned*)(myws + WS_DONE + (((t.r + 1 + (lane < 28 ? lane >> 2 : 0)) & 7) * 4 + (lane & 3)) * 128);
    const unsigned need = NEED(32u * call);
    const u64 t0 = __builtin_amdgcn_s_memrealtime();
    for (unsigned i = 1; __any((int)(__hip_atomic_load(fp, __ATOMIC_RELAXED, __HIP_MEMORY_SCOPE_SYSTEM) - need) < 0); ++i) {
      if ((i & 15) == 0 && (__hip_atomic_load(tmo, __ATOMIC_RELAXED, __HIP_MEMORY_SCOPE_AGENT) == call || __builtin_amdgcn_s_memrealtime() - t0 > DEADLINE)) {
        __hip_atomic_store(tmo, call, __ATOMIC_RELAXED, __HIP_MEMORY_SCOPE_AGENT);
        bad = true;
        break;
      }
      __builtin_amdgcn_s_sleep(1);
    }
  }
  if (__builtin_expect(bad, 0)) {
    waitv<0>();
    for (int g = 0; g < 2 * MF2; ++g)
      __builtin_amdgcn_raw_buffer_store_b128((u32x4){0x7fc07fc0u, 0x7fc07fc0u, 0x7fc07fc0u, 0x7fc07fc0u}, rs2, o2 + (g >> 1) * 16 * OPITCH + (g & 1) * 64, 0, AUX_SYS);
  }
  STAMP(6);
}
}

__global__ __launch_bounds__(512, 2) void gemm_a2a(Params p) {
  extern __shared__ __attribute__((aligned(16))) char smem[];
  LAS char* lds = (LAS char*)smem;
  const int lane = threadIdx.x & 63, wid = __builtin_amdgcn_readfirstlane(threadIdx.x >> 6);
  Tile t;
  t.c = blockIdx.x; t.r = p.rank;
  const int xcd = t.c & 7, j = t.c >> 3, e = j >> 2, hm = j & 1, pb = (j >> 1) & 1, h2 = xcd & 1;
  t.own = j >= 28;
  t.peer = (t.r + 1 + e) & 7;
  t.pl1 = xcd + 8 * pb;
  t.pl2 = 4 * (xcd >> 1) + (j & 3);
  t.row1 = 32 * MF1 * hm;
  if (((wid >> 1) & 1) == 0) { t.src2 = t.peer; t.row2 = t.own ? 128 * h2 : RROWS + 128 * h2; }
  else { t.src2 = h2 ? t.r : t.peer; t.row2 = t.own ? 128 * h2 + 64 : h2 ? (e < 4 ? 256 + 64 * e : -1) : RROWS + 64; }
  t.rot1 = 2 * hm + (j >> 1); t.rot2 = e + j;
  STAMP(0);
  if (t.own) xpush(p, t, wid, lane);
  if (wid >= 4) producer(p, t, lds, wid - 4, lane); else consumer(p, t, lds, wid, lane);
}

extern "C" void kernel_launch(int rank, int nranks, ncclComm_t comm,
                              hipStream_t stream,
                              void* const* d_in, const int* in_sizes, int n_in,
                              void* d_out, int out_size,
                              const int* in_dtypes, int out_dtype,
                              void* const* d_out_all,
                              void* const* d_ws_all, size_t ws_bytes) {
  if (nranks != NR || (unsigned)rank >= (unsigned)NR || n_in != 2 || in_sizes[0] != MLOC * KD || in_sizes[1] != KD * NG ||
      out_size != NR * MLOC * NLOC || ws_bytes < WS_NEED) { fprintf(stderr, "gemm_a2a: unexpected problem shape\n"); return; }
  static bool attr_set[NR] = {};
  if (!attr_set[rank]) { HIP_CHECK(hipFuncSetAttribute((const void*)gemm_a2a, hipFuncAttributeMaxDynamicSharedMemorySize, LDS_BYTES)); attr_set[rank] = true; }
  Params p;
  p.x = (const char*)d_in[0]; p.w = (const char*)d_in[1]; p.rank = rank;
  for (int i = 0; i < NR; ++i) { p.out[i] = (char*)d_out_all[i]; p.ws[i] = (char*)d_ws_all[i]; }
  hipLaunchKernelGGL(gemm_a2a, dim3(256), dim3(512), LDS_BYTES, stream, p);
  HIP_CHECK(hipGetLastError());
}
```

```cpp
#include <hip/hip_runtime.h>
#include <rccl/rccl.h>
#include <cstdio>
#include <type_traits>

#define HIP_CHECK(expr) do { hipError_t _e = (expr); if (_e != hipSuccess) { \
    fprintf(stderr, "HIP error %s at %s:%d\n", hipGetErrorString(_e), __FILE__, __LINE__); \
    return; } } while (0)

namespace {
#define LAS __attribute__((address_space(3)))
typedef short s16x4 __attribute__((ext_vector_type(4)));
typedef short s16x8 __attribute__((ext_vector_type(8)));
typedef __bf16 bf16x8 __attribute__((ext_vector_type(8)));
typedef float f32x4 __attribute__((ext_vector_type(4)));
typedef unsigned u32x4 __attribute__((ext_vector_type(4)));
typedef unsigned long long u64;

constexpr int NR = 8, MLOC = 512, KD = 4096, NG = 16384, NLOC = 2048;
constexpr int MF1 = 5, MF2 = 4;
constexpr int RROWS = 64 * MF1, FROWS = MLOC - RROWS;
constexpr int AMAX = 32 * MF1 * 128, SLOT = AMAX + 16384, RING = 4 * SLOT, LDS_BYTES = RING;
static_assert(FROWS == 192 && LDS_BYTES <= 163840, "tiles");
constexpr unsigned XPITCH = KD * 2, WPITCH = NG * 2, OPITCH = NLOC * 2;
constexpr unsigned OUT_BYTES = (unsigned)NR * MLOC * OPITCH;
constexpr unsigned XBLK = FROWS * XPITCH;
constexpr unsigned WS_CTR = 0;
constexpr unsigned WS_TMO = 32768;
constexpr unsigned WS_XFLAG = 36864;
constexpr unsigned WS_DONE = 49152;
constexpr unsigned WS_X = 1u << 20;
constexpr unsigned WS_NEED = WS_X + NR * XBLK;
constexpr u64 DEADLINE = 40000000ull;
#define AUXO 17
constexpr int AUX_SYS = AUXO;

#define GL 6
constexpr int NIN = 63 / GL < 2 * MF1 ? 63 / GL : 2 * MF1;

struct Params { const char* x; const char* w; char* out[NR]; char* ws[NR]; int rank; };
struct Tile { int c, r, peer, own, pl1, pl2, row1, src2, row2, rot1, rot2; };

__device__ __forceinline__ unsigned pk_bf16(float lo, float hi) {
  typedef __bf16 bf16x2 __attribute__((ext_vector_type(2)));
  bf16x2 v = {(__bf16)lo, (__bf16)hi};
  return __builtin_bit_cast(unsigned, v);
}
constexpr float GELU_A = -2.0f * 0.7978845608028654f * 1.4426950408889634f, GELU_B = GELU_A * 0.044715f;
__device__ __forceinline__ float gelu(float y) {
  const float e = __builtin_amdgcn_exp2f(y * __builtin_fmaf(GELU_B, y * y, GELU_A));
  return y * __builtin_amdgcn_rcpf(1.0f + e);
}
__device__ __forceinline__ u32x4 epi8(const f32x4 u, const f32x4 v, bool odd) {
  const unsigned a0 = pk_bf16(gelu(u[0]), gelu(u[1])), a1 = pk_bf16(gelu(u[2]), gelu(u[3]));
  const unsigned b0 = pk_bf16(gelu(v[0]), gelu(v[1])), b1 = pk_bf16(gelu(v[2]), gelu(v[3]));
  u32x4 w; w.x = odd ? b0 : a0; w.y = odd ? b1 : a1; w.z = odd ? a0 : b0; w.w = odd ? a1 : b1;
  return w;
}
template <int N_> __device__ __forceinline__ void waitv() { asm volatile("s_waitcnt vmcnt(%0)" :: "i"(N_) : "memory"); }
#define WAIT_L0() __builtin_amdgcn_s_waitcnt(0xC07F)
#define BAR() __builtin_amdgcn_s_barrier()
#define SCHED() __builtin_amdgcn_sched_barrier(0)
template <int I, int N, class F> __device__ __forceinline__ void static_for(F&& f) { if constexpr (I < N) { f(std::integral_constant<int, I>{}); static_for<I + 1, N>(f); } }
__device__ __forceinline__ LAS char* lp(unsigned a) { return (LAS char*)(size_t)a; }
__device__ __forceinline__ auto rsrc(const void* base, unsigned bytes) { return __builtin_amdgcn_make_buffer_rsrc((void*)base, 0, (int)bytes, 0x00020000); }

struct Frag { s16x8 a[MF1]; s16x4 blo[4], bhi[4]; };
template <int K, int KS> __device__ __forceinline__ void rd1(Frag& f, unsigned aaddr, const unsigned (&baddr)[4]) {
  if constexpr (K == 2) f.a[0] = *(LAS s16x8*)lp(aaddr);
  else if constexpr (K >= 9) f.a[K - 8] = *(LAS s16x8*)(lp(aaddr) + (K - 8) * 2048);
  else {
    constexpr int q = K < 2 ? K : K - 1, gn = q >> 1;
    if constexpr (q & 1) f.bhi[gn] = __builtin_amdgcn_ds_read_tr16_b64_v4i16((LAS s16x4*)(lp(baddr[gn]) + KS * 8192 + 1024));
    else f.blo[gn] = __builtin_amdgcn_ds_read_tr16_b64_v4i16((LAS s16x4*)(lp(baddr[gn]) + KS * 8192));
  }
}
template <int Z, int MF> __device__ __forceinline__ void mm1(f32x4 (&acc)[MF][4], const Frag& f) {
  constexpr int m = Z >> 2, gn = Z & 3;
  const s16x8 b = __builtin_shufflevector(f.blo[gn], f.bhi[gn], 0, 1, 2, 3, 4, 5, 6, 7);
  asm volatile("v_mfma_f32_16x16x32_bf16 %0, %1, %2, %0" : "+v"(acc[m][gn]) : "v"(b), "v"(f.a[m]));
}
struct GeluState { f32x4 cu, cv; float ga, gb, ka; unsigned pk0, pk1, pk2, pk3, ov0, ov1, ov2, ov3; };
template <int F> __device__ __forceinline__ void gelu_piece(GeluState& g) {
  constexpr int pr = F / 10, st = F % 10;
  constexpr u64 ODDM = 0xffff0000ffff0000ull;
  if constexpr (F < 40) {
    const float ya = pr < 2 ? g.cu[2 * (pr & 1)] : g.cv[2 * (pr & 1)], yb = pr < 2 ? g.cu[2 * (pr & 1) + 1] : g.cv[2 * (pr & 1) + 1];
    if constexpr (st == 0) asm volatile("v_mul_f32 %0, %2, %2\n\tv_mul_f32 %1, %3, %3" : "=&v"(g.ga), "=v"(g.gb) : "v"(ya), "v"(yb));
    if constexpr (st == 1) asm volatile("v_fmamk_f32 %0, %0, %3, %2\n\tv_fmamk_f32 %1, %1, %3, %2" : "+v"(g.ga), "+v"(g.gb) : "v"(g.ka), "i"(__builtin_bit_cast(unsigned, GELU_B)));
    if constexpr (st == 2 || st == 8) asm volatile("v_mul_f32 %0, %2, %0\n\tv_mul_f32 %1, %3, %1" : "+v"(g.ga), "+v"(g.gb) : "v"(ya), "v"(yb));
    if constexpr (st == 3) asm volatile("v_exp_f32 %0, %0" : "+v"(g.ga));
    if constexpr (st == 4) asm volatile("v_exp_f32 %0, %0" : "+v"(g.gb));
    if constexpr (st == 5) asm volatile("v_add_f32 %0, 1.0, %0\n\tv_add_f32 %1, 1.0, %1" : "+v"(g.ga), "+v"(g.gb));
    if constexpr (st == 6) asm volatile("v_rcp_f32 %0, %0" : "+v"(g.ga));
    if constexpr (st == 7) asm volatile("v_rcp_f32 %0, %0" : "+v"(g.gb));
    if constexpr (st == 9) { unsigned& d = pr == 0 ? g.pk0 : pr == 1 ? g.pk1 : pr == 2 ? g.pk2 : g.pk3; asm volatile("v_cvt_pk_bf16_f32 %0, %1, %2" : "=v"(d) : "v"(g.ga), "v"(g.gb)); }
  } else if constexpr (F == 40) asm volatile("v_cndmask_b32_e64 %0, %2, %4, %6\n\tv_cndmask_b32_e64 %1, %3, %5, %6" : "=&v"(g.ov0), "=v"(g.ov1) : "v"(g.pk0), "v"(g.pk1), "v"(g.pk2), "v"(g.pk3), "s"(ODDM));
  else if constexpr (F == 41) asm volatile("v_cndmask_b32_e64 %0, %4, %2, %6\n\tv_cndmask_b32_e64 %1, %5, %3, %6" : "=&v"(g.ov2), "=v"(g.ov3) : "v"(g.pk0), "v"(g.pk1), "v"(g.pk2), "v"(g.pk3), "s"(ODDM));
}
template <int G> __device__ __forceinline__ void pick(GeluState& s, const f32x4 (&acc)[MF1][4]) {
  constexpr int m = G >> 1, h = G & 1;
#pragma unroll
  for (int i = 0; i < 4; ++i) {
    const float u0 = acc[m][2 * h][i], u1 = acc[m][2 - 2 * h][i], v0 = acc[m][2 * h + 1][i], v1 = acc[m][3 - 2 * h][i];
    s.cu[i] = __int_as_float(__builtin_amdgcn_update_dpp(__float_as_int(u0), __float_as_int(u1), 0x128, 0xf, h ? 0x3 : 0xc, false));
    s.cv[i] = __int_as_float(__builtin_amdgcn_update_dpp(__float_as_int(v0), __float_as_int(v1), 0x128, 0xf, h ? 0x3 : 0xc, false));
  }
}
constexpr int dbl_gaps(int mf) { return mf == 2 ? 5 : mf == 3 ? 4 : mf == 4 ? 1 : 0; }

#define STAMP(i)
#define NEED(n) (n)

#define XFLY 3
__device__ __forceinline__ void xpush(const Params& p, const Tile& t, int wid, int lane) {
  const u64 wsl = (u64)p.ws[(t.r + 1 + (lane < 7 ? lane : 0)) & 7];
  const int w = ((t.c >> 3) - 28) * 64 + (t.c & 7) * 8 + wid;
  u32x4 v[2][3]; unsigned po[3];
#pragma unroll
  for (int i = 0; i < 3; ++i) { const int q = 3 * w + i; po[i] = (q >> 2) * XPITCH + (q & 3) * 1024 + lane * 16;
    v[0][i] = *(const u32x4*)(p.x + (size_t)RROWS * XPITCH + po[i]); v[1][i] = *(const u32x4*)(p.x + (size_t)RROWS * XPITCH + po[i] + 4096); }
#pragma unroll
  for (int h = 0; h < 2; ++h) {
    int e = w % 7;
#pragma unroll 1
    for (int n = 0; n < 7; ++n) {
      char* base = (char*)(((u64)(unsigned)__builtin_amdgcn_readlane((unsigned)(wsl >> 32), e) << 32) | (unsigned)__builtin_amdgcn_readlane((unsigned)wsl, e));
      const auto rs = rsrc(base + WS_X + t.r * XBLK, XBLK);
#pragma unroll
      for (int i = 0; i < 3; ++i) { __builtin_amdgcn_raw_buffer_store_b128(v[h][i], rs, po[i], h * 4096, 17); waitv<XFLY>(); }
      e = e == 6 ? 0 : e + 1;
    }
    waitv<0>(); BAR();
    if (wid == 0 && lane < 7) __hip_atomic_fetch_add((unsigned*)((char*)wsl + WS_XFLAG + (t.r * 2 + h) * 128), 1u, __ATOMIC_RELAXED, __HIP_MEMORY_SCOPE_SYSTEM);
  }
}

__device__ __forceinline__ void producer(const Params& p, const Tile& t, LAS char* lds, int pw, int lane) {
  char* const myws = p.ws[t.r];
  const unsigned va0 = (lane >> 3) * XPITCH + (((lane & 7) ^ (lane >> 4)) << 4), va1 = va0 ^ 64;
  const unsigned vb0 = (lane >> 4) * WPITCH + (((lane & 15) ^ (((lane >> 4) & 1) | ((lane >> 5) << 2))) << 4), vb1 = vb0 ^ 128;
  const unsigned ve0 = va0 ^ ((pw & 1) << 6), ve1 = ve0 ^ 64;
  const auto ra1 = rsrc(p.x + (size_t)(t.row1 + 40 * pw) * XPITCH, 40 * XPITCH);
  const auto rb1 = rsrc(p.w + (size_t)(t.peer * 16 + t.pl1) * 256, (KD - 1) * WPITCH + 256);
  const int s2 = t.src2, r2 = t.row2 + 32 * (pw & 1);
  const auto ra2 = rsrc(s2 == t.r ? p.x + (size_t)r2 * XPITCH : myws + WS_X + s2 * XBLK + (size_t)(r2 - RROWS) * XPITCH, t.row2 < 0 ? 0 : 32 * XPITCH);
  const auto rb2 = rsrc(p.w + (size_t)(t.r * 16 + t.pl2) * 256, (KD - 1) * WPITCH + 256);
  unsigned so = 0;
  auto issue1 = [&](int s) {
    const int kt = (s & 60) | ((s + t.rot1) & 3);
#pragma unroll
    for (int i = 0; i < MF1; ++i) __builtin_amdgcn_raw_ptr_buffer_load_lds(ra1, lds + so + (pw * MF1 + i) * 1024, 16, (i & 1) ? ve1 : ve0, kt * 128 + i * 8 * XPITCH, 0, 0);
#pragma unroll
    for (int i = 0; i < 4; ++i) __builtin_amdgcn_raw_ptr_buffer_load_lds(rb1, lds + so + AMAX + (pw * 4 + i) * 1024, 16, (i & 2) ? vb1 : vb0, (kt * 64 + pw * 16 + i * 4) * WPITCH, 0, 0);
    so = so + SLOT == RING ? 0 : so + SLOT;
  };
  auto issue2 = [&](int s) {
    const int kt = (s & 60) | ((s + t.rot2) & 3);
#pragma unroll
    for (int i = 0; i < MF2; ++i) __builtin_amdgcn_raw_ptr_buffer_load_lds(ra2, lds + so + (pw * MF2 + i) * 1024, 16, (i & 1) ? va1 : va0, kt * 128 + i * 8 * XPITCH, 0, 0);
#pragma unroll
    for (int i = 0; i < 4; ++i) __builtin_amdgcn_raw_ptr_buffer_load_lds(rb2, lds + so + AMAX + (pw * 4 + i) * 1024, 16, (i & 2) ? vb1 : vb0, (kt * 64 + pw * 16 + i * 4) * WPITCH, 0, 0);
    so = so + SLOT == RING ? 0 : so + SLOT;
  };
  constexpr int L1 = MF1 + 4, L2 = MF2 + 4;
  unsigned callv = 0, fv = 0;
  const bool poller = (pw == 0) && !t.own;
  auto poll = [&](int q) {
    if (poller) { const char* fp = myws + WS_XFLAG + (t.peer * 2 + q) * 128; asm volatile("global_load_dword %0, %1, off sc0 sc1" : "+v"(fv) : "v"(fp) : "memory"); }
  };
  auto gate = [&](int q) {
    if (poller) {
      asm volatile("" : "+v"(fv), "+v"(callv));
      const unsigned call = __builtin_amdgcn_readfirstlane(callv) + 1, need = NEED(32u * call);
      if (__any((int)(fv - need) < 0)) {
        unsigned* const tmo = (unsigned*)(myws + WS_TMO);
        const u64 t0 = __builtin_amdgcn_s_memrealtime();
        while (__any((int)(__hip_atomic_load((unsigned*)(myws + WS_XFLAG + (t.peer * 2 + q) * 128), __ATOMIC_RELAXED, __HIP_MEMORY_SCOPE_SYSTEM) - need) < 0)) {
          if (__hip_atomic_load(tmo, __ATOMIC_RELAXED, __HIP_MEMORY_SCOPE_AGENT) == call || __builtin_amdgcn_s_memrealtime() - t0 > DEADLINE) {
            __hip_atomic_store(tmo, call, __ATOMIC_RELAXED, __HIP_MEMORY_SCOPE_AGENT);
            waitv<0>();
            break;
          }
          __builtin_amdgcn_s_sleep(2);
        }
      }
      asm volatile("buffer_inv sc0 sc1" ::: "memory");
    }
  };
  if (!t.own) {
    issue1(0); issue1(1); issue1(2); issue1(3);
    { const char* ctr = myws + WS_CTR + t.c * 128; asm volatile("global_load_dword %0, %1, off sc1" : "+v"(callv) : "v"(ctr) : "memory"); }
    waitv<3 * L1 + 1>(); BAR();
    waitv<2 * L1 + 1>(); BAR(); issue1(4);
#pragma unroll 1
    for (int i = 1; i < 60; ++i) {
      waitv<2 * L1>();
      if (i == 58) gate(0);
      BAR();
      if (i == 56) poll(0);
      issue1(i + 4);
    }
    waitv<2 * L1>(); BAR(); issue2(0);
    waitv<L1 + L2>(); BAR(); issue2(1);
    waitv<2 * L2>(); BAR(); issue2(2);
    waitv<2 * L2>(); BAR(); issue2(3);
  } else {
    waitv<0>();
    issue2(0); issue2(1); issue2(2); issue2(3);
    waitv<3 * L2>(); BAR();
  }
#pragma unroll 1
  for (int i = 0; i < 60; ++i) {
    waitv<2 * L2>();
    if (i == 26) gate(1);
    BAR();
    if (i == 24) poll(1);
    issue2(i + 4);
  }
  waitv<2 * L2>(); BAR(); waitv<L2>(); BAR(); waitv<0>(); BAR(); BAR();
  asm volatile("" :: "v"(callv), "v"(fv));
}

__device__ __forceinline__ void consumer(const Params& p, const Tile& t, LAS char* lds, int wid, int lane) {
  char* const myws = p.ws[t.r];
  __builtin_amdgcn_s_setprio(2);
  const int wr = wid >> 1, wc = wid & 1, fr = lane & 15, fq = lane >> 4;
  const bool odd = fq & 1;
  const unsigned callv = __hip_atomic_load((unsigned*)(myws + WS_CTR + t.c * 128), __ATOMIC_RELAXED, __HIP_MEMORY_SCOPE_AGENT);

  const unsigned lb = (unsigned)(size_t)lds;
  unsigned a1[2], a2[2], baddr[4];
#pragma unroll
  for (int ks = 0; ks < 2; ++ks) { const unsigned u = lb + fr * 128 + 16 * ((4 * ks + fq) ^ ((fr >> 1) & 7)); a1[ks] = u + 16 * MF1 * wr * 128; a2[ks] = u + 16 * MF2 * wr * 128; }
  { const int q = (lane & 15) >> 2, pp = lane & 3, kr = 8 * fq + q, f = (kr & 1) | ((kr & 2) << 1) | (kr & 8);
#pragma unroll
    for (int gn = 0; gn < 4; ++gn) baddr[gn] = lb + AMAX + kr * 256 + 16 * ((8 * wc + 4 * (gn >> 1) + pp) ^ f) + 8 * ((gn & 1) ^ (pp & 1)); }
  const auto rs1 = rsrc(p.out[t.peer], t.own ? 0 : OUT_BYTES);
  const unsigned o1 = (t.r * MLOC + t.row1 + 16 * MF1 * wr + (fr & 7)) * OPITCH + (t.pl1 * 128 + 64 * wc + 32 * (fr >> 3) + 8 * fq) * 2;
#define OSTEP(g) (8 * OPITCH)
#define OADDR(g) ((g) * 8 * OPITCH)
  const auto rs2 = rsrc(p.out[t.r], t.row2 < 0 ? 0 : OUT_BYTES);
  const unsigned o2 = (t.src2 * MLOC + t.row2 + fr) * OPITCH + (t.pl2 * 128 + 64 * wc + 8 * fq) * 2;

  Frag F0, F1;
  f32x4 acc1[MF1][4], acc2[MF2][4];
#pragma unroll
  for (int m = 0; m < MF1; ++m)
#pragma unroll
    for (int gn = 0; gn < 4; ++gn) { acc1[m][gn] = (f32x4){0.f, 0.f, 0.f, 0.f}; asm volatile("" : "+v"(acc1[m][gn])); }
  unsigned so = 0, bc[4], ac[2];
#pragma unroll
  for (int gn = 0; gn < 4; ++gn) bc[gn] = baddr[gn];
  ac[0] = t.own ? a2[0] : a1[0]; ac[1] = t.own ? a2[1] : a1[1];
#pragma unroll
  for (int m = 0; m < MF2; ++m)
#pragma unroll
    for (int gn = 0; gn < 4; ++gn) { acc2[m][gn] = (f32x4){0.f, 0.f, 0.f, 0.f}; asm volatile("" : "+v"(acc2[m][gn])); }
  asm volatile("s_nop 1");
  BAR();
  STAMP(1);
  static_for<0, 8 + MF1>([&](auto K) { rd1<K, 0>(F0, ac[0], bc); });
  GeluState gs; gs.ka = GELU_A; asm volatile("" : "+v"(gs.ka));
  auto half_a = [&](auto MF_, auto& acc, const unsigned (&an)[2], auto EV, auto G0_) {
    constexpr int MF = MF_, NM = 4 * MF, NRD = 8 + MF, DBL = dbl_gaps(MF), LAST = NRD - DBL - 1, G0 = G0_;
    static_for<0, NM>([&](auto Z) {
      constexpr int z = Z;
      SCHED(); mm1<z, MF>(acc, F0); SCHED();
      if constexpr (z < DBL) { rd1<2 * z, 1>(F1, ac[1], bc); rd1<2 * z + 1, 1>(F1, ac[1], bc); }
      else if constexpr (z <= LAST) rd1<z + DBL, 1>(F1, ac[1], bc);
      if constexpr (z == LAST - 1) { so = so + SLOT == RING ? 0 : so + SLOT; asm volatile("" : "+s"(so)); }
      if constexpr (z == LAST + 1) { bc[0] = baddr[0] + so; bc[1] = baddr[1] + so; asm volatile("" : "+v"(bc[0]), "+v"(bc[1])); }
      if constexpr (z == LAST + 2) { bc[2] = baddr[2] + so; bc[3] = baddr[3] + so; asm volatile("" : "+v"(bc[2]), "+v"(bc[3])); }
      if constexpr (z == LAST + 3) { ac[0] = an[0] + so; ac[1] = an[1] + so; asm volatile("" : "+v"(ac[0]), "+v"(ac[1])); }
      if constexpr (G0 >= 0 && z > LAST + 3) gelu_piece<G0 + z - LAST - 4>(gs);
    });
    static_assert(LAST + 3 < NM, "half step too short");
    SCHED();
  };
  auto half_b = [&](auto MF_, auto MFN_, auto& acc, auto ZERO, auto G0_) {
    constexpr int MF = MF_, MFN = MFN_, NM = 4 * MF, NRD = 8 + MFN, DBL = dbl_gaps(MF), G0 = G0_;
    SCHED(); WAIT_L0(); BAR();
    static_for<0, NM>([&](auto Z) {
      constexpr int z = Z;
      SCHED(); mm1<z, MF>(acc, F1); SCHED();
      if constexpr (z < DBL) { rd1<2 * z, 0>(F0, ac[0], bc); rd1<2 * z + 1, 0>(F0, ac[0], bc); }
      else if constexpr (z + DBL < NRD) rd1<z + DBL, 0>(F0, ac[0], bc);
      else if constexpr (G0 >= 0) gelu_piece<G0 + z + DBL - NRD>(gs);
    });
    SCHED();
  };
  constexpr std::integral_constant<int, MF1> c1{}; constexpr std::integral_constant<int, MF2> c2{};
  constexpr std::integral_constant<int, -1> nog{}; constexpr std::true_type yes{}; constexpr std::false_type no{};
  int it = 63;
  if (!t.own) {
#pragma unroll 1
    do {
      half_a(c1, acc1, a1, no, nog);
      half_b(c1, c1, acc1, no, nog);
      --it; asm volatile("" : "+s"(it));
    } while (it != 0);
    half_a(c1, acc1, a2, no, nog);
    half_b(c1, c2, acc1, no, nog);
    asm volatile("s_nop 15" ::: "memory"); SCHED();
  }
  so = __builtin_amdgcn_readfirstlane(so);
  STAMP(2);
  constexpr int FA = 4 * MF2 - (8 + MF2 - dbl_gaps(MF2)) - 3, FB = 4 * MF2 - (8 + MF2 - dbl_gaps(MF2));
  static_assert(GL * (FA + FB) >= 42 && NIN * GL < 64, "GELU pieces do not fit");
  constexpr std::integral_constant<int, GL * FB + (GL - 1) * FA> gtail{};
  unsigned oo = o1;
  it = 0;
#pragma unroll 1
  do {
    half_a(c2, acc2, a2, no, gtail);
    if (it) { __builtin_amdgcn_raw_buffer_store_b128((u32x4){gs.ov0, gs.ov1, gs.ov2, gs.ov3}, rs1, oo, 0, AUX_SYS); oo += OSTEP(it); }
#define PICK(g) case g: pick<(g < NIN ? g : 0)>(gs, acc1); break
    switch (it) { PICK(0); PICK(1); PICK(2); PICK(3); PICK(4); PICK(5); PICK(6); PICK(7); PICK(8); PICK(9); default: __builtin_unreachable(); }
    asm volatile("" : "+v"(gs.cu), "+v"(gs.cv));
    half_b(c2, c2, acc2, no, std::integral_constant<int, 0>{});
    static_for<1, GL>([&](auto K) { constexpr int k = K;
      half_a(c2, acc2, a2, no, std::integral_constant<int, FB + (k - 1) * (FA + FB)>{});
      half_b(c2, c2, acc2, no, std::integral_constant<int, k * (FA + FB)>{}); });
    ++it; asm volatile("" : "+s"(it));
  } while (it != NIN);
  half_a(c2, acc2, a2, no, gtail);
  __builtin_amdgcn_raw_buffer_store_b128((u32x4){gs.ov0, gs.ov1, gs.ov2, gs.ov3}, rs1, oo, 0, AUX_SYS);
  half_b(c2, c2, acc2, no, nog);
  if constexpr (NIN * GL + 1 < 64) {
    it = 63 - NIN * GL;
#pragma unroll 1
    do { half_a(c2, acc2, a2, no, nog); half_b(c2, c2, acc2, no, nog); --it; asm volatile("" : "+s"(it)); } while (it != 0);
  }
  asm volatile("s_nop 15" ::: "memory"); SCHED();
  STAMP(3);
  static_for<NIN, 2 * MF1>([&](auto G) { constexpr int g = G;
    pick<g>(gs, acc1); __builtin_amdgcn_raw_buffer_store_b128(epi8(gs.cu, gs.cv, odd), rs1, o1 + OADDR(g), 0, AUX_SYS); });
  const unsigned tmov = __hip_atomic_load((unsigned*)(myws + WS_TMO), __ATOMIC_RELAXED, __HIP_MEMORY_SCOPE_AGENT);
  const unsigned call = __builtin_amdgcn_readfirstlane(callv) + 1;
  SCHED();
#define DONEAT 4
  static_for<0, 2 * MF2>([&](auto G) { constexpr int g = G;
    if constexpr (g == DONEAT) {
      SCHED(); waitv<DONEAT>();
      if (!t.own && lane == 0) __hip_atomic_fetch_add((unsigned*)(p.ws[t.peer] + WS_DONE + (t.r * 4 + wid) * 128), 1u, __ATOMIC_RELAXED, __HIP_MEMORY_SCOPE_SYSTEM);
      SCHED(); STAMP(4);
    }
    __builtin_amdgcn_raw_buffer_store_b128(epi8(acc2[g >> 1][2 * (g & 1)], acc2[g >> 1][2 * (g & 1) + 1], odd), rs2, o2 + (g >> 1) * 16 * OPITCH + (g & 1) * 64, 0, AUX_SYS); });
  STAMP(5);
  if (wid == 0 && lane == 0) __hip_atomic_store((unsigned*)(myws + WS_CTR + t.c * 128), call, __ATOMIC_RELAXED, __HIP_MEMORY_SCOPE_AGENT);
  bool bad = tmov == call;
  if (t.c == 255 && wid == 0) {
    unsigned* const tmo = (unsigned*)(myws + WS_TMO);
    unsigned* const fp = (unsigned*)(myws + WS_DONE + (((t.r + 1 + (lane < 28 ? lane >> 2 : 0)) & 7) * 4 + (lane & 3)) * 128);
    const unsigned need = NEED(32u * call);
    const u64 t0 = __builtin_amdgcn_s_memrealtime();
    for (unsigned i = 1; __any((int)(__hip_atomic_load(fp, __ATOMIC_RELAXED, __HIP_MEMORY_SCOPE_SYSTEM) - need) < 0); ++i) {
      if ((i & 15) == 0 && (__hip_atomic_load(tmo, __ATOMIC_RELAXED, __HIP_MEMORY_SCOPE_AGENT) == call || __builtin_amdgcn_s_memrealtime() - t0 > DEADLINE)) {
        __hip_atomic_store(tmo, call, __ATOMIC_RELAXED, __HIP_MEMORY_SCOPE_AGENT);
        bad = true;
        break;
      }
      __builtin_amdgcn_s_sleep(1);
    }
  }
  if (__builtin_expect(bad, 0)) {
    waitv<0>();
    for (int g = 0; g < 2 * MF2; ++g)
      __builtin_amdgcn_raw_buffer_store_b128((u32x4){0x7fc07fc0u, 0x7fc07fc0u, 0x7fc07fc0u, 0x7fc07fc0u}, rs2, o2 + (g >> 1) * 16 * OPITCH + (g & 1) * 64, 0, AUX_SYS);
  }
  STAMP(6);
}
}

__global__ __launch_bounds__(512, 2) void gemm_a2a(Params p) {
  extern __shared__ __attribute__((aligned(16))) char smem[];
  LAS char* lds = (LAS char*)smem;
  const int lane = threadIdx.x & 63, wid = __builtin_amdgcn_readfirstlane(threadIdx.x >> 6);
  Tile t;
  t.c = blockIdx.x; t.r = p.rank;
  const int xcd = t.c & 7, j = t.c >> 3, e = j >> 2, hm = j & 1, pb = (j >> 1) & 1, h2 = xcd & 1;
  t.own = j >= 28;
  t.peer = (t.r + 1 + e) & 7;
  t.pl1 = xcd + 8 * pb;
  t.pl2 = 4 * (xcd >> 1) + (j & 3);
  t.row1 = 32 * MF1 * hm;
  if (((wid >> 1) & 1) == 0) { t.src2 = t.peer; t.row2 = t.own ? 128 * h2 : RROWS + 128 * h2; }
  else { t.src2 = h2 ? t.r : t.peer; t.row2 = t.own ? 128 * h2 + 64 : h2 ? (e < 4 ? 256 + 64 * e : -1) : RROWS + 64; }
  t.rot1 = 2 * hm + (j >> 1); t.rot2 = e + j;
  STAMP(0);
  if (t.own) xpush(p, t, wid, lane);
  if (wid >= 4) producer(p, t, lds, wid - 4, lane); else consumer(p, t, lds, wid, lane);
}

extern "C" void kernel_launch(int rank, int nranks, ncclComm_t comm,
                              hipStream_t stream,
                              void* const* d_in, const int* in_sizes, int n_in,
                              void* d_out, int out_size,
                              const int* in_dtypes, int out_dtype,
                              void* const* d_out_all,
                              void* const* d_ws_all, size_t ws_bytes) {
  if (nranks != NR || (unsigned)rank >= (unsigned)NR || n_in != 2 || in_sizes[0] != MLOC * KD || in_sizes[1] != KD * NG ||
      out_size != NR * MLOC * NLOC || ws_bytes < WS_NEED) { fprintf(stderr, "gemm_a2a: unexpected problem shape\n"); return; }
  static bool attr_set[NR] = {};
  if (!attr_set[rank]) { HIP_CHECK(hipFuncSetAttribute((const void*)gemm_a2a, hipFuncAttributeMaxDynamicSharedMemorySize, LDS_BYTES)); attr_set[rank] = true; }
  Params p;
  p.x = (const char*)d_in[0]; p.w = (const char*)d_in[1]; p.rank = rank;
  for (int i = 0; i < NR; ++i) { p.out[i] = (char*)d_out_all[i]; p.ws[i] = (char*)d_ws_all[i]; }
  hipLaunchKernelGGL(gemm_a2a, dim3(256), dim3(512), LDS_BYTES, stream, p);
  HIP_CHECK(hipGetLastError());
}
```

```cpp
#define DIAG 1
#define DIAGAVG 0
#include <hip/hip_runtime.h>
#include <rccl/rccl.h>
#include <cstdio>
#include <type_traits>

#define HIP_CHECK(expr) do { hipError_t _e = (expr); if (_e != hipSuccess) { \
    fprintf(stderr, "HIP error %s at %s:%d\n", hipGetErrorString(_e), __FILE__, __LINE__); \
    return; } } while (0)

namespace {
#define LAS __attribute__((address_space(3)))
typedef short s16x4 __attribute__((ext_vector_type(4)));
typedef short s16x8 __attribute__((ext_vector_type(8)));
typedef __bf16 bf16x8 __attribute__((ext_vector_type(8)));
typedef float f32x4 __attribute__((ext_vector_type(4)));
typedef unsigned u32x4 __attribute__((ext_vector_type(4)));
typedef unsigned long long u64;

constexpr int NR = 8, MLOC = 512, KD = 4096, NG = 16384, NLOC = 2048;
constexpr int MF1 = 5, MF2 = 4;
constexpr int RROWS = 64 * MF1, FROWS = MLOC - RROWS;
constexpr int AMAX = 32 * MF1 * 128, SLOT = AMAX + 16384, RING = 4 * SLOT, LDS_BYTES = RING;
static_assert(FROWS == 192 && LDS_BYTES <= 163840, "tiles");
constexpr unsigned XPITCH = KD * 2, WPITCH = NG * 2, OPITCH = NLOC * 2;
constexpr unsigned OUT_BYTES = (unsigned)NR * MLOC * OPITCH;
constexpr unsigned XBLK = FROWS * XPITCH;
constexpr unsigned WS_CTR = 0;
constexpr unsigned WS_TMO = 32768;
constexpr unsigned WS_XFLAG = 36864;
constexpr unsigned WS_DONE = 49152;
constexpr unsigned WS_X = 1u << 20;
constexpr unsigned WS_NEED = WS_X + NR * XBLK;
constexpr u64 DEADLINE = 40000000ull;
#define AUXO 17
constexpr int AUX_SYS = AUXO;

#define GL 6
constexpr int NIN = 63 / GL < 2 * MF1 ? 63 / GL : 2 * MF1;

struct Params { const char* x; const char* w; char* out[NR]; char* ws[NR]; int rank; };
struct Tile { int c, r, peer, own, pl1, pl2, row1, src2, row2, rot1, rot2; unsigned t0; };

__device__ __forceinline__ unsigned pk_bf16(float lo, float hi) {
  typedef __bf16 bf16x2 __attribute__((ext_vector_type(2)));
  bf16x2 v = {(__bf16)lo, (__bf16)hi};
  return __builtin_bit_cast(unsigned, v);
}
constexpr float GELU_A = -2.0f * 0.7978845608028654f * 1.4426950408889634f, GELU_B = GELU_A * 0.044715f;
__device__ __forceinline__ float gelu(float y) {
  const float e = __builtin_amdgcn_exp2f(y * __builtin_fmaf(GELU_B, y * y, GELU_A));
  return y * __builtin_amdgcn_rcpf(1.0f + e);
}
__device__ __forceinline__ u32x4 epi8(const f32x4 u, const f32x4 v, bool odd) {
  const unsigned a0 = pk_bf16(gelu(u[0]), gelu(u[1])), a1 = pk_bf16(gelu(u[2]), gelu(u[3]));
  const unsigned b0 = pk_bf16(gelu(v[0]), gelu(v[1])), b1 = pk_bf16(gelu(v[2]), gelu(v[3]));
  u32x4 w; w.x = odd ? b0 : a0; w.y = odd ? b1 : a1; w.z = odd ? a0 : b0; w.w = odd ? a1 : b1;
  return w;
}
template <int N_> __device__ __forceinline__ void waitv() { asm volatile("s_waitcnt vmcnt(%0)" :: "i"(N_) : "memory"); }
#define WAIT_L0() __builtin_amdgcn_s_waitcnt(0xC07F)
#define BAR() __builtin_amdgcn_s_barrier()
#define SCHED() __builtin_amdgcn_sched_barrier(0)
template <int I, int N, class F> __device__ __forceinline__ void static_for(F&& f) { if constexpr (I < N) { f(std::integral_constant<int, I>{}); static_for<I + 1, N>(f); } }
__device__ __forceinline__ LAS char* lp(unsigned a) { return (LAS char*)(size_t)a; }
__device__ __forceinline__ auto rsrc(const void* base, unsigned bytes) { return __builtin_amdgcn_make_buffer_rsrc((void*)base, 0, (int)bytes, 0x00020000); }

struct Frag { s16x8 a[MF1]; s16x4 blo[4], bhi[4]; };
template <int K, int KS> __device__ __forceinline__ void rd1(Frag& f, unsigned aaddr, const unsigned (&baddr)[4]) {
  if constexpr (K == 2) f.a[0] = *(LAS s16x8*)lp(aaddr);
  else if constexpr (K >= 9) f.a[K - 8] = *(LAS s16x8*)(lp(aaddr) + (K - 8) * 2048);
  else {
    constexpr int q = K < 2 ? K : K - 1, gn = q >> 1;
    if constexpr (q & 1) f.bhi[gn] = __builtin_amdgcn_ds_read_tr16_b64_v4i16((LAS s16x4*)(lp(baddr[gn]) + KS * 8192 + 1024));
    else f.blo[gn] = __builtin_amdgcn_ds_read_tr16_b64_v4i16((LAS s16x4*)(lp(baddr[gn]) + KS * 8192));
  }
}
template <int Z, int MF> __device__ __forceinline__ void mm1(f32x4 (&acc)[MF][4], const Frag& f) {
  constexpr int m = Z >> 2, gn = Z & 3;
  const s16x8 b = __builtin_shufflevector(f.blo[gn], f.bhi[gn], 0, 1, 2, 3, 4, 5, 6, 7);
  asm volatile("v_mfma_f32_16x16x32_bf16 %0, %1, %2, %0" : "+v"(acc[m][gn]) : "v"(b), "v"(f.a[m]));
}
struct GeluState { f32x4 cu, cv; float ga, gb, ka; unsigned pk0, pk1, pk2, pk3, ov0, ov1, ov2, ov3; };
template <int F> __device__ __forceinline__ void gelu_piece(GeluState& g) {
  constexpr int pr = F / 10, st = F % 10;
  constexpr u64 ODDM = 0xffff0000ffff0000ull;
  if constexpr (F < 40) {
    const float ya = pr < 2 ? g.cu[2 * (pr & 1)] : g.cv[2 * (pr & 1)], yb = pr < 2 ? g.cu[2 * (pr & 1) + 1] : g.cv[2 * (pr & 1) + 1];
    if constexpr (st == 0) asm volatile("v_mul_f32 %0, %2, %2\n\tv_mul_f32 %1, %3, %3" : "=&v"(g.ga), "=v"(g.gb) : "v"(ya), "v"(yb));
    if constexpr (st == 1) asm volatile("v_fmamk_f32 %0, %0, %3, %2\n\tv_fmamk_f32 %1, %1, %3, %2" : "+v"(g.ga), "+v"(g.gb) : "v"(g.ka), "i"(__builtin_bit_cast(unsigned, GELU_B)));
    if constexpr (st == 2 || st == 8) asm volatile("v_mul_f32 %0, %2, %0\n\tv_mul_f32 %1, %3, %1" : "+v"(g.ga), "+v"(g.gb) : "v"(ya), "v"(yb));
    if constexpr (st == 3) asm volatile("v_exp_f32 %0, %0" : "+v"(g.ga));
    if constexpr (st == 4) asm volatile("v_exp_f32 %0, %0" : "+v"(g.gb));
    if constexpr (st == 5) asm volatile("v_add_f32 %0, 1.0, %0\n\tv_add_f32 %1, 1.0, %1" : "+v"(g.ga), "+v"(g.gb));
    if constexpr (st == 6) asm volatile("v_rcp_f32 %0, %0" : "+v"(g.ga));
    if constexpr (st == 7) asm volatile("v_rcp_f32 %0, %0" : "+v"(g.gb));
    if constexpr (st == 9) { unsigned& d = pr == 0 ? g.pk0 : pr == 1 ? g.pk1 : pr == 2 ? g.pk2 : g.pk3; asm volatile("v_cvt_pk_bf16_f32 %0, %1, %2" : "=v"(d) : "v"(g.ga), "v"(g.gb)); }
  } else if constexpr (F == 40) asm volatile("v_cndmask_b32_e64 %0, %2, %4, %6\n\tv_cndmask_b32_e64 %1, %3, %5, %6" : "=&v"(g.ov0), "=v"(g.ov1) : "v"(g.pk0), "v"(g.pk1), "v"(g.pk2), "v"(g.pk3), "s"(ODDM));
  else if constexpr (F == 41) asm volatile("v_cndmask_b32_e64 %0, %4, %2, %6\n\tv_cndmask_b32_e64 %1, %5, %3, %6" : "=&v"(g.ov2), "=v"(g.ov3) : "v"(g.pk0), "v"(g.pk1), "v"(g.pk2), "v"(g.pk3), "s"(ODDM));
}
template <int G> __device__ __forceinline__ void pick(GeluState& s, const f32x4 (&acc)[MF1][4]) {
  constexpr int m = G >> 1, h = G & 1;
#pragma unroll
  for (int i = 0; i < 4; ++i) {
    const float u0 = acc[m][2 * h][i], u1 = acc[m][2 - 2 * h][i], v0 = acc[m][2 * h + 1][i], v1 = acc[m][3 - 2 * h][i];
    s.cu[i] = __int_as_float(__builtin_amdgcn_update_dpp(__float_as_int(u0), __float_as_int(u1), 0x128, 0xf, h ? 0x3 : 0xc, false));
    s.cv[i] = __int_as_float(__builtin_amdgcn_update_dpp(__float_as_int(v0), __float_as_int(v1), 0x128, 0xf, h ? 0x3 : 0xc, false));
  }
}
constexpr int dbl_gaps(int mf) { return mf == 2 ? 5 : mf == 3 ? 4 : mf == 4 ? 1 : 0; }

#define STAMP(i)
#define NEED(n) (n)

#define XFLY 3
__device__ __forceinline__ void xpush(const Params& p, const Tile& t, int wid, int lane) {
  const u64 wsl = (u64)p.ws[(t.r + 1 + (lane < 7 ? lane : 0)) & 7];
  const int w = ((t.c >> 3) - 28) * 64 + (t.c & 7) * 8 + wid;
  u32x4 v[2][3]; unsigned po[3];
#pragma unroll
  for (int i = 0; i < 3; ++i) { const int q = 3 * w + i; po[i] = (q >> 2) * XPITCH + (q & 3) * 1024 + lane * 16;
    v[0][i] = *(const u32x4*)(p.x + (size_t)RROWS * XPITCH + po[i]); v[1][i] = *(const u32x4*)(p.x + (size_t)RROWS * XPITCH + po[i] + 4096); }
#pragma unroll
  for (int h = 0; h < 2; ++h) {
    int e = w % 7;
#pragma unroll 1
    for (int n = 0; n < 7; ++n) {
      char* base = (char*)(((u64)(unsigned)__builtin_amdgcn_readlane((unsigned)(wsl >> 32), e) << 32) | (unsigned)__builtin_amdgcn_readlane((unsigned)wsl, e));
      const auto rs = rsrc(base + WS_X + t.r * XBLK, XBLK);
#pragma unroll
      for (int i = 0; i < 3; ++i) { __builtin_amdgcn_raw_buffer_store_b128(v[h][i], rs, po[i], h * 4096, 17); waitv<XFLY>(); }
      e = e == 6 ? 0 : e + 1;
    }
    waitv<0>(); BAR();
    if (wid == 0 && lane < 7) __hip_atomic_fetch_add((unsigned*)((char*)wsl + WS_XFLAG + (t.r * 2 + h) * 128), 1u, __ATOMIC_RELAXED, __HIP_MEMORY_SCOPE_SYSTEM);
  }
}

__device__ __forceinline__ void producer(const Params& p, const Tile& t, LAS char* lds, int pw, int lane) {
  char* const myws = p.ws[t.r];
  const unsigned va0 = (lane >> 3) * XPITCH + (((lane & 7) ^ (lane >> 4)) << 4), va1 = va0 ^ 64;
  const unsigned vb0 = (lane >> 4) * WPITCH + (((lane & 15) ^ (((lane >> 4) & 1) | ((lane >> 5) << 2))) << 4), vb1 = vb0 ^ 128;
  const unsigned ve0 = va0 ^ ((pw & 1) << 6), ve1 = ve0 ^ 64;
  const auto ra1 = rsrc(p.x + (size_t)(t.row1 + 40 * pw) * XPITCH, 40 * XPITCH);
  const auto rb1 = rsrc(p.w + (size_t)(t.peer * 16 + t.pl1) * 256, (KD - 1) * WPITCH + 256);
  const int s2 = t.src2, r2 = t.row2 + 32 * (pw & 1);
  const auto ra2 = rsrc(s2 == t.r ? p.x + (size_t)r2 * XPITCH : myws + WS_X + s2 * XBLK + (size_t)(r2 - RROWS) * XPITCH, t.row2 < 0 ? 0 : 32 * XPITCH);
  const auto rb2 = rsrc(p.w + (size_t)(t.r * 16 + t.pl2) * 256, (KD - 1) * WPITCH + 256);
  unsigned so = 0;
  auto issue1 = [&](int s) {
    const int kt = (s & 60) | ((s + t.rot1) & 3);
#pragma unroll
    for (int i = 0; i < MF1; ++i) __builtin_amdgcn_raw_ptr_buffer_load_lds(ra1, lds + so + (pw * MF1 + i) * 1024, 16, (i & 1) ? ve1 : ve0, kt * 128 + i * 8 * XPITCH, 0, 0);
#pragma unroll
    for (int i = 0; i < 4; ++i) __builtin_amdgcn_raw_ptr_buffer_load_lds(rb1, lds + so + AMAX + (pw * 4 + i) * 1024, 16, (i & 2) ? vb1 : vb0, (kt * 64 + pw * 16 + i * 4) * WPITCH, 0, 0);
    so = so + SLOT == RING ? 0 : so + SLOT;
  };
  auto issue2 = [&](int s) {
    const int kt = (s & 60) | ((s + t.rot2) & 3);
#pragma unroll
    for (int i = 0; i < MF2; ++i) __builtin_amdgcn_raw_ptr_buffer_load_lds(ra2, lds + so + (pw * MF2 + i) * 1024, 16, (i & 1) ? va1 : va0, kt * 128 + i * 8 * XPITCH, 0, 0);
#pragma unroll
    for (int i = 0; i < 4; ++i) __builtin_amdgcn_raw_ptr_buffer_load_lds(rb2, lds + so + AMAX + (pw * 4 + i) * 1024, 16, (i & 2) ? vb1 : vb0, (kt * 64 + pw * 16 + i * 4) * WPITCH, 0, 0);
    so = so + SLOT == RING ? 0 : so + SLOT;
  };
  constexpr int L1 = MF1 + 4, L2 = MF2 + 4;
  unsigned callv = 0, fv = 0;
  const bool poller = (pw == 0) && !t.own;
  auto poll = [&](int q) {
    if (poller) { const char* fp = myws + WS_XFLAG + (t.peer * 2 + q) * 128; asm volatile("global_load_dword %0, %1, off sc0 sc1" : "+v"(fv) : "v"(fp) : "memory"); }
  };
  auto gate = [&](int q) {
    if (poller) {
      asm volatile("" : "+v"(fv), "+v"(callv));
      const unsigned call = __builtin_amdgcn_readfirstlane(callv) + 1, need = NEED(32u * call);
      if (__any((int)(fv - need) < 0)) {
        unsigned* const tmo = (unsigned*)(myws + WS_TMO);
        const u64 t0 = __builtin_amdgcn_s_memrealtime();
        while (__any((int)(__hip_atomic_load((unsigned*)(myws + WS_XFLAG + (t.peer * 2 + q) * 128), __ATOMIC_RELAXED, __HIP_MEMORY_SCOPE_SYSTEM) - need) < 0)) {
          if (__hip_atomic_load(tmo, __ATOMIC_RELAXED, __HIP_MEMORY_SCOPE_AGENT) == call || __builtin_amdgcn_s_memrealtime() - t0 > DEADLINE) {
            __hip_atomic_store(tmo, call, __ATOMIC_RELAXED, __HIP_MEMORY_SCOPE_AGENT);
            waitv<0>();
            break;
          }
          __builtin_amdgcn_s_sleep(2);
        }
      }
      asm volatile("buffer_inv sc0 sc1" ::: "memory");
    }
  };
  if (!t.own) {
    issue1(0); issue1(1); issue1(2); issue1(3);
    { const char* ctr = myws + WS_CTR + t.c * 128; asm volatile("global_load_dword %0, %1, off sc1" : "+v"(callv) : "v"(ctr) : "memory"); }
    waitv<3 * L1 + 1>(); BAR();
    waitv<2 * L1 + 1>(); BAR(); issue1(4);
#pragma unroll 1
    for (int i = 1; i < 60; ++i) {
      waitv<2 * L1>();
      if (i == 58) gate(0);
      BAR();
      if (i == 56) poll(0);
      issue1(i + 4);
    }
    waitv<2 * L1>(); BAR(); issue2(0);
    waitv<L1 + L2>(); BAR(); issue2(1);
    waitv<2 * L2>(); BAR(); issue2(2);
    waitv<2 * L2>(); BAR(); issue2(3);
  } else {
    waitv<0>();
    issue2(0); issue2(1); issue2(2); issue2(3);
    waitv<3 * L2>(); BAR();
  }
#pragma unroll 1
  for (int i = 0; i < 60; ++i) {
    waitv<2 * L2>();
    if (i == 26) gate(1);
    BAR();
    if (i == 24) poll(1);
    issue2(i + 4);
  }
  waitv<2 * L2>(); BAR(); waitv<L2>(); BAR(); waitv<0>(); BAR(); BAR();
  asm volatile("" :: "v"(callv), "v"(fv));
}

__device__ __forceinline__ void consumer(const Params& p, const Tile& t, LAS char* lds, int wid, int lane) {
  char* const myws = p.ws[t.r];
  __builtin_amdgcn_s_setprio(2);
  const int wr = wid >> 1, wc = wid & 1, fr = lane & 15, fq = lane >> 4;
  const bool odd = fq & 1;
  const unsigned callv = __hip_atomic_load((unsigned*)(myws + WS_CTR + t.c * 128), __ATOMIC_RELAXED, __HIP_MEMORY_SCOPE_AGENT);

  const unsigned lb = (unsigned)(size_t)lds;
  unsigned a1[2], a2[2], baddr[4];
#pragma unroll
  for (int ks = 0; ks < 2; ++ks) { const unsigned u = lb + fr * 128 + 16 * ((4 * ks + fq) ^ ((fr >> 1) & 7)); a1[ks] = u + 16 * MF1 * wr * 128; a2[ks] = u + 16 * MF2 * wr * 128; }
  { const int q = (lane & 15) >> 2, pp = lane & 3, kr = 8 * fq + q, f = (kr & 1) | ((kr & 2) << 1) | (kr & 8);
#pragma unroll
    for (int gn = 0; gn < 4; ++gn) baddr[gn] = lb + AMAX + kr * 256 + 16 * ((8 * wc + 4 * (gn >> 1) + pp) ^ f) + 8 * ((gn & 1) ^ (pp & 1)); }
  const auto rs1 = rsrc(p.out[t.peer], t.own ? 0 : OUT_BYTES);
  const unsigned o1 = (t.r * MLOC + t.row1 + 16 * MF1 * wr + (fr & 7)) * OPITCH + (t.pl1 * 128 + 64 * wc + 32 * (fr >> 3) + 8 * fq) * 2;
#define OSTEP(g) (8 * OPITCH)
#define OADDR(g) ((g) * 8 * OPITCH)
  const auto rs2 = rsrc(p.out[t.r], t.row2 < 0 ? 0 : OUT_BYTES);
  const unsigned o2 = (t.src2 * MLOC + t.row2 + fr) * OPITCH + (t.pl2 * 128 + 64 * wc + 8 * fq) * 2;

  Frag F0, F1;
  f32x4 acc1[MF1][4], acc2[MF2][4];
#pragma unroll
  for (int m = 0; m < MF1; ++m)
#pragma unroll
    for (int gn = 0; gn < 4; ++gn) { acc1[m][gn] = (f32x4){0.f, 0.f, 0.f, 0.f}; asm volatile("" : "+v"(acc1[m][gn])); }
  unsigned so = 0, bc[4], ac[2];
#pragma unroll
  for (int gn = 0; gn < 4; ++gn) bc[gn] = baddr[gn];
  ac[0] = t.own ? a2[0] : a1[0]; ac[1] = t.own ? a2[1] : a1[1];
#pragma unroll
  for (int m = 0; m < MF2; ++m)
#pragma unroll
    for (int gn = 0; gn < 4; ++gn) { acc2[m][gn] = (f32x4){0.f, 0.f, 0.f, 0.f}; asm volatile("" : "+v"(acc2[m][gn])); }
  asm volatile("s_nop 1");
  BAR();
  STAMP(1);
  static_for<0, 8 + MF1>([&](auto K) { rd1<K, 0>(F0, ac[0], bc); });
  GeluState gs; gs.ka = GELU_A; asm volatile("" : "+v"(gs.ka));
  auto half_a = [&](auto MF_, auto& acc, const unsigned (&an)[2], auto EV, auto G0_) {
    constexpr int MF = MF_, NM = 4 * MF, NRD = 8 + MF, DBL = dbl_gaps(MF), LAST = NRD - DBL - 1, G0 = G0_;
    static_for<0, NM>([&](auto Z) {
      constexpr int z = Z;
      SCHED(); mm1<z, MF>(acc, F0); SCHED();
      if constexpr (z < DBL) { rd1<2 * z, 1>(F1, ac[1], bc); rd1<2 * z + 1, 1>(F1, ac[1], bc); }
      else if constexpr (z <= LAST) rd1<z + DBL, 1>(F1, ac[1], bc);
      if constexpr (z == LAST - 1) { so = so + SLOT == RING ? 0 : so + SLOT; asm volatile("" : "+s"(so)); }
      if constexpr (z == LAST + 1) { bc[0] = baddr[0] + so; bc[1] = baddr[1] + so; asm volatile("" : "+v"(bc[0]), "+v"(bc[1])); }
      if constexpr (z == LAST + 2) { bc[2] = baddr[2] + so; bc[3] = baddr[3] + so; asm volatile("" : "+v"(bc[2]), "+v"(bc[3])); }
      if constexpr (z == LAST + 3) { ac[0] = an[0] + so; ac[1] = an[1] + so; asm volatile("" : "+v"(ac[0]), "+v"(ac[1])); }
      if constexpr (G0 >= 0 && z > LAST + 3) gelu_piece<G0 + z - LAST - 4>(gs);
    });
    static_assert(LAST + 3 < NM, "half step too short");
    SCHED();
  };
  auto half_b = [&](auto MF_, auto MFN_, auto& acc, auto ZERO, auto G0_) {
    constexpr int MF = MF_, MFN = MFN_, NM = 4 * MF, NRD = 8 + MFN, DBL = dbl_gaps(MF), G0 = G0_;
    SCHED(); WAIT_L0(); BAR();
    static_for<0, NM>([&](auto Z) {
      constexpr int z = Z;
      SCHED(); mm1<z, MF>(acc, F1); SCHED();
      if constexpr (z < DBL) { rd1<2 * z, 0>(F0, ac[0], bc); rd1<2 * z + 1, 0>(F0, ac[0], bc); }
      else if constexpr (z + DBL < NRD) rd1<z + DBL, 0>(F0, ac[0], bc);
      else if constexpr (G0 >= 0) gelu_piece<G0 + z + DBL - NRD>(gs);
    });
    SCHED();
  };
  constexpr std::integral_constant<int, MF1> c1{}; constexpr std::integral_constant<int, MF2> c2{};
  constexpr std::integral_constant<int, -1> nog{}; constexpr std::true_type yes{}; constexpr std::false_type no{};
  int it = 63;
  if (!t.own) {
#pragma unroll 1
    do {
      half_a(c1, acc1, a1, no, nog);
      half_b(c1, c1, acc1, no, nog);
      --it; asm volatile("" : "+s"(it));
    } while (it != 0);
    half_a(c1, acc1, a2, no, nog);
    half_b(c1, c2, acc1, no, nog);
    asm volatile("s_nop 15" ::: "memory"); SCHED();
  }
  so = __builtin_amdgcn_readfirstlane(so);
  const unsigned dg2 = (unsigned)__builtin_amdgcn_s_memrealtime(); __builtin_amdgcn_s_waitcnt(0xC07F);
  STAMP(2);
  constexpr int FA = 4 * MF2 - (8 + MF2 - dbl_gaps(MF2)) - 3, FB = 4 * MF2 - (8 + MF2 - dbl_gaps(MF2));
  static_assert(GL * (FA + FB) >= 42 && NIN * GL < 64, "GELU pieces do not fit");
  constexpr std::integral_constant<int, GL * FB + (GL - 1) * FA> gtail{};
  unsigned oo = o1;
  it = 0;
#pragma unroll 1
  do {
    half_a(c2, acc2, a2, no, gtail);
    if (it) { __builtin_amdgcn_raw_buffer_store_b128((u32x4){gs.ov0, gs.ov1, gs.ov2, gs.ov3}, rs1, oo, 0, AUX_SYS); oo += OSTEP(it); }
#define PICK(g) case g: pick<(g < NIN ? g : 0)>(gs, acc1); break
    switch (it) { PICK(0); PICK(1); PICK(2); PICK(3); PICK(4); PICK(5); PICK(6); PICK(7); PICK(8); PICK(9); default: __builtin_unreachable(); }
    asm volatile("" : "+v"(gs.cu), "+v"(gs.cv));
    half_b(c2, c2, acc2, no, std::integral_constant<int, 0>{});
    static_for<1, GL>([&](auto K) { constexpr int k = K;
      half_a(c2, acc2, a2, no, std::integral_constant<int, FB + (k - 1) * (FA + FB)>{});
      half_b(c2, c2, acc2, no, std::integral_constant<int, k * (FA + FB)>{}); });
    ++it; asm volatile("" : "+s"(it));
  } while (it != NIN);
  half_a(c2, acc2, a2, no, gtail);
  __builtin_amdgcn_raw_buffer_store_b128((u32x4){gs.ov0, gs.ov1, gs.ov2, gs.ov3}, rs1, oo, 0, AUX_SYS);
  half_b(c2, c2, acc2, no, nog);
  if constexpr (NIN * GL + 1 < 64) {
    it = 63 - NIN * GL;
#pragma unroll 1
    do { half_a(c2, acc2, a2, no, nog); half_b(c2, c2, acc2, no, nog); --it; asm volatile("" : "+s"(it)); } while (it != 0);
  }
  asm volatile("s_nop 15" ::: "memory"); SCHED();
  const unsigned dg3 = (unsigned)__builtin_amdgcn_s_memrealtime(); __builtin_amdgcn_s_waitcnt(0xC07F);
  STAMP(3);
  static_for<NIN, 2 * MF1>([&](auto G) { constexpr int g = G;
    pick<g>(gs, acc1); __builtin_amdgcn_raw_buffer_store_b128(epi8(gs.cu, gs.cv, odd), rs1, o1 + OADDR(g), 0, AUX_SYS); });
  const unsigned tmov = __hip_atomic_load((unsigned*)(myws + WS_TMO), __ATOMIC_RELAXED, __HIP_MEMORY_SCOPE_AGENT);
  const unsigned call = __builtin_amdgcn_readfirstlane(callv) + 1;
  SCHED();
#define DONEAT 4
  static_for<0, 2 * MF2>([&](auto G) { constexpr int g = G;
    if constexpr (g == DONEAT) {
      SCHED(); waitv<DONEAT>();
      if (!t.own && lane == 0) __hip_atomic_fetch_add((unsigned*)(p.ws[t.peer] + WS_DONE + (t.r * 4 + wid) * 128), 1u, __ATOMIC_RELAXED, __HIP_MEMORY_SCOPE_SYSTEM);
      SCHED(); STAMP(4);
    }
    __builtin_amdgcn_raw_buffer_store_b128(epi8(acc2[g >> 1][2 * (g & 1)], acc2[g >> 1][2 * (g & 1) + 1], odd), rs2, o2 + (g >> 1) * 16 * OPITCH + (g & 1) * 64, 0, AUX_SYS); });
  STAMP(5);
  if (wid == 0 && lane == 0) __hip_atomic_store((unsigned*)(myws + WS_CTR + t.c * 128), call, __ATOMIC_RELAXED, __HIP_MEMORY_SCOPE_AGENT);
  unsigned* const dgw = (unsigned*)(myws + (40u << 20)) + (call & 4095) * 512;
  waitv<0>();
  { const unsigned dg5 = (unsigned)__builtin_amdgcn_s_memrealtime();
    const unsigned val = DIAG == 1 ? dg2 : DIAG == 3 ? dg3 : DIAG == 6 ? t.t0 + (dg3 - dg2) : DIAG == 7 ? (t.own ? dg2 : t.t0) : dg5;
    if (lane == 0 && wid == 0) { __hip_atomic_store(dgw + t.c * 2, t.t0, __ATOMIC_RELAXED, __HIP_MEMORY_SCOPE_AGENT); waitv<0>(); __hip_atomic_store(dgw + t.c * 2 + 1, val | 1u, __ATOMIC_RELAXED, __HIP_MEMORY_SCOPE_AGENT); } }
  bool bad = tmov == call;
  if (t.c == 255 && wid == 0) {
    unsigned* const tmo = (unsigned*)(myws + WS_TMO);
    unsigned* const fp = (unsigned*)(myws + WS_DONE + (((t.r + 1 + (lane < 28 ? lane >> 2 : 0)) & 7) * 4 + (lane & 3)) * 128);
    const unsigned need = NEED(32u * call);
    const u64 t0 = __builtin_amdgcn_s_memrealtime();
    for (unsigned i = 1; __any((int)(__hip_atomic_load(fp, __ATOMIC_RELAXED, __HIP_MEMORY_SCOPE_SYSTEM) - need) < 0); ++i) {
      if ((i & 15) == 0 && (__hip_atomic_load(tmo, __ATOMIC_RELAXED, __HIP_MEMORY_SCOPE_AGENT) == call || __builtin_amdgcn_s_memrealtime() - t0 > DEADLINE)) {
        __hip_atomic_store(tmo, call, __ATOMIC_RELAXED, __HIP_MEMORY_SCOPE_AGENT);
        bad = true;
        break;
      }
      __builtin_amdgcn_s_sleep(1);
    }
  }
  if (t.c == 255 && wid == 0) {
    const unsigned dg6 = (unsigned)__builtin_amdgcn_s_memrealtime();
    while ((unsigned)__builtin_amdgcn_s_memrealtime() - t.t0 < 14000u) __builtin_amdgcn_s_sleep(8);
    int so_ = 0, mo_ = -100000, sum_ = 0;
    for (int q = 0; q < 4; ++q) { const int a_ = (int)(__hip_atomic_load(dgw + (q * 64 + lane) * 2, __ATOMIC_RELAXED, __HIP_MEMORY_SCOPE_AGENT) - t.t0), b_ = (int)(__hip_atomic_load(dgw + (q * 64 + lane) * 2 + 1, __ATOMIC_RELAXED, __HIP_MEMORY_SCOPE_AGENT) - t.t0);
      so_ = min(so_, a_); mo_ = max(mo_, b_); sum_ += b_; }
    for (int o = 32; o >= 1; o >>= 1) { so_ = min(so_, __shfl_xor(so_, o)); mo_ = max(mo_, __shfl_xor(mo_, o)); sum_ += __shfl_xor(sum_, o); }
    const unsigned st = t.t0 + so_, mv = t.t0 + (DIAGAVG ? sum_ / 256 : mo_);
    const unsigned m = DIAG == 0 ? 0 : DIAG == 9 ? dg6 - st : mv - st;
    while ((unsigned)__builtin_amdgcn_s_memrealtime() - st < 15000u + (m < 20000u ? m : 20000u)) __builtin_amdgcn_s_sleep(1);
  }
  if (__builtin_expect(bad, 0)) {
    waitv<0>();
    for (int g = 0; g < 2 * MF2; ++g)
      __builtin_amdgcn_raw_buffer_store_b128((u32x4){0x7fc07fc0u, 0x7fc07fc0u, 0x7fc07fc0u, 0x7fc07fc0u}, rs2, o2 + (g >> 1) * 16 * OPITCH + (g & 1) * 64, 0, AUX_SYS);
  }
  STAMP(6);
}
}

__global__ __launch_bounds__(512, 2) void gemm_a2a(Params p) {
  extern __shared__ __attribute__((aligned(16))) char smem[];
  LAS char* lds = (LAS char*)smem;
  const int lane = threadIdx.x & 63, wid = __builtin_amdgcn_readfirstlane(threadIdx.x >> 6);
  Tile t;
  t.t0 = (unsigned)__builtin_amdgcn_s_memrealtime();
  t.c = blockIdx.x; t.r = p.rank;
  const int xcd = t.c & 7, j = t.c >> 3, e = j >> 2, hm = j & 1, pb = (j >> 1) & 1, h2 = xcd & 1;
  t.own = j >= 28;
  t.peer = (t.r + 1 + e) & 7;
  t.pl1 = xcd + 8 * pb;
  t.pl2 = 4 * (xcd >> 1) + (j & 3);
  t.row1 = 32 * MF1 * hm;
  if (((wid >> 1) & 1) == 0) { t.src2 = t.peer; t.row2 = t.own ? 128 * h2 : RROWS + 128 * h2; }
  else { t.src2 = h2 ? t.r : t.peer; t.row2 = t.own ? 128 * h2 + 64 : h2 ? (e < 4 ? 256 + 64 * e : -1) : RROWS + 64; }
  t.rot1 = 2 * hm + (j >> 1); t.rot2 = e + j;
  STAMP(0);
  if (t.own) xpush(p, t, wid, lane);
  if (wid >= 4) producer(p, t, lds, wid - 4, lane); else consumer(p, t, lds, wid, lane);
}

extern "C" void kernel_launch(int rank, int nranks, ncclComm_t comm,
                              hipStream_t stream,
                              void* const* d_in, const int* in_sizes, int n_in,
                              void* d_out, int out_size,
                              const int* in_dtypes, int out_dtype,
                              void* const* d_out_all,
                              void* const* d_ws_all, size_t ws_bytes) {
  if (nranks != NR || (unsigned)rank >= (unsigned)NR || n_in != 2 || in_sizes[0] != MLOC * KD || in_sizes[1] != KD * NG ||
      out_size != NR * MLOC * NLOC || ws_bytes < WS_NEED) { fprintf(stderr, "gemm_a2a: unexpected problem shape\n"); return; }
  static bool attr_set[NR] = {};
  if (!attr_set[rank]) { HIP_CHECK(hipFuncSetAttribute((const void*)gemm_a2a, hipFuncAttributeMaxDynamicSharedMemorySize, LDS_BYTES)); attr_set[rank] = true; }
  Params p;
  p.x = (const char*)d_in[0]; p.w = (const char*)d_in[1]; p.rank = rank;
  for (int i = 0; i < NR; ++i) { p.out[i] = (char*)d_out_all[i]; p.ws[i] = (char*)d_ws_all[i]; }
  hipLaunchKernelGGL(gemm_a2a, dim3(256), dim3(512), LDS_BYTES, stream, p);
  HIP_CHECK(hipGetLastError());
}
```

```cpp
#include <hip/hip_runtime.h>
#include <rccl/rccl.h>
#include <cstdio>
#include <type_traits>

#define HIP_CHECK(expr) do { hipError_t _e = (expr); if (_e != hipSuccess) { \
    fprintf(stderr, "HIP error %s at %s:%d\n", hipGetErrorString(_e), __FILE__, __LINE__); \
    return; } } while (0)

namespace {
#define LAS __attribute__((address_space(3)))
typedef short s16x4 __attribute__((ext_vector_type(4)));
typedef short s16x8 __attribute__((ext_vector_type(8)));
typedef __bf16 bf16x8 __attribute__((ext_vector_type(8)));
typedef float f32x4 __attribute__((ext_vector_type(4)));
typedef unsigned u32x4 __attribute__((ext_vector_type(4)));
typedef unsigned long long u64;

constexpr int NR = 8, MLOC = 512, KD = 4096, NG = 16384, NLOC = 2048;
constexpr int MF1 = 5, MF2 = 4;
constexpr int RROWS = 64 * MF1, FROWS = MLOC - RROWS;
#define NS2 5
constexpr int AMAX = 32 * MF1 * 128, SLOT = AMAX + 16384, RING = 4 * SLOT;
constexpr int AMAX2 = 32 * MF2 * 128, SLOT2 = AMAX2 + 16384, RING2 = NS2 * SLOT2, LDS_BYTES = RING > RING2 ? RING : RING2;
static_assert(FROWS == 192 && LDS_BYTES <= 163840, "tiles");
constexpr unsigned XPITCH = KD * 2, WPITCH = NG * 2, OPITCH = NLOC * 2;
constexpr unsigned OUT_BYTES = (unsigned)NR * MLOC * OPITCH;
constexpr unsigned XBLK = FROWS * XPITCH;
constexpr unsigned WS_CTR = 0;
constexpr unsigned WS_TMO = 32768;
constexpr unsigned WS_XFLAG = 36864;
constexpr unsigned WS_DONE = 49152;
constexpr unsigned WS_X = 1u << 20;
constexpr unsigned WS_NEED = WS_X + NR * XBLK;
constexpr u64 DEADLINE = 40000000ull;
#define AUXO 17
constexpr int AUX_SYS = AUXO;

#define GL 6
constexpr int NIN = 63 / GL < 2 * MF1 ? 63 / GL : 2 * MF1;

struct Params { const char* x; const char* w; char* out[NR]; char* ws[NR]; int rank; };
struct Tile { int c, r, peer, own, pl1, pl2, row1, src2, row2, rot1, rot2; };

__device__ __forceinline__ unsigned pk_bf16(float lo, float hi) {
  typedef __bf16 bf16x2 __attribute__((ext_vector_type(2)));
  bf16x2 v = {(__bf16)lo, (__bf16)hi};
  return __builtin_bit_cast(unsigned, v);
}
constexpr float GELU_A = -2.0f * 0.7978845608028654f * 1.4426950408889634f, GELU_B = GELU_A * 0.044715f;
__device__ __forceinline__ float gelu(float y) {
  const float e = __builtin_amdgcn_exp2f(y * __builtin_fmaf(GELU_B, y * y, GELU_A));
  return y * __builtin_amdgcn_rcpf(1.0f + e);
}
__device__ __forceinline__ u32x4 epi8(const f32x4 u, const f32x4 v, bool odd) {
  const unsigned a0 = pk_bf16(gelu(u[0]), gelu(u[1])), a1 = pk_bf16(gelu(u[2]), gelu(u[3]));
  const unsigned b0 = pk_bf16(gelu(v[0]), gelu(v[1])), b1 = pk_bf16(gelu(v[2]), gelu(v[3]));
  u32x4 w; w.x = odd ? b0 : a0; w.y = odd ? b1 : a1; w.z = odd ? a0 : b0; w.w = odd ? a1 : b1;
  return w;
}
template <int N_> __device__ __forceinline__ void waitv() { asm volatile("s_waitcnt vmcnt(%0)" :: "i"(N_) : "memory"); }
#define WAIT_L0() __builtin_amdgcn_s_waitcnt(0xC07F)
#define BAR() __builtin_amdgcn_s_barrier()
#define SCHED() __builtin_amdgcn_sched_barrier(0)
template <int I, int N, class F> __device__ __forceinline__ void static_for(F&& f) { if constexpr (I < N) { f(std::integral_constant<int, I>{}); static_for<I + 1, N>(f); } }
__device__ __forceinline__ LAS char* lp(unsigned a) { return (LAS char*)(size_t)a; }
__device__ __forceinline__ auto rsrc(const void* base, unsigned bytes) { return __builtin_amdgcn_make_buffer_rsrc((void*)base, 0, (int)bytes, 0x00020000); }

struct Frag { s16x8 a[MF1]; s16x4 blo[4], bhi[4]; };
template <int K, int KS> __device__ __forceinline__ void rd1(Frag& f, unsigned aaddr, const unsigned (&baddr)[4]) {
  if constexpr (K == 2) f.a[0] = *(LAS s16x8*)lp(aaddr);
  else if constexpr (K >= 9) f.a[K - 8] = *(LAS s16x8*)(lp(aaddr) + (K - 8) * 2048);
  else {
    constexpr int q = K < 2 ? K : K - 1, gn = q >> 1;
    if constexpr (q & 1) f.bhi[gn] = __builtin_amdgcn_ds_read_tr16_b64_v4i16((LAS s16x4*)(lp(baddr[gn]) + KS * 8192 + 1024));
    else f.blo[gn] = __builtin_amdgcn_ds_read_tr16_b64_v4i16((LAS s16x4*)(lp(baddr[gn]) + KS * 8192));
  }
}
template <int Z, int MF> __device__ __forceinline__ void mm1(f32x4 (&acc)[MF][4], const Frag& f) {
  constexpr int m = Z >> 2, gn = Z & 3;
  const s16x8 b = __builtin_shufflevector(f.blo[gn], f.bhi[gn], 0, 1, 2, 3, 4, 5, 6, 7);
  asm volatile("v_mfma_f32_16x16x32_bf16 %0, %1, %2, %0" : "+v"(acc[m][gn]) : "v"(b), "v"(f.a[m]));
}
struct GeluState { f32x4 cu, cv; float ga, gb, ka; unsigned pk0, pk1, pk2, pk3, ov0, ov1, ov2, ov3; };
template <int F> __device__ __forceinline__ void gelu_piece(GeluState& g) {
  constexpr int pr = F / 10, st = F % 10;
  constexpr u64 ODDM = 0xffff0000ffff0000ull;
  if constexpr (F < 40) {
    const float ya = pr < 2 ? g.cu[2 * (pr & 1)] : g.cv[2 * (pr & 1)], yb = pr < 2 ? g.cu[2 * (pr & 1) + 1] : g.cv[2 * (pr & 1) + 1];
    if constexpr (st == 0) asm volatile("v_mul_f32 %0, %2, %2\n\tv_mul_f32 %1, %3, %3" : "=&v"(g.ga), "=v"(g.gb) : "v"(ya), "v"(yb));
    if constexpr (st == 1) asm volatile("v_fmamk_f32 %0, %0, %3, %2\n\tv_fmamk_f32 %1, %1, %3, %2" : "+v"(g.ga), "+v"(g.gb) : "v"(g.ka), "i"(__builtin_bit_cast(unsigned, GELU_B)));
    if constexpr (st == 2 || st == 8) asm volatile("v_mul_f32 %0, %2, %0\n\tv_mul_f32 %1, %3, %1" : "+v"(g.ga), "+v"(g.gb) : "v"(ya), "v"(yb));
    if constexpr (st == 3) asm volatile("v_exp_f32 %0, %0" : "+v"(g.ga));
    if constexpr (st == 4) asm volatile("v_exp_f32 %0, %0" : "+v"(g.gb));
    if constexpr (st == 5) asm volatile("v_add_f32 %0, 1.0, %0\n\tv_add_f32 %1, 1.0, %1" : "+v"(g.ga), "+v"(g.gb));
    if constexpr (st == 6) asm volatile("v_rcp_f32 %0, %0" : "+v"(g.ga));
    if constexpr (st == 7) asm volatile("v_rcp_f32 %0, %0" : "+v"(g.gb));
    if constexpr (st == 9) { unsigned& d = pr == 0 ? g.pk0 : pr == 1 ? g.pk1 : pr == 2 ? g.pk2 : g.pk3; asm volatile("v_cvt_pk_bf16_f32 %0, %1, %2" : "=v"(d) : "v"(g.ga), "v"(g.gb)); }
  } else if constexpr (F == 40) asm volatile("v_cndmask_b32_e64 %0, %2, %4, %6\n\tv_cndmask_b32_e64 %1, %3, %5, %6" : "=&v"(g.ov0), "=v"(g.ov1) : "v"(g.pk0), "v"(g.pk1), "v"(g.pk2), "v"(g.pk3), "s"(ODDM));
  else if constexpr (F == 41) asm volatile("v_cndmask_b32_e64 %0, %4, %2, %6\n\tv_cndmask_b32_e64 %1, %5, %3, %6" : "=&v"(g.ov2), "=v"(g.ov3) : "v"(g.pk0), "v"(g.pk1), "v"(g.pk2), "v"(g.pk3), "s"(ODDM));
}
template <int G> __device__ __forceinline__ void pick(GeluState& s, const f32x4 (&acc)[MF1][4]) {
  constexpr int m = G >> 1, h = G & 1;
#pragma unroll
  for (int i = 0; i < 4; ++i) {
    const float u0 = acc[m][2 * h][i], u1 = acc[m][2 - 2 * h][i], v0 = acc[m][2 * h + 1][i], v1 = acc[m][3 - 2 * h][i];
    s.cu[i] = __int_as_float(__builtin_amdgcn_update_dpp(__float_as_int(u0), __float_as_int(u1), 0x128, 0xf, h ? 0x3 : 0xc, false));
    s.cv[i] = __int_as_float(__builtin_amdgcn_update_dpp(__float_as_int(v0), __float_as_int(v1), 0x128, 0xf, h ? 0x3 : 0xc, false));
  }
}
constexpr int dbl_gaps(int mf) { return mf == 2 ? 5 : mf == 3 ? 4 : mf == 4 ? 1 : 0; }

#define STAMP(i)
#define NEED(n) (n)

#define AUXX 17
#define XFLY 3
__device__ __forceinline__ void xpush(const Params& p, const Tile& t, int wid, int lane) {
  const u64 wsl = (u64)p.ws[(t.r + 1 + (lane < 7 ? lane : 0)) & 7];
  const int w = ((t.c >> 3) - 28) * 64 + (t.c & 7) * 8 + wid;
  u32x4 v[2][3]; unsigned po[3];
#pragma unroll
  for (int i = 0; i < 3; ++i) { const int q = 3 * w + i; po[i] = (q >> 2) * XPITCH + (q & 3) * 1024 + lane * 16;
    v[0][i] = *(const u32x4*)(p.x + (size_t)RROWS * XPITCH + po[i]); v[1][i] = *(const u32x4*)(p.x + (size_t)RROWS * XPITCH + po[i] + 4096); }
#pragma unroll
  for (int h = 0; h < 2; ++h) {
    int e = w % 7;
#pragma unroll 1
    for (int n = 0; n < 7; ++n) {
      char* base = (char*)(((u64)(unsigned)__builtin_amdgcn_readlane((unsigned)(wsl >> 32), e) << 32) | (unsigned)__builtin_amdgcn_readlane((unsigned)wsl, e));
      const auto rs = rsrc(base + WS_X + t.r * XBLK, XBLK);
#pragma unroll
      for (int i = 0; i < 3; ++i) { __builtin_amdgcn_raw_buffer_store_b128(v[h][i], rs, po[i], h * 4096, AUXX); waitv<XFLY>(); }
      e = e == 6 ? 0 : e + 1;
    }
    waitv<0>(); BAR();
    if (wid == 0 && lane < 7) __hip_atomic_fetch_add((unsigned*)((char*)wsl + WS_XFLAG + (t.r * 2 + h) * 128), 1u, __ATOMIC_RELAXED, __HIP_MEMORY_SCOPE_SYSTEM);
  }
}

__device__ __forceinline__ void producer(const Params& p, const Tile& t, LAS char* lds, int pw, int lane) {
  char* const myws = p.ws[t.r];
  const unsigned va0 = (lane >> 3) * XPITCH + (((lane & 7) ^ (lane >> 4)) << 4), va1 = va0 ^ 64;
  const unsigned vb0 = (lane >> 4) * WPITCH + (((lane & 15) ^ (((lane >> 4) & 1) | ((lane >> 5) << 2))) << 4), vb1 = vb0 ^ 128;
  const unsigned ve0 = va0 ^ ((pw & 1) << 6), ve1 = ve0 ^ 64;
  const auto ra1 = rsrc(p.x + (size_t)(t.row1 + 40 * pw) * XPITCH, 40 * XPITCH);
  const auto rb1 = rsrc(p.w + (size_t)(t.peer * 16 + t.pl1) * 256, (KD - 1) * WPITCH + 256);
  const int s2 = t.src2, r2 = t.row2 + 32 * (pw & 1);
  const auto ra2 = rsrc(s2 == t.r ? p.x + (size_t)r2 * XPITCH : myws + WS_X + s2 * XBLK + (size_t)(r2 - RROWS) * XPITCH, t.row2 < 0 ? 0 : 32 * XPITCH);
  const auto rb2 = rsrc(p.w + (size_t)(t.r * 16 + t.pl2) * 256, (KD - 1) * WPITCH + 256);
  unsigned so = 0;
  auto issue1 = [&](int s) {
    const int kt = (s & 60) | ((s + t.rot1) & 3);
#pragma unroll
    for (int i = 0; i < MF1; ++i) __builtin_amdgcn_raw_ptr_buffer_load_lds(ra1, lds + so + (pw * MF1 + i) * 1024, 16, (i & 1) ? ve1 : ve0, kt * 128 + i * 8 * XPITCH, 0, 0);
#pragma unroll
    for (int i = 0; i < 4; ++i) __builtin_amdgcn_raw_ptr_buffer_load_lds(rb1, lds + so + AMAX + (pw * 4 + i) * 1024, 16, (i & 2) ? vb1 : vb0, (kt * 64 + pw * 16 + i * 4) * WPITCH, 0, 0);
    so = so + SLOT == RING ? 0 : so + SLOT;
  };
  auto issue2 = [&](int s) {
    const int kt = (s & 60) | ((s + t.rot2) & 3);
#pragma unroll
    for (int i = 0; i < MF2; ++i) __builtin_amdgcn_raw_ptr_buffer_load_lds(ra2, lds + so + (pw * MF2 + i) * 1024, 16, (i & 1) ? va1 : va0, kt * 128 + i * 8 * XPITCH, 0, 0);
#pragma unroll
    for (int i = 0; i < 4; ++i) __builtin_amdgcn_raw_ptr_buffer_load_lds(rb2, lds + so + AMAX2 + (pw * 4 + i) * 1024, 16, (i & 2) ? vb1 : vb0, (kt * 64 + pw * 16 + i * 4) * WPITCH, 0, 0);
    so = so + SLOT2 == RING2 ? 0 : so + SLOT2;
  };
  constexpr int L1 = MF1 + 4, L2 = MF2 + 4;
  unsigned callv = 0, fv = 0;
  const bool poller = (pw == 0) && !t.own;
  auto poll = [&](int q) {
    if (poller) { const char* fp = myws + WS_XFLAG + (t.peer * 2 + q) * 128; asm volatile("global_load_dword %0, %1, off sc0 sc1" : "+v"(fv) : "v"(fp) : "memory"); }
  };
  auto gate = [&](int q) {
    if (poller) {
      asm volatile("" : "+v"(fv), "+v"(callv));
      const unsigned call = __builtin_amdgcn_readfirstlane(callv) + 1, need = NEED(32u * call);
      if (__any((int)(fv - need) < 0)) {
        unsigned* const tmo = (unsigned*)(myws + WS_TMO);
        const u64 t0 = __builtin_amdgcn_s_memrealtime();
        while (__any((int)(__hip_atomic_load((unsigned*)(myws + WS_XFLAG + (t.peer * 2 + q) * 128), __ATOMIC_RELAXED, __HIP_MEMORY_SCOPE_SYSTEM) - need) < 0)) {
          if (__hip_atomic_load(tmo, __ATOMIC_RELAXED, __HIP_MEMORY_SCOPE_AGENT) == call || __builtin_amdgcn_s_memrealtime() - t0 > DEADLINE) {
            __hip_atomic_store(tmo, call, __ATOMIC_RELAXED, __HIP_MEMORY_SCOPE_AGENT);
            waitv<0>();
            break;
          }
          __builtin_amdgcn_s_sleep(2);
        }
      }
      asm volatile("buffer_inv sc0 sc1" ::: "memory");
    }
  };
  if (!t.own) {
    issue1(0); issue1(1); issue1(2); issue1(3);
    { const char* ctr = myws + WS_CTR + t.c * 128; asm volatile("global_load_dword %0, %1, off sc1" : "+v"(callv) : "v"(ctr) : "memory"); }
    waitv<3 * L1 + 1>(); BAR();
    waitv<2 * L1 + 1>(); BAR(); issue1(4);
#pragma unroll 1
    for (int i = 1; i < 60; ++i) {
      waitv<2 * L1>();
      if (i == 58) gate(0);
      BAR();
      if (i == 56) poll(0);
      issue1(i + 4);
    }
    waitv<2 * L1>(); BAR(); issue2(0);
    waitv<L1 + L2>(); BAR(); issue2(1);
    waitv<2 * L2>(); BAR(); issue2(2);
    waitv<2 * L2>(); BAR(); issue2(3);
    if (NS2 == 5) issue2(4);
  } else {
    waitv<0>();
#pragma unroll
    for (int n = 0; n < NS2; ++n) issue2(n);
    waitv<(NS2 - 1) * L2>(); BAR();
  }
  constexpr int W2 = (NS2 - 2) * L2, IG = 32 - NS2 - (NS2 - 2), IP = IG - (NS2 - 2);
#pragma unroll 1
  for (int i = 0; i < 64 - NS2; ++i) {
    waitv<W2>();
    if (i == IG) gate(1);
    BAR();
    if (i == IP) poll(1);
    issue2(i + NS2);
  }
  static_for<0, NS2 - 1>([&](auto Q) { waitv<(NS2 - 2 - Q) * L2>(); BAR(); });
  BAR();
  asm volatile("" :: "v"(callv), "v"(fv));
}

__device__ __forceinline__ void consumer(const Params& p, const Tile& t, LAS char* lds, int wid, int lane) {
  char* const myws = p.ws[t.r];
  __builtin_amdgcn_s_setprio(2);
  const int wr = wid >> 1, wc = wid & 1, fr = lane & 15, fq = lane >> 4;
  const bool odd = fq & 1;
  const unsigned callv = __hip_atomic_load((unsigned*)(myws + WS_CTR + t.c * 128), __ATOMIC_RELAXED, __HIP_MEMORY_SCOPE_AGENT);

  const unsigned lb = (unsigned)(size_t)lds;
  unsigned a1[2], a2[2], baddr[4];
#pragma unroll
  for (int ks = 0; ks < 2; ++ks) { const unsigned u = lb + fr * 128 + 16 * ((4 * ks + fq) ^ ((fr >> 1) & 7)); a1[ks] = u + 16 * MF1 * wr * 128; a2[ks] = u + 16 * MF2 * wr * 128; }
  { const int q = (lane & 15) >> 2, pp = lane & 3, kr = 8 * fq + q, f = (kr & 1) | ((kr & 2) << 1) | (kr & 8);
#pragma unroll
    for (int gn = 0; gn < 4; ++gn) baddr[gn] = lb + kr * 256 + 16 * ((8 * wc + 4 * (gn >> 1) + pp) ^ f) + 8 * ((gn & 1) ^ (pp & 1)); }
  const auto rs1 = rsrc(p.out[t.peer], t.own ? 0 : OUT_BYTES);
  const unsigned o1 = (t.r * MLOC + t.row1 + 16 * MF1 * wr + (fr & 7)) * OPITCH + (t.pl1 * 128 + 64 * wc + 32 * (fr >> 3) + 8 * fq) * 2;
#define OSTEP(g) (8 * OPITCH)
#define OADDR(g) ((g) * 8 * OPITCH)
  const auto rs2 = rsrc(p.out[t.r], t.row2 < 0 ? 0 : OUT_BYTES);
  const unsigned o2 = (t.src2 * MLOC + t.row2 + fr) * OPITCH + (t.pl2 * 128 + 64 * wc + 8 * fq) * 2;

  Frag F0, F1;
  f32x4 acc1[MF1][4], acc2[MF2][4];
#pragma unroll
  for (int m = 0; m < MF1; ++m)
#pragma unroll
    for (int gn = 0; gn < 4; ++gn) { acc1[m][gn] = (f32x4){0.f, 0.f, 0.f, 0.f}; asm volatile("" : "+v"(acc1[m][gn])); }
  unsigned so = 0, sw, bo = t.own ? AMAX2 : AMAX, bc[4], ac[2];
#pragma unroll
  for (int gn = 0; gn < 4; ++gn) bc[gn] = baddr[gn] + bo;
  ac[0] = t.own ? a2[0] : a1[0]; ac[1] = t.own ? a2[1] : a1[1];
#pragma unroll
  for (int m = 0; m < MF2; ++m)
#pragma unroll
    for (int gn = 0; gn < 4; ++gn) { acc2[m][gn] = (f32x4){0.f, 0.f, 0.f, 0.f}; asm volatile("" : "+v"(acc2[m][gn])); }
  asm volatile("s_nop 1");
  BAR();
  STAMP(1);
  static_for<0, 8 + MF1>([&](auto K) { rd1<K, 0>(F0, ac[0], bc); });
  GeluState gs; gs.ka = GELU_A; asm volatile("" : "+v"(gs.ka));
  auto half_a = [&](auto MF_, auto& acc, const unsigned (&an)[2], auto EV, auto G0_) {
    constexpr int MF = MF_, NM = 4 * MF, NRD = 8 + MF, DBL = dbl_gaps(MF), LAST = NRD - DBL - 1, G0 = G0_;
    static_for<0, NM>([&](auto Z) {
      constexpr int z = Z;
      SCHED(); mm1<z, MF>(acc, F0); SCHED();
      if constexpr (z < DBL) { rd1<2 * z, 1>(F1, ac[1], bc); rd1<2 * z + 1, 1>(F1, ac[1], bc); }
      else if constexpr (z <= LAST) rd1<z + DBL, 1>(F1, ac[1], bc);
      if constexpr (z == LAST - 1) { if constexpr (MF == MF1) so = so + SLOT == RING ? 0 : so + SLOT; else so = so + SLOT2 == RING2 ? 0 : so + SLOT2; sw = so + bo; asm volatile("" : "+s"(so), "+s"(sw)); }
      if constexpr (z == LAST + 1) { bc[0] = baddr[0] + sw; bc[1] = baddr[1] + sw; asm volatile("" : "+v"(bc[0]), "+v"(bc[1])); }
      if constexpr (z == LAST + 2) { bc[2] = baddr[2] + sw; bc[3] = baddr[3] + sw; asm volatile("" : "+v"(bc[2]), "+v"(bc[3])); }
      if constexpr (z == LAST + 3) { ac[0] = an[0] + so; ac[1] = an[1] + so; asm volatile("" : "+v"(ac[0]), "+v"(ac[1])); }
      if constexpr (G0 >= 0 && z > LAST + 3) gelu_piece<G0 + z - LAST - 4>(gs);
    });
    static_assert(LAST + 3 < NM, "half step too short");
    SCHED();
  };
  auto half_b = [&](auto MF_, auto MFN_, auto& acc, auto ZERO, auto G0_) {
    constexpr int MF = MF_, MFN = MFN_, NM = 4 * MF, NRD = 8 + MFN, DBL = dbl_gaps(MF), G0 = G0_;
    SCHED(); WAIT_L0(); BAR();
    static_for<0, NM>([&](auto Z) {
      constexpr int z = Z;
      SCHED(); mm1<z, MF>(acc, F1); SCHED();
      if constexpr (z < DBL) { rd1<2 * z, 0>(F0, ac[0], bc); rd1<2 * z + 1, 0>(F0, ac[0], bc); }
      else if constexpr (z + DBL < NRD) rd1<z + DBL, 0>(F0, ac[0], bc);
      else if constexpr (G0 >= 0) gelu_piece<G0 + z + DBL - NRD>(gs);
    });
    SCHED();
  };
  constexpr std::integral_constant<int, MF1> c1{}; constexpr std::integral_constant<int, MF2> c2{};
  constexpr std::integral_constant<int, -1> nog{}; constexpr std::true_type yes{}; constexpr std::false_type no{};
  int it = 63;
  if (!t.own) {
#pragma unroll 1
    do {
      half_a(c1, acc1, a1, no, nog);
      half_b(c1, c1, acc1, no, nog);
      --it; asm volatile("" : "+s"(it));
    } while (it != 0);
    bo = AMAX2;
    half_a(c1, acc1, a2, no, nog);
    half_b(c1, c2, acc1, no, nog);
    asm volatile("s_nop 15" ::: "memory"); SCHED();
  }
  so = __builtin_amdgcn_readfirstlane(so); bo = AMAX2;
  STAMP(2);
  constexpr int FA = 4 * MF2 - (8 + MF2 - dbl_gaps(MF2)) - 3, FB = 4 * MF2 - (8 + MF2 - dbl_gaps(MF2));
  static_assert(GL * (FA + FB) >= 42 && NIN * GL < 64, "GELU pieces do not fit");
  constexpr std::integral_constant<int, GL * FB + (GL - 1) * FA> gtail{};
  unsigned oo = o1;
  it = 0;
#pragma unroll 1
  do {
    half_a(c2, acc2, a2, no, gtail);
    if (it) { __builtin_amdgcn_raw_buffer_store_b128((u32x4){gs.ov0, gs.ov1, gs.ov2, gs.ov3}, rs1, oo, 0, AUX_SYS); oo += OSTEP(it); }
#define PICK(g) case g: pick<(g < NIN ? g : 0)>(gs, acc1); break
    switch (it) { PICK(0); PICK(1); PICK(2); PICK(3); PICK(4); PICK(5); PICK(6); PICK(7); PICK(8); PICK(9); default: __builtin_unreachable(); }
    asm volatile("" : "+v"(gs.cu), "+v"(gs.cv));
    half_b(c2, c2, acc2, no, std::integral_constant<int, 0>{});
    static_for<1, GL>([&](auto K) { constexpr int k = K;
      half_a(c2, acc2, a2, no, std::integral_constant<int, FB + (k - 1) * (FA + FB)>{});
      half_b(c2, c2, acc2, no, std::integral_constant<int, k * (FA + FB)>{}); });
    ++it; asm volatile("" : "+s"(it));
  } while (it != NIN);
  half_a(c2, acc2, a2, no, gtail);
  __builtin_amdgcn_raw_buffer_store_b128((u32x4){gs.ov0, gs.ov1, gs.ov2, gs.ov3}, rs1, oo, 0, AUX_SYS);
  half_b(c2, c2, acc2, no, nog);
  if constexpr (NIN * GL + 1 < 64) {
    it = 63 - NIN * GL;
#pragma unroll 1
    do { half_a(c2, acc2, a2, no, nog); half_b(c2, c2, acc2, no, nog); --it; asm volatile("" : "+s"(it)); } while (it != 0);
  }
  asm volatile("s_nop 15" ::: "memory"); SCHED();
  STAMP(3);
  static_for<NIN, 2 * MF1>([&](auto G) { constexpr int g = G;
    pick<g>(gs, acc1); __builtin_amdgcn_raw_buffer_store_b128(epi8(gs.cu, gs.cv, odd), rs1, o1 + OADDR(g), 0, AUX_SYS); });
  const unsigned tmov = __hip_atomic_load((unsigned*)(myws + WS_TMO), __ATOMIC_RELAXED, __HIP_MEMORY_SCOPE_AGENT);
  const unsigned call = __builtin_amdgcn_readfirstlane(callv) + 1;
  SCHED();
#define AUXL AUX_SYS
#define DONEAT 4
  static_for<0, 2 * MF2>([&](auto G) { constexpr int g = G;
    if constexpr (g == DONEAT) {
      SCHED(); waitv<DONEAT>();
      if (!t.own && lane == 0) __hip_atomic_fetch_add((unsigned*)(p.ws[t.peer] + WS_DONE + (t.r * 4 + wid) * 128), 1u, __ATOMIC_RELAXED, __HIP_MEMORY_SCOPE_SYSTEM);
      SCHED(); STAMP(4);
    }
    __builtin_amdgcn_raw_buffer_store_b128(epi8(acc2[g >> 1][2 * (g & 1)], acc2[g >> 1][2 * (g & 1) + 1], odd), rs2, o2 + (g >> 1) * 16 * OPITCH + (g & 1) * 64, 0, AUXL); });
  STAMP(5);
  if (wid == 0 && lane == 0) __hip_atomic_store((unsigned*)(myws + WS_CTR + t.c * 128), call, __ATOMIC_RELAXED, __HIP_MEMORY_SCOPE_AGENT);
  bool bad = tmov == call;
  if (t.c == 255 && wid == 0) {
    unsigned* const tmo = (unsigned*)(myws + WS_TMO);
    unsigned* const fp = (unsigned*)(myws + WS_DONE + (((t.r + 1 + (lane < 28 ? lane >> 2 : 0)) & 7) * 4 + (lane & 3)) * 128);
    const unsigned need = NEED(32u * call);
    const u64 t0 = __builtin_amdgcn_s_memrealtime();
    for (unsigned i = 1; __any((int)(__hip_atomic_load(fp, __ATOMIC_RELAXED, __HIP_MEMORY_SCOPE_SYSTEM) - need) < 0); ++i) {
      if ((i & 15) == 0 && (__hip_atomic_load(tmo, __ATOMIC_RELAXED, __HIP_MEMORY_SCOPE_AGENT) == call || __builtin_amdgcn_s_memrealtime() - t0 > DEADLINE)) {
        __hip_atomic_store(tmo, call, __ATOMIC_RELAXED, __HIP_MEMORY_SCOPE_AGENT);
        bad = true;
        break;
      }
      __builtin_amdgcn_s_sleep(1);
    }
  }
  if (__builtin_expect(bad, 0)) {
    waitv<0>();
    for (int g = 0; g < 2 * MF2; ++g)
      __builtin_amdgcn_raw_buffer_store_b128((u32x4){0x7fc07fc0u, 0x7fc07fc0u, 0x7fc07fc0u, 0x7fc07fc0u}, rs2, o2 + (g >> 1) * 16 * OPITCH + (g & 1) * 64, 0, AUX_SYS);
  }
  STAMP(6);
}
}

__global__ __launch_bounds__(512, 2) void gemm_a2a(Params p) {
  extern __shared__ __attribute__((aligned(16))) char smem[];
  LAS char* lds = (LAS char*)smem;
  const int lane = threadIdx.x & 63, wid = __builtin_amdgcn_readfirstlane(threadIdx.x >> 6);
  Tile t;
  t.c = blockIdx.x; t.r = p.rank;
  const int xcd = t.c & 7, j = t.c >> 3, e = j >> 2, hm = j & 1, pb = (j >> 1) & 1, h2 = xcd & 1;
  t.own = j >= 28;
  t.peer = (t.r + 1 + e) & 7;
  t.pl1 = xcd + 8 * pb;
  t.pl2 = 4 * (xcd >> 1) + (j & 3);
  t.row1 = 32 * MF1 * hm;
  if (((wid >> 1) & 1) == 0) { t.src2 = t.peer; t.row2 = t.own ? 128 * h2 : RROWS + 128 * h2; }
  else { t.src2 = h2 ? t.r : t.peer; t.row2 = t.own ? 128 * h2 + 64 : h2 ? (e < 4 ? 256 + 64 * e : -1) : RROWS + 64; }
#define R1M 2
#define R1J 1
#define R2M 1
  t.rot1 = R1M * hm + R1J * (j >> 1); t.rot2 = R2M * (e + j);
  STAMP(0);
  if (t.own) xpush(p, t, wid, lane);
  if (wid >= 4) producer(p, t, lds, wid - 4, lane); else consumer(p, t, lds, wid, lane);
}

extern "C" void kernel_launch(int rank, int nranks, ncclComm_t comm,
                              hipStream_t stream,
                              void* const* d_in, const int* in_sizes, int n_in,
                              void* d_out, int out_size,
                              const int* in_dtypes, int out_dtype,
                              void* const* d_out_all,
                              void* const* d_ws_all, size_t ws_bytes) {
  if (nranks != NR || (unsigned)rank >= (unsigned)NR || n_in != 2 || in_sizes[0] != MLOC * KD || in_sizes[1] != KD * NG ||
      out_size != NR * MLOC * NLOC || ws_bytes < WS_NEED) { fprintf(stderr, "gemm_a2a: unexpected problem shape\n"); return; }
  static bool attr_set[NR] = {};
  if (!attr_set[rank]) { HIP_CHECK(hipFuncSetAttribute((const void*)gemm_a2a, hipFuncAttributeMaxDynamicSharedMemorySize, LDS_BYTES)); attr_set[rank] = true; }
  Params p;
  p.x = (const char*)d_in[0]; p.w = (const char*)d_in[1]; p.rank = rank;
  for (int i = 0; i < NR; ++i) { p.out[i] = (char*)d_out_all[i]; p.ws[i] = (char*)d_ws_all[i]; }
  hipLaunchKernelGGL(gemm_a2a, dim3(256), dim3(512), LDS_BYTES, stream, p);
  HIP_CHECK(hipGetLastError());
}
```

```cpp
#include <hip/hip_runtime.h>
#include <rccl/rccl.h>
#include <cstdio>
#include <type_traits>

#define HIP_CHECK(expr) do { hipError_t _e = (expr); if (_e != hipSuccess) { \
    fprintf(stderr, "HIP error %s at %s:%d\n", hipGetErrorString(_e), __FILE__, __LINE__); \
    return; } } while (0)

namespace {
#define LAS __attribute__((address_space(3)))
typedef short s16x4 __attribute__((ext_vector_type(4)));
typedef short s16x8 __attribute__((ext_vector_type(8)));
typedef __bf16 bf16x8 __attribute__((ext_vector_type(8)));
typedef float f32x4 __attribute__((ext_vector_type(4)));
typedef unsigned u32x4 __attribute__((ext_vector_type(4)));
typedef unsigned long long u64;

constexpr int NR = 8, MLOC = 512, KD = 4096, NG = 16384, NLOC = 2048;
constexpr int MF1 = 5, MF2 = 4;
constexpr int RROWS = 64 * MF1, FROWS = MLOC - RROWS;
#define NS2 5
constexpr int AMAX = 32 * MF1 * 128, SLOT = AMAX + 16384, RING = 4 * SLOT;
constexpr int AMAX2 = 32 * MF2 * 128, SLOT2 = AMAX2 + 16384, RING2 = NS2 * SLOT2, LDS_BYTES = RING > RING2 ? RING : RING2;
static_assert(FROWS == 192 && LDS_BYTES <= 163840, "tiles");
constexpr unsigned XPITCH = KD * 2, WPITCH = NG * 2, OPITCH = NLOC * 2;
constexpr unsigned OUT_BYTES = (unsigned)NR * MLOC * OPITCH;
constexpr unsigned XBLK = FROWS * XPITCH;
constexpr unsigned WS_CTR = 0;
constexpr unsigned WS_TMO = 32768;
constexpr unsigned WS_XFLAG = 36864;
constexpr unsigned WS_DONE = 49152;
constexpr unsigned WS_X = 1u << 20;
constexpr unsigned WS_NEED = WS_X + 2 * NR * XBLK;
constexpr u64 DEADLINE = 40000000ull;
#define AUXO 17
constexpr int AUX_SYS = AUXO;

#define GL 6
#define G0N 1
#define DLAG 3

struct Params { const char* x; const char* w; char* out[NR]; char* ws[NR]; int rank; };
struct Tile { int c, r, peer, own, pl1, pl2, row1, src2, row2, rot1, rot2; };

__device__ __forceinline__ unsigned pk_bf16(float lo, float hi) {
  typedef __bf16 bf16x2 __attribute__((ext_vector_type(2)));
  bf16x2 v = {(__bf16)lo, (__bf16)hi};
  return __builtin_bit_cast(unsigned, v);
}
constexpr float GELU_A = -2.0f * 0.7978845608028654f * 1.4426950408889634f, GELU_B = GELU_A * 0.044715f;
__device__ __forceinline__ float gelu(float y) {
  const float e = __builtin_amdgcn_exp2f(y * __builtin_fmaf(GELU_B, y * y, GELU_A));
  return y * __builtin_amdgcn_rcpf(1.0f + e);
}
__device__ __forceinline__ u32x4 epi8(const f32x4 u, const f32x4 v, bool odd) {
  const unsigned a0 = pk_bf16(gelu(u[0]), gelu(u[1])), a1 = pk_bf16(gelu(u[2]), gelu(u[3]));
  const unsigned b0 = pk_bf16(gelu(v[0]), gelu(v[1])), b1 = pk_bf16(gelu(v[2]), gelu(v[3]));
  u32x4 w; w.x = odd ? b0 : a0; w.y = odd ? b1 : a1; w.z = odd ? a0 : b0; w.w = odd ? a1 : b1;
  return w;
}
template <int N_> __device__ __forceinline__ void waitv() { asm volatile("s_waitcnt vmcnt(%0)" :: "i"(N_) : "memory"); }
#define WAIT_L0() __builtin_amdgcn_s_waitcnt(0xC07F)
#define BAR() __builtin_amdgcn_s_barrier()
#define SCHED() __builtin_amdgcn_sched_barrier(0)
template <int I, int N, class F> __device__ __forceinline__ void static_for(F&& f) { if constexpr (I < N) { f(std::integral_constant<int, I>{}); static_for<I + 1, N>(f); } }
__device__ __forceinline__ LAS char* lp(unsigned a) { return (LAS char*)(size_t)a; }
__device__ __forceinline__ auto rsrc(const void* base, unsigned bytes) { return __builtin_amdgcn_make_buffer_rsrc((void*)base, 0, (int)bytes, 0x00020000); }

struct Frag { s16x8 a[MF1]; s16x4 blo[4], bhi[4]; };
template <int K, int KS> __device__ __forceinline__ void rd1(Frag& f, unsigned aaddr, const unsigned (&baddr)[4]) {
  if constexpr (K == 2) f.a[0] = *(LAS s16x8*)lp(aaddr);
  else if constexpr (K >= 9) f.a[K - 8] = *(LAS s16x8*)(lp(aaddr) + (K - 8) * 2048);
  else {
    constexpr int q = K < 2 ? K : K - 1, gn = q >> 1;
    if constexpr (q & 1) f.bhi[gn] = __builtin_amdgcn_ds_read_tr16_b64_v4i16((LAS s16x4*)(lp(baddr[gn]) + KS * 8192 + 1024));
    else f.blo[gn] = __builtin_amdgcn_ds_read_tr16_b64_v4i16((LAS s16x4*)(lp(baddr[gn]) + KS * 8192));
  }
}
template <int Z, int MF> __device__ __forceinline__ void mm1(f32x4 (&acc)[MF][4], const Frag& f) {
  constexpr int m = Z >> 2, gn = Z & 3;
  const s16x8 b = __builtin_shufflevector(f.blo[gn], f.bhi[gn], 0, 1, 2, 3, 4, 5, 6, 7);
  asm volatile("v_mfma_f32_16x16x32_bf16 %0, %1, %2, %0" : "+v"(acc[m][gn]) : "v"(b), "v"(f.a[m]));
}
struct GeluState { f32x4 cu, cv; float ga, gb, ka; unsigned pk0, pk1, pk2, pk3, ov0, ov1, ov2, ov3; };
template <int F> __device__ __forceinline__ void gelu_piece(GeluState& g) {
  constexpr int pr = F / 10, st = F % 10;
  constexpr u64 ODDM = 0xffff0000ffff0000ull;
  if constexpr (F < 40) {
    const float ya = pr < 2 ? g.cu[2 * (pr & 1)] : g.cv[2 * (pr & 1)], yb = pr < 2 ? g.cu[2 * (pr & 1) + 1] : g.cv[2 * (pr & 1) + 1];
    if constexpr (st == 0) asm volatile("v_mul_f32 %0, %2, %2\n\tv_mul_f32 %1, %3, %3" : "=&v"(g.ga), "=v"(g.gb) : "v"(ya), "v"(yb));
    if constexpr (st == 1) asm volatile("v_fmamk_f32 %0, %0, %3, %2\n\tv_fmamk_f32 %1, %1, %3, %2" : "+v"(g.ga), "+v"(g.gb) : "v"(g.ka), "i"(__builtin_bit_cast(unsigned, GELU_B)));
    if constexpr (st == 2 || st == 8) asm volatile("v_mul_f32 %0, %2, %0\n\tv_mul_f32 %1, %3, %1" : "+v"(g.ga), "+v"(g.gb) : "v"(ya), "v"(yb));
    if constexpr (st == 3) asm volatile("v_exp_f32 %0, %0" : "+v"(g.ga));
    if constexpr (st == 4) asm volatile("v_exp_f32 %0, %0" : "+v"(g.gb));
    if constexpr (st == 5) asm volatile("v_add_f32 %0, 1.0, %0\n\tv_add_f32 %1, 1.0, %1" : "+v"(g.ga), "+v"(g.gb));
    if constexpr (st == 6) asm volatile("v_rcp_f32 %0, %0" : "+v"(g.ga));
    if constexpr (st == 7) asm volatile("v_rcp_f32 %0, %0" : "+v"(g.gb));
    if constexpr (st == 9) { unsigned& d = pr == 0 ? g.pk0 : pr == 1 ? g.pk1 : pr == 2 ? g.pk2 : g.pk3; asm volatile("v_cvt_pk_bf16_f32 %0, %1, %2" : "=v"(d) : "v"(g.ga), "v"(g.gb)); }
  } else if constexpr (F == 40) asm volatile("v_cndmask_b32_e64 %0, %2, %4, %6\n\tv_cndmask_b32_e64 %1, %3, %5, %6" : "=&v"(g.ov0), "=v"(g.ov1) : "v"(g.pk0), "v"(g.pk1), "v"(g.pk2), "v"(g.pk3), "s"(ODDM));
  else if constexpr (F == 41) asm volatile("v_cndmask_b32_e64 %0, %4, %2, %6\n\tv_cndmask_b32_e64 %1, %5, %3, %6" : "=&v"(g.ov2), "=v"(g.ov3) : "v"(g.pk0), "v"(g.pk1), "v"(g.pk2), "v"(g.pk3), "s"(ODDM));
}
template <int G> __device__ __forceinline__ void pick(GeluState& s, const f32x4 (&acc)[MF1][4]) {
  constexpr int m = G >> 1, h = G & 1;
#pragma unroll
  for (int i = 0; i < 4; ++i) {
    const float u0 = acc[m][2 * h][i], u1 = acc[m][2 - 2 * h][i], v0 = acc[m][2 * h + 1][i], v1 = acc[m][3 - 2 * h][i];
    s.cu[i] = __int_as_float(__builtin_amdgcn_update_dpp(__float_as_int(u0), __float_as_int(u1), 0x128, 0xf, h ? 0x3 : 0xc, false));
    s.cv[i] = __int_as_float(__builtin_amdgcn_update_dpp(__float_as_int(v0), __float_as_int(v1), 0x128, 0xf, h ? 0x3 : 0xc, false));
  }
}
constexpr int dbl_gaps(int mf) { return mf == 2 ? 5 : mf == 3 ? 4 : mf == 4 ? 1 : 0; }

#define STAMP(i)
#define NEED(n) (n)

#define AUXX 17
#define XFLY 3
__device__ __forceinline__ void xpush(const Params& p, const Tile& t, int wid, int lane) {
  const u64 wsl = (u64)p.ws[(t.r + 1 + (lane < 7 ? lane : 0)) & 7];
  const int w = t.c * 8 + wid;
  const unsigned par = (__builtin_amdgcn_readfirstlane(__hip_atomic_load((unsigned*)(p.ws[t.r] + WS_CTR + t.c * 128), __ATOMIC_RELAXED, __HIP_MEMORY_SCOPE_AGENT)) + 1) & 1;
  u32x4 v[2][3]; unsigned po[3];
#pragma unroll
  for (int i = 0; i < 3; ++i) { const int q = 3 * w + i; po[i] = (q >> 2) * XPITCH + (q & 3) * 1024 + lane * 16;
    v[0][i] = *(const u32x4*)(p.x + (size_t)RROWS * XPITCH + po[i]); v[1][i] = *(const u32x4*)(p.x + (size_t)RROWS * XPITCH + po[i] + 4096); }
#pragma unroll
  for (int h = 0; h < 2; ++h) {
    int e = w % 7;
#pragma unroll 1
    for (int n = 0; n < 7; ++n) {
      char* base = (char*)(((u64)(unsigned)__builtin_amdgcn_readlane((unsigned)(wsl >> 32), e) << 32) | (unsigned)__builtin_amdgcn_readlane((unsigned)wsl, e));
      const auto rs = rsrc(base + WS_X + (par * NR + t.r) * XBLK, XBLK);
#pragma unroll
      for (int i = 0; i < 3; ++i) { __builtin_amdgcn_raw_buffer_store_b128(v[h][i], rs, po[i], h * 4096, AUXX); waitv<XFLY>(); }
      e = e == 6 ? 0 : e + 1;
    }
    waitv<0>(); BAR();
    if (wid == 0 && lane < 7) __hip_atomic_fetch_add((unsigned*)((char*)wsl + WS_XFLAG + (t.r * 2 + h) * 128), 1u, __ATOMIC_RELAXED, __HIP_MEMORY_SCOPE_SYSTEM);
  }
}

__device__ __forceinline__ void producer(const Params& p, const Tile& t, LAS char* lds, int pw, int lane) {
  char* const myws = p.ws[t.r];
  const unsigned va0 = (lane >> 3) * XPITCH + (((lane & 7) ^ (lane >> 4)) << 4), va1 = va0 ^ 64;
  const unsigned vb0 = (lane >> 4) * WPITCH + (((lane & 15) ^ (((lane >> 4) & 1) | ((lane >> 5) << 2))) << 4), vb1 = vb0 ^ 128;
  const unsigned ve0 = va0 ^ ((pw & 1) << 6), ve1 = ve0 ^ 64;
  const auto ra1 = rsrc(p.x + (size_t)(t.row1 + 40 * pw) * XPITCH, 40 * XPITCH);
  const auto rb1 = rsrc(p.w + (size_t)(t.peer * 16 + t.pl1) * 256, (KD - 1) * WPITCH + 256);
  const int s2 = t.src2, r2 = t.row2 + 32 * (pw & 1);
  auto ra2 = rsrc(p.x + (size_t)r2 * XPITCH, t.row2 < 0 ? 0 : 32 * XPITCH);
  const auto rb2 = rsrc(p.w + (size_t)(t.r * 16 + t.pl2) * 256, (KD - 1) * WPITCH + 256);
  unsigned so = 0;
  auto issue1 = [&](int s) {
    const int kt = (s & 60) | ((s + t.rot1) & 3);
#pragma unroll
    for (int i = 0; i < MF1; ++i) __builtin_amdgcn_raw_ptr_buffer_load_lds(ra1, lds + so + (pw * MF1 + i) * 1024, 16, (i & 1) ? ve1 : ve0, kt * 128 + i * 8 * XPITCH, 0, 0);
#pragma unroll
    for (int i = 0; i < 4; ++i) __builtin_amdgcn_raw_ptr_buffer_load_lds(rb1, lds + so + AMAX + (pw * 4 + i) * 1024, 16, (i & 2) ? vb1 : vb0, (kt * 64 + pw * 16 + i * 4) * WPITCH, 0, 0);
    so = so + SLOT == RING ? 0 : so + SLOT;
  };
  auto issue2 = [&](int s) {
    const int kt = (s & 60) | ((s + t.rot2) & 3);
#pragma unroll
    for (int i = 0; i < MF2; ++i) __builtin_amdgcn_raw_ptr_buffer_load_lds(ra2, lds + so + (pw * MF2 + i) * 1024, 16, (i & 1) ? va1 : va0, kt * 128 + i * 8 * XPITCH, 0, 0);
#pragma unroll
    for (int i = 0; i < 4; ++i) __builtin_amdgcn_raw_ptr_buffer_load_lds(rb2, lds + so + AMAX2 + (pw * 4 + i) * 1024, 16, (i & 2) ? vb1 : vb0, (kt * 64 + pw * 16 + i * 4) * WPITCH, 0, 0);
    so = so + SLOT2 == RING2 ? 0 : so + SLOT2;
  };
  constexpr int L1 = MF1 + 4, L2 = MF2 + 4;
  unsigned callv = 0, fv = 0;
  const bool poller = (pw == 0) && !t.own;
  auto poll = [&](int q) {
    if (poller) { const char* fp = myws + WS_XFLAG + (t.peer * 2 + q) * 128; asm volatile("global_load_dword %0, %1, off sc0 sc1" : "+v"(fv) : "v"(fp) : "memory"); }
  };
  auto gate = [&](int q) {
    if (poller) {
      asm volatile("" : "+v"(fv), "+v"(callv));
      const unsigned call = __builtin_amdgcn_readfirstlane(callv) + 1, need = NEED(32u * call);
      if (__any((int)(fv - need) < 0)) {
        unsigned* const tmo = (unsigned*)(myws + WS_TMO);
        const u64 t0 = __builtin_amdgcn_s_memrealtime();
        while (__any((int)(__hip_atomic_load((unsigned*)(myws + WS_XFLAG + (t.peer * 2 + q) * 128), __ATOMIC_RELAXED, __HIP_MEMORY_SCOPE_SYSTEM) - need) < 0)) {
          if (__hip_atomic_load(tmo, __ATOMIC_RELAXED, __HIP_MEMORY_SCOPE_AGENT) == call || __builtin_amdgcn_s_memrealtime() - t0 > DEADLINE) {
            __hip_atomic_store(tmo, call, __ATOMIC_RELAXED, __HIP_MEMORY_SCOPE_AGENT);
            waitv<0>();
            break;
          }
          __builtin_amdgcn_s_sleep(2);
        }
      }
      asm volatile("buffer_inv sc0 sc1" ::: "memory");
    }
  };
  if (!t.own) {
    issue1(0); issue1(1); issue1(2); issue1(3);
    { const char* ctr = myws + WS_CTR + t.c * 128; asm volatile("global_load_dword %0, %1, off sc1" : "+v"(callv) : "v"(ctr) : "memory"); }
    waitv<3 * L1 + 1>(); BAR();
    waitv<2 * L1 + 1>(); BAR(); issue1(4);
#pragma unroll 1
    for (int i = 1; i < 60; ++i) {
      waitv<2 * L1>();
      if (i == 58) gate(0);
      BAR();
      if (i == 56) poll(0);
      issue1(i + 4);
    }
    asm volatile("" : "+v"(callv));
    if (s2 != t.r) ra2 = rsrc(myws + WS_X + (((__builtin_amdgcn_readfirstlane(callv) + 1) & 1) * NR + s2) * XBLK + (size_t)(r2 - RROWS) * XPITCH, 32 * XPITCH);
    waitv<2 * L1>(); BAR(); issue2(0);
    waitv<L1 + L2>(); BAR(); issue2(1);
    waitv<2 * L2>(); BAR(); issue2(2);
    waitv<2 * L2>(); BAR(); issue2(3);
    if (NS2 == 5) issue2(4);
  } else {
    waitv<0>();
#pragma unroll
    for (int n = 0; n < NS2; ++n) issue2(n);
    waitv<(NS2 - 1) * L2>(); BAR();
  }
  constexpr int W2 = (NS2 - 2) * L2, IG = 32 - NS2 - (NS2 - 2), IP = IG - (NS2 - 2);
#pragma unroll 1
  for (int i = 0; i < 64 - NS2; ++i) {
    waitv<W2>();
    if (i == IG) gate(1);
    BAR();
    if (i == IP) poll(1);
    issue2(i + NS2);
  }
  static_for<0, NS2 - 1>([&](auto Q) { waitv<(NS2 - 2 - Q) * L2>(); BAR(); });
  BAR();
  asm volatile("" :: "v"(callv), "v"(fv));
}

__device__ __forceinline__ void consumer(const Params& p, const Tile& t, LAS char* lds, int wid, int lane) {
  char* const myws = p.ws[t.r];
  __builtin_amdgcn_s_setprio(2);
  const int wr = wid >> 1, wc = wid & 1, fr = lane & 15, fq = lane >> 4;
  const bool odd = fq & 1;
  const unsigned callv = __hip_atomic_load((unsigned*)(myws + WS_CTR + t.c * 128), __ATOMIC_RELAXED, __HIP_MEMORY_SCOPE_AGENT);

  const unsigned lb = (unsigned)(size_t)lds;
  unsigned a1[2], a2[2], baddr[4];
#pragma unroll
  for (int ks = 0; ks < 2; ++ks) { const unsigned u = lb + fr * 128 + 16 * ((4 * ks + fq) ^ ((fr >> 1) & 7)); a1[ks] = u + 16 * MF1 * wr * 128; a2[ks] = u + 16 * MF2 * wr * 128; }
  { const int q = (lane & 15) >> 2, pp = lane & 3, kr = 8 * fq + q, f = (kr & 1) | ((kr & 2) << 1) | (kr & 8);
#pragma unroll
    for (int gn = 0; gn < 4; ++gn) baddr[gn] = lb + kr * 256 + 16 * ((8 * wc + 4 * (gn >> 1) + pp) ^ f) + 8 * ((gn & 1) ^ (pp & 1)); }
  const auto rs1 = rsrc(p.out[t.peer], t.own ? 0 : OUT_BYTES);
  const unsigned o1 = (t.r * MLOC + t.row1 + 16 * MF1 * wr + (fr & 7)) * OPITCH + (t.pl1 * 128 + 64 * wc + 32 * (fr >> 3) + 8 * fq) * 2;
#define OSTEP(g) (8 * OPITCH)
#define OADDR(g) ((g) * 8 * OPITCH)
  const auto rs2 = rsrc(p.out[t.r], t.row2 < 0 ? 0 : OUT_BYTES);
  const unsigned o2 = (t.src2 * MLOC + t.row2 + fr) * OPITCH + (t.pl2 * 128 + 64 * wc + 8 * fq) * 2;

  Frag F0, F1;
  f32x4 acc1[MF1][4], acc2[MF2][4];
#pragma unroll
  for (int m = 0; m < MF1; ++m)
#pragma unroll
    for (int gn = 0; gn < 4; ++gn) { acc1[m][gn] = (f32x4){0.f, 0.f, 0.f, 0.f}; asm volatile("" : "+v"(acc1[m][gn])); }
  unsigned so = 0, sw, bo = t.own ? AMAX2 : AMAX, bc[4], ac[2];
#pragma unroll
  for (int gn = 0; gn < 4; ++gn) bc[gn] = baddr[gn] + bo;
  ac[0] = t.own ? a2[0] : a1[0]; ac[1] = t.own ? a2[1] : a1[1];
#pragma unroll
  for (int m = 0; m < MF2; ++m)
#pragma unroll
    for (int gn = 0; gn < 4; ++gn) { acc2[m][gn] = (f32x4){0.f, 0.f, 0.f, 0.f}; asm volatile("" : "+v"(acc2[m][gn])); }
  asm volatile("s_nop 1");
  BAR();
  STAMP(1);
  static_for<0, 8 + MF1>([&](auto K) { rd1<K, 0>(F0, ac[0], bc); });
  GeluState gs; gs.ka = GELU_A; asm volatile("" : "+v"(gs.ka));
  auto half_a = [&](auto MF_, auto& acc, const unsigned (&an)[2], auto EV, auto G0_) {
    constexpr int MF = MF_, NM = 4 * MF, NRD = 8 + MF, DBL = dbl_gaps(MF), LAST = NRD - DBL - 1, G0 = G0_;
    static_for<0, NM>([&](auto Z) {
      constexpr int z = Z;
      SCHED(); mm1<z, MF>(acc, F0); SCHED();
      if constexpr (z < DBL) { rd1<2 * z, 1>(F1, ac[1], bc); rd1<2 * z + 1, 1>(F1, ac[1], bc); }
      else if constexpr (z <= LAST) rd1<z + DBL, 1>(F1, ac[1], bc);
      if constexpr (z == LAST - 1) { if constexpr (MF == MF1) so = so + SLOT == RING ? 0 : so + SLOT; else so = so + SLOT2 == RING2 ? 0 : so + SLOT2; sw = so + bo; asm volatile("" : "+s"(so), "+s"(sw)); }
      if constexpr (z == LAST + 1) { bc[0] = baddr[0] + sw; bc[1] = baddr[1] + sw; asm volatile("" : "+v"(bc[0]), "+v"(bc[1])); }
      if constexpr (z == LAST + 2) { bc[2] = baddr[2] + sw; bc[3] = baddr[3] + sw; asm volatile("" : "+v"(bc[2]), "+v"(bc[3])); }
      if constexpr (z == LAST + 3) { ac[0] = an[0] + so; ac[1] = an[1] + so; asm volatile("" : "+v"(ac[0]), "+v"(ac[1])); }
      if constexpr (G0 >= 0 && z > LAST + 3) gelu_piece<G0 + z - LAST - 4>(gs);
    });
    static_assert(LAST + 3 < NM, "half step too short");
    SCHED();
  };
  auto half_b = [&](auto MF_, auto MFN_, auto& acc, auto ZERO, auto G0_) {
    constexpr int MF = MF_, MFN = MFN_, NM = 4 * MF, NRD = 8 + MFN, DBL = dbl_gaps(MF), G0 = G0_;
    SCHED(); WAIT_L0(); BAR();
    static_for<0, NM>([&](auto Z) {
      constexpr int z = Z;
      SCHED(); mm1<z, MF>(acc, F1); SCHED();
      if constexpr (z < DBL) { rd1<2 * z, 0>(F0, ac[0], bc); rd1<2 * z + 1, 0>(F0, ac[0], bc); }
      else if constexpr (z + DBL < NRD) rd1<z + DBL, 0>(F0, ac[0], bc);
      else if constexpr (G0 >= 0) gelu_piece<G0 + z + DBL - NRD>(gs);
    });
    SCHED();
  };
  constexpr std::integral_constant<int, MF1> c1{}; constexpr std::integral_constant<int, MF2> c2{};
  constexpr std::integral_constant<int, -1> nog{}; constexpr std::true_type yes{}; constexpr std::false_type no{};
  int it = 63;
  if (!t.own) {
#pragma unroll 1
    do {
      half_a(c1, acc1, a1, no, nog);
      half_b(c1, c1, acc1, no, nog);
      --it; asm volatile("" : "+s"(it));
    } while (it != 0);
    bo = AMAX2;
    half_a(c1, acc1, a2, no, nog);
    half_b(c1, c2, acc1, no, nog);
    asm volatile("s_nop 15" ::: "memory"); SCHED();
  }
  so = __builtin_amdgcn_readfirstlane(so); bo = AMAX2;
  STAMP(2);
  constexpr int FA = 4 * MF2 - (8 + MF2 - dbl_gaps(MF2)) - 3, FB = 4 * MF2 - (8 + MF2 - dbl_gaps(MF2));
  static_assert(GL * (FA + FB) >= 42, "GELU pieces do not fit");
  constexpr std::integral_constant<int, GL * FB + (GL - 1) * FA> gtail{};
  const unsigned call = __builtin_amdgcn_readfirstlane(callv) + 1;
  static_for<0, G0N>([&](auto G) { constexpr int g = G;
    pick<g>(gs, acc1); __builtin_amdgcn_raw_buffer_store_b128(epi8(gs.cu, gs.cv, odd), rs1, o1 + OADDR(g), 0, AUX_SYS); });
  unsigned oo = o1 + OADDR(G0N);
  it = G0N;
#pragma unroll 1
  do {
    half_a(c2, acc2, a2, no, gtail);
    if (it != G0N) { __builtin_amdgcn_raw_buffer_store_b128((u32x4){gs.ov0, gs.ov1, gs.ov2, gs.ov3}, rs1, oo, 0, AUX_SYS); oo += OSTEP(it); }
#define PICK(g) case g: pick<g>(gs, acc1); break
    switch (it) { PICK(0); PICK(1); PICK(2); PICK(3); PICK(4); PICK(5); PICK(6); PICK(7); PICK(8); PICK(9); default: __builtin_unreachable(); }
    asm volatile("" : "+v"(gs.cu), "+v"(gs.cv));
    half_b(c2, c2, acc2, no, std::integral_constant<int, 0>{});
    static_for<1, GL>([&](auto K) { constexpr int k = K;
      half_a(c2, acc2, a2, no, std::integral_constant<int, FB + (k - 1) * (FA + FB)>{});
      half_b(c2, c2, acc2, no, std::integral_constant<int, k * (FA + FB)>{}); });
    ++it; asm volatile("" : "+s"(it));
  } while (it != 2 * MF1);
  half_a(c2, acc2, a2, no, gtail);
  __builtin_amdgcn_raw_buffer_store_b128((u32x4){gs.ov0, gs.ov1, gs.ov2, gs.ov3}, rs1, oo, 0, AUX_SYS);
  half_b(c2, c2, acc2, no, nog);
  constexpr int NTR = 63 - (2 * MF1 - G0N) * GL;
  static_assert(NTR >= 1, "all groups leave inside the loop");
  it = NTR;
#pragma unroll 1
  do {
    half_a(c2, acc2, a2, no, nog);
    if (it == (NTR > DLAG ? NTR - DLAG : 1)) {
      waitv<0>();
      if (!t.own && lane == 0) __hip_atomic_fetch_add((unsigned*)(p.ws[t.peer] + WS_DONE + (t.r * 4 + wid) * 128), 1u, __ATOMIC_RELAXED, __HIP_MEMORY_SCOPE_SYSTEM);
      STAMP(4);
    }
    half_b(c2, c2, acc2, no, nog);
    --it; asm volatile("" : "+s"(it));
  } while (it != 0);
  asm volatile("s_nop 15" ::: "memory"); SCHED();
  STAMP(3);
  const unsigned tmov = __hip_atomic_load((unsigned*)(myws + WS_TMO), __ATOMIC_RELAXED, __HIP_MEMORY_SCOPE_AGENT);
  SCHED();
#define AUXL AUX_SYS
  static_for<0, 2 * MF2>([&](auto G) { constexpr int g = G;
    __builtin_amdgcn_raw_buffer_store_b128(epi8(acc2[g >> 1][2 * (g & 1)], acc2[g >> 1][2 * (g & 1) + 1], odd), rs2, o2 + (g >> 1) * 16 * OPITCH + (g & 1) * 64, 0, AUXL); });
  STAMP(5);
  if (wid == 0 && lane == 0) __hip_atomic_store((unsigned*)(myws + WS_CTR + t.c * 128), call, __ATOMIC_RELAXED, __HIP_MEMORY_SCOPE_AGENT);
  bool bad = tmov == call;
  if (t.c == 0 && wid == 0) {
    unsigned* const tmo = (unsigned*)(myws + WS_TMO);
    unsigned* const fp = (unsigned*)(myws + WS_DONE + (((t.r + 1 + (lane < 28 ? lane >> 2 : 0)) & 7) * 4 + (lane & 3)) * 128);
    const unsigned need = NEED(32u * call);
    const u64 t0 = __builtin_amdgcn_s_memrealtime();
    for (unsigned i = 1; __any((int)(__hip_atomic_load(fp, __ATOMIC_RELAXED, __HIP_MEMORY_SCOPE_SYSTEM) - need) < 0); ++i) {
      if ((i & 15) == 0 && (__hip_atomic_load(tmo, __ATOMIC_RELAXED, __HIP_MEMORY_SCOPE_AGENT) == call || __builtin_amdgcn_s_memrealtime() - t0 > DEADLINE)) {
        __hip_atomic_store(tmo, call, __ATOMIC_RELAXED, __HIP_MEMORY_SCOPE_AGENT);
        bad = true;
        break;
      }
      __builtin_amdgcn_s_sleep(1);
    }
  }
  if (__builtin_expect(bad, 0)) {
    waitv<0>();
    for (int g = 0; g < 2 * MF2; ++g)
      __builtin_amdgcn_raw_buffer_store_b128((u32x4){0x7fc07fc0u, 0x7fc07fc0u, 0x7fc07fc0u, 0x7fc07fc0u}, rs2, o2 + (g >> 1) * 16 * OPITCH + (g & 1) * 64, 0, AUX_SYS);
  }
  STAMP(6);
}
}

__global__ __launch_bounds__(512, 2) void gemm_a2a(Params p) {
  extern __shared__ __attribute__((aligned(16))) char smem[];
  LAS char* lds = (LAS char*)smem;
  const int lane = threadIdx.x & 63, wid = __builtin_amdgcn_readfirstlane(threadIdx.x >> 6);
  Tile t;
  t.c = blockIdx.x; t.r = p.rank;
  const int xcd = t.c & 7, j = ((t.c >> 3) + 28) & 31, e = j >> 2, hm = j & 1, pb = (j >> 1) & 1, h2 = xcd & 1;
  t.own = j >= 28;
  t.peer = (t.r + 1 + e) & 7;
  t.pl1 = xcd + 8 * pb;
  t.pl2 = 4 * (xcd >> 1) + (j & 3);
  t.row1 = 32 * MF1 * hm;
  if (((wid >> 1) & 1) == 0) { t.src2 = t.peer; t.row2 = t.own ? 128 * h2 : RROWS + 128 * h2; }
  else { t.src2 = h2 ? t.r : t.peer; t.row2 = t.own ? 128 * h2 + 64 : h2 ? (e < 4 ? 256 + 64 * e : -1) : RROWS + 64; }
#define R1M 2
#define R1J 1
#define R2M 1
  t.rot1 = R1M * hm + R1J * (j >> 1); t.rot2 = R2M * (e + j);
  STAMP(0);
  if (t.own) xpush(p, t, wid, lane);
  if (wid >= 4) producer(p, t, lds, wid - 4, lane); else consumer(p, t, lds, wid, lane);
}

extern "C" void kernel_launch(int rank, int nranks, ncclComm_t comm,
                              hipStream_t stream,
                              void* const* d_in, const int* in_sizes, int n_in,
                              void* d_out, int out_size,
                              const int* in_dtypes, int out_dtype,
                              void* const* d_out_all,
                              void* const* d_ws_all, size_t ws_bytes) {
  if (nranks != NR || (unsigned)rank >= (unsigned)NR || n_in != 2 || in_sizes[0] != MLOC * KD || in_sizes[1] != KD * NG ||
      out_size != NR * MLOC * NLOC || ws_bytes < WS_NEED) { fprintf(stderr, "gemm_a2a: unexpected problem shape\n"); return; }
  static bool attr_set[NR] = {};
  if (!attr_set[rank]) { HIP_CHECK(hipFuncSetAttribute((const void*)gemm_a2a, hipFuncAttributeMaxDynamicSharedMemorySize, LDS_BYTES)); attr_set[rank] = true; }
  Params p;
  p.x = (const char*)d_in[0]; p.w = (const char*)d_in[1]; p.rank = rank;
  for (int i = 0; i < NR; ++i) { p.out[i] = (char*)d_out_all[i]; p.ws[i] = (char*)d_ws_all[i]; }
  hipLaunchKernelGGL(gemm_a2a, dim3(256), dim3(512), LDS_BYTES, stream, p);
  HIP_CHECK(hipGetLastError());
}
```

```cpp
#include <hip/hip_runtime.h>
#include <rccl/rccl.h>
#include <cstdio>
#include <type_traits>

#define HIP_CHECK(expr) do { hipError_t _e = (expr); if (_e != hipSuccess) { \
    fprintf(stderr, "HIP error %s at %s:%d\n", hipGetErrorString(_e), __FILE__, __LINE__); \
    return; } } while (0)

namespace {
#define LAS __attribute__((address_space(3)))
typedef short s16x4 __attribute__((ext_vector_type(4)));
typedef short s16x8 __attribute__((ext_vector_type(8)));
typedef __bf16 bf16x8 __attribute__((ext_vector_type(8)));
typedef float f32x4 __attribute__((ext_vector_type(4)));
typedef unsigned u32x4 __attribute__((ext_vector_type(4)));
typedef unsigned long long u64;

constexpr int NR = 8, MLOC = 512, KD = 4096, NG = 16384, NLOC = 2048;
constexpr int MF1 = 5, MF2 = 4;
constexpr int RROWS = 64 * MF1, FROWS = MLOC - RROWS;
constexpr int NS2 = 5;
constexpr int AMAX = 32 * MF1 * 128, SLOT = AMAX + 16384, RING = 4 * SLOT;
constexpr int AMAX2 = 32 * MF2 * 128, SLOT2 = AMAX2 + 16384, RING2 = NS2 * SLOT2, LDS_BYTES = RING > RING2 ? RING : RING2;
static_assert(FROWS == 192 && LDS_BYTES <= 163840, "tiles");
constexpr unsigned XPITCH = KD * 2, WPITCH = NG * 2, OPITCH = NLOC * 2;
constexpr unsigned OUT_BYTES = (unsigned)NR * MLOC * OPITCH;
constexpr unsigned XBLK = FROWS * XPITCH;
constexpr unsigned WS_CTR = 0;
constexpr unsigned WS_TMO = 32768;
constexpr unsigned WS_XFLAG = 36864;
constexpr unsigned WS_DONE = 49152;
constexpr unsigned WS_X = 1u << 20;
constexpr unsigned WS_NEED = WS_X + 2 * NR * XBLK;
constexpr u64 DEADLINE = 40000000ull;
constexpr int AUX_SYS = 17;

constexpr int GL = 6;
constexpr int G0N = 1;
constexpr int DLAG = 3;

struct Params { const char* x; const char* w; char* myws; int rank, pad; char* out[NR]; char* ws[NR]; };
struct Tile { const char* x; const char* w; char* myws; int c, r, peer, own, pl1, pl2, row1, src2, row2, rot1, rot2; };

__device__ __forceinline__ unsigned pk_bf16(float lo, float hi) {
  typedef __bf16 bf16x2 __attribute__((ext_vector_type(2)));
  bf16x2 v = {(__bf16)lo, (__bf16)hi};
  return __builtin_bit_cast(unsigned, v);
}
constexpr float GELU_A = -2.0f * 0.7978845608028654f * 1.4426950408889634f, GELU_B = GELU_A * 0.044715f;
__device__ __forceinline__ float gelu(float y) {
  const float e = __builtin_amdgcn_exp2f(y * __builtin_fmaf(GELU_B, y * y, GELU_A));
  return y * __builtin_amdgcn_rcpf(1.0f + e);
}
__device__ __forceinline__ u32x4 epi8(const f32x4 u, const f32x4 v, bool odd) {
  const unsigned a0 = pk_bf16(gelu(u[0]), gelu(u[1])), a1 = pk_bf16(gelu(u[2]), gelu(u[3]));
  const unsigned b0 = pk_bf16(gelu(v[0]), gelu(v[1])), b1 = pk_bf16(gelu(v[2]), gelu(v[3]));
  u32x4 w; w.x = odd ? b0 : a0; w.y = odd ? b1 : a1; w.z = odd ? a0 : b0; w.w = odd ? a1 : b1;
  return w;
}
template <int N_> __device__ __forceinline__ void waitv() { asm volatile("s_waitcnt vmcnt(%0)" :: "i"(N_) : "memory"); }
#define WAIT_L0() __builtin_amdgcn_s_waitcnt(0xC07F)
#define BAR() __builtin_amdgcn_s_barrier()
#define SCHED() __builtin_amdgcn_sched_barrier(0)
template <int I, int N, class F> __device__ __forceinline__ void static_for(F&& f) { if constexpr (I < N) { f(std::integral_constant<int, I>{}); static_for<I + 1, N>(f); } }
__device__ __forceinline__ LAS char* lp(unsigned a) { return (LAS char*)(size_t)a; }
__device__ __forceinline__ auto rsrc(const void* base, unsigned bytes) { return __builtin_amdgcn_make_buffer_rsrc((void*)base, 0, (int)bytes, 0x00020000); }

struct Frag { s16x8 a[MF1]; s16x4 blo[4], bhi[4]; };
template <int K, int KS> __device__ __forceinline__ void rd1(Frag& f, unsigned aaddr, const unsigned (&baddr)[4]) {
  if constexpr (K == 2) f.a[0] = *(LAS s16x8*)lp(aaddr);
  else if constexpr (K >= 9) f.a[K - 8] = *(LAS s16x8*)(lp(aaddr) + (K - 8) * 2048);
  else {
    constexpr int q = K < 2 ? K : K - 1, gn = q >> 1;
    if constexpr (q & 1) f.bhi[gn] = __builtin_amdgcn_ds_read_tr16_b64_v4i16((LAS s16x4*)(lp(baddr[gn]) + KS * 8192 + 1024));
    else f.blo[gn] = __builtin_amdgcn_ds_read_tr16_b64_v4i16((LAS s16x4*)(lp(baddr[gn]) + KS * 8192));
  }
}
template <int Z, int MF> __device__ __forceinline__ void mm1(f32x4 (&acc)[MF][4], const Frag& f) {
  constexpr int m = Z >> 2, gn = Z & 3;
  const s16x8 b = __builtin_shufflevector(f.blo[gn], f.bhi[gn], 0, 1, 2, 3, 4, 5, 6, 7);
  asm volatile("v_mfma_f32_16x16x32_bf16 %0, %1, %2, %0" : "+v"(acc[m][gn]) : "v"(b), "v"(f.a[m]));
}
struct GeluState { f32x4 cu, cv; float ga, gb, ka; unsigned pk0, pk1, pk2, pk3, ov0, ov1, ov2, ov3; };
template <int F> __device__ __forceinline__ void gelu_piece(GeluState& g) {
  constexpr int pr = F / 10, st = F % 10;
  constexpr u64 ODDM = 0xffff0000ffff0000ull;
  if constexpr (F < 40) {
    const float ya = pr < 2 ? g.cu[2 * (pr & 1)] : g.cv[2 * (pr & 1)], yb = pr < 2 ? g.cu[2 * (pr & 1) + 1] : g.cv[2 * (pr & 1) + 1];
    if constexpr (st == 0) asm volatile("v_mul_f32 %0, %2, %2\n\tv_mul_f32 %1, %3, %3" : "=&v"(g.ga), "=v"(g.gb) : "v"(ya), "v"(yb));
    if constexpr (st == 1) asm volatile("v_fmamk_f32 %0, %0, %3, %2\n\tv_fmamk_f32 %1, %1, %3, %2" : "+v"(g.ga), "+v"(g.gb) : "v"(g.ka), "i"(__builtin_bit_cast(unsigned, GELU_B)));
    if constexpr (st == 2 || st == 8) asm volatile("v_mul_f32 %0, %2, %0\n\tv_mul_f32 %1, %3, %1" : "+v"(g.ga), "+v"(g.gb) : "v"(ya), "v"(yb));
    if constexpr (st == 3) asm volatile("v_exp_f32 %0, %0" : "+v"(g.ga));
    if constexpr (st == 4) asm volatile("v_exp_f32 %0, %0" : "+v"(g.gb));
    if constexpr (st == 5) asm volatile("v_add_f32 %0, 1.0, %0\n\tv_add_f32 %1, 1.0, %1" : "+v"(g.ga), "+v"(g.gb));
    if constexpr (st == 6) asm volatile("v_rcp_f32 %0, %0" : "+v"(g.ga));
    if constexpr (st == 7) asm volatile("v_rcp_f32 %0, %0" : "+v"(g.gb));
    if constexpr (st == 9) { unsigned& d = pr == 0 ? g.pk0 : pr == 1 ? g.pk1 : pr == 2 ? g.pk2 : g.pk3; asm volatile("v_cvt_pk_bf16_f32 %0, %1, %2" : "=v"(d) : "v"(g.ga), "v"(g.gb)); }
  } else if constexpr (F == 40) asm volatile("v_cndmask_b32_e64 %0, %2, %4, %6\n\tv_cndmask_b32_e64 %1, %3, %5, %6" : "=&v"(g.ov0), "=v"(g.ov1) : "v"(g.pk0), "v"(g.pk1), "v"(g.pk2), "v"(g.pk3), "s"(ODDM));
  else if constexpr (F == 41) asm volatile("v_cndmask_b32_e64 %0, %4, %2, %6\n\tv_cndmask_b32_e64 %1, %5, %3, %6" : "=&v"(g.ov2), "=v"(g.ov3) : "v"(g.pk0), "v"(g.pk1), "v"(g.pk2), "v"(g.pk3), "s"(ODDM));
}
template <int G> __device__ __forceinline__ void pick(GeluState& s, const f32x4 (&acc)[MF1][4]) {
  constexpr int m = G >> 1, h = G & 1;
#pragma unroll
  for (int i = 0; i < 4; ++i) {
    const float u0 = acc[m][2 * h][i], u1 = acc[m][2 - 2 * h][i], v0 = acc[m][2 * h + 1][i], v1 = acc[m][3 - 2 * h][i];
    s.cu[i] = __int_as_float(__builtin_amdgcn_update_dpp(__float_as_int(u0), __float_as_int(u1), 0x128, 0xf, h ? 0x3 : 0xc, false));
    s.cv[i] = __int_as_float(__builtin_amdgcn_update_dpp(__float_as_int(v0), __float_as_int(v1), 0x128, 0xf, h ? 0x3 : 0xc, false));
  }
}
constexpr int dbl_gaps(int mf) { return mf == 4 ? 1 : 0; }


constexpr int XFLY = 3;
__device__ __forceinline__ void xpush(const Params& p, const Tile& t, int wid, int lane) {
  const u64 wsl = (u64)p.ws[(t.r + 1 + (lane < 7 ? lane : 0)) & 7];
  const int w = t.c * 8 + wid;
  const unsigned par = (__builtin_amdgcn_readfirstlane(__hip_atomic_load((unsigned*)(t.myws + WS_CTR + t.c * 128), __ATOMIC_RELAXED, __HIP_MEMORY_SCOPE_AGENT)) + 1) & 1;
  u32x4 v[2][3]; unsigned po[3];
#pragma unroll
  for (int i = 0; i < 3; ++i) { const int q = 3 * w + i; po[i] = (q >> 2) * XPITCH + (q & 3) * 1024 + lane * 16;
    v[0][i] = *(const u32x4*)(t.x + (size_t)RROWS * XPITCH + po[i]); v[1][i] = *(const u32x4*)(t.x + (size_t)RROWS * XPITCH + po[i] + 4096); }
#pragma unroll
  for (int h = 0; h < 2; ++h) {
    int e = w % 7;
#pragma unroll 1
    for (int n = 0; n < 7; ++n) {
      char* base = (char*)(((u64)(unsigned)__builtin_amdgcn_readlane((unsigned)(wsl >> 32), e) << 32) | (unsigned)__builtin_amdgcn_readlane((unsigned)wsl, e));
      const auto rs = rsrc(base + WS_X + (par * NR + t.r) * XBLK, XBLK);
#pragma unroll
      for (int i = 0; i < 3; ++i) { __builtin_amdgcn_raw_buffer_store_b128(v[h][i], rs, po[i], h * 4096, AUX_SYS); waitv<XFLY>(); }
      e = e == 6 ? 0 : e + 1;
    }
    waitv<0>(); BAR();
    if (wid == 0 && lane < 7) __hip_atomic_fetch_add((unsigned*)((char*)wsl + WS_XFLAG + (t.r * 2 + h) * 128), 1u, __ATOMIC_RELAXED, __HIP_MEMORY_SCOPE_SYSTEM);
  }
}

__device__ __forceinline__ void producer(const Params& p, const Tile& t, LAS char* lds, int pw, int lane) {
  char* const myws = t.myws;
  const unsigned va0 = (lane >> 3) * XPITCH + (((lane & 7) ^ (lane >> 4)) << 4), va1 = va0 ^ 64;
  const unsigned vb0 = (lane >> 4) * WPITCH + (((lane & 15) ^ (((lane >> 4) & 1) | ((lane >> 5) << 2))) << 4), vb1 = vb0 ^ 128;
  const unsigned ve0 = va0 ^ ((pw & 1) << 6), ve1 = ve0 ^ 64;
  const auto ra1 = rsrc(t.x + (size_t)(t.row1 + 40 * pw) * XPITCH, 40 * XPITCH);
  const auto rb1 = rsrc(t.w + (size_t)(t.peer * 16 + t.pl1) * 256, (KD - 1) * WPITCH + 256);
  const int s2 = t.src2, r2 = t.row2 + 32 * (pw & 1);
  auto ra2 = rsrc(t.x + (size_t)r2 * XPITCH, t.row2 < 0 ? 0 : 32 * XPITCH);
  const auto rb2 = rsrc(t.w + (size_t)(t.r * 16 + t.pl2) * 256, (KD - 1) * WPITCH + 256);
  unsigned so = 0;
  auto issue1 = [&](int s) {
    const int kt = (s & 60) | ((s + t.rot1) & 3);
#pragma unroll
    for (int i = 0; i < MF1; ++i) __builtin_amdgcn_raw_ptr_buffer_load_lds(ra1, lds + so + (pw * MF1 + i) * 1024, 16, (i & 1) ? ve1 : ve0, kt * 128 + i * 8 * XPITCH, 0, 0);
#pragma unroll
    for (int i = 0; i < 4; ++i) __builtin_amdgcn_raw_ptr_buffer_load_lds(rb1, lds + so + AMAX + (pw * 4 + i) * 1024, 16, (i & 2) ? vb1 : vb0, (kt * 64 + pw * 16 + i * 4) * WPITCH, 0, 0);
    so = so + SLOT == RING ? 0 : so + SLOT;
  };
  auto issue2 = [&](int s) {
    const int kt = (s & 60) | ((s + t.rot2) & 3);
#pragma unroll
    for (int i = 0; i < MF2; ++i) __builtin_amdgcn_raw_ptr_buffer_load_lds(ra2, lds + so + (pw * MF2 + i) * 1024, 16, (i & 1) ? va1 : va0, kt * 128 + i * 8 * XPITCH, 0, 0);
#pragma unroll
    for (int i = 0; i < 4; ++i) __builtin_amdgcn_raw_ptr_buffer_load_lds(rb2, lds + so + AMAX2 + (pw * 4 + i) * 1024, 16, (i & 2) ? vb1 : vb0, (kt * 64 + pw * 16 + i * 4) * WPITCH, 0, 0);
    so = so + SLOT2 == RING2 ? 0 : so + SLOT2;
  };
  constexpr int L1 = MF1 + 4, L2 = MF2 + 4;
  unsigned callv = 0, fv = 0;
  const bool poller = (pw == 0) && !t.own;
  auto poll = [&](int q) {
    if (poller) { const char* fp = myws + WS_XFLAG + (t.peer * 2 + q) * 128; asm volatile("global_load_dword %0, %1, off sc0 sc1" : "+v"(fv) : "v"(fp) : "memory"); }
  };
  auto gate = [&](int q) {
    if (poller) {
      asm volatile("" : "+v"(fv), "+v"(callv));
      const unsigned call = __builtin_amdgcn_readfirstlane(callv) + 1, need = 32u * call;
      if (__any((int)(fv - need) < 0)) {
        unsigned* const tmo = (unsigned*)(myws + WS_TMO);
        const u64 t0 = __builtin_amdgcn_s_memrealtime();
        while (__any((int)(__hip_atomic_load((unsigned*)(myws + WS_XFLAG + (t.peer * 2 + q) * 128), __ATOMIC_RELAXED, __HIP_MEMORY_SCOPE_SYSTEM) - need) < 0)) {
          if (__hip_atomic_load(tmo, __ATOMIC_RELAXED, __HIP_MEMORY_SCOPE_AGENT) == call || __builtin_amdgcn_s_memrealtime() - t0 > DEADLINE) {
            __hip_atomic_store(tmo, call, __ATOMIC_RELAXED, __HIP_MEMORY_SCOPE_AGENT);
            waitv<0>();
            break;
          }
          __builtin_amdgcn_s_sleep(2);
        }
      }
      asm volatile("buffer_inv sc0 sc1" ::: "memory");
    }
  };
  if (!t.own) {
    issue1(0); issue1(1); issue1(2); issue1(3);
    { const char* ctr = myws + WS_CTR + t.c * 128; asm volatile("global_load_dword %0, %1, off sc1" : "+v"(callv) : "v"(ctr) : "memory"); }
    waitv<3 * L1 + 1>(); BAR();
    waitv<2 * L1 + 1>(); BAR(); issue1(4);
#pragma unroll 1
    for (int i = 1; i < 60; ++i) {
      waitv<2 * L1>();
      if (i == 58) gate(0);
      BAR();
      if (i == 56) poll(0);
      issue1(i + 4);
    }
    asm volatile("" : "+v"(callv));
    if (s2 != t.r) ra2 = rsrc(myws + WS_X + (((__builtin_amdgcn_readfirstlane(callv) + 1) & 1) * NR + s2) * XBLK + (size_t)(r2 - RROWS) * XPITCH, 32 * XPITCH);
    waitv<2 * L1>(); BAR(); issue2(0);
    waitv<L1 + L2>(); BAR(); issue2(1);
    waitv<2 * L2>(); BAR(); issue2(2);
    waitv<2 * L2>(); BAR(); issue2(3);
    if (NS2 == 5) issue2(4);
  } else {
    waitv<0>();
#pragma unroll
    for (int n = 0; n < NS2; ++n) issue2(n);
    waitv<(NS2 - 1) * L2>(); BAR();
  }
  constexpr int W2 = (NS2 - 2) * L2, IG = 32 - NS2 - (NS2 - 2), IP = IG - (NS2 - 2);
#pragma unroll 1
  for (int i = 0; i < 64 - NS2; ++i) {
    waitv<W2>();
    if (i == IG) gate(1);
    BAR();
    if (i == IP) poll(1);
    issue2(i + NS2);
  }
  static_for<0, NS2 - 1>([&](auto Q) { waitv<(NS2 - 2 - Q) * L2>(); BAR(); });
  BAR();
  asm volatile("" :: "v"(callv), "v"(fv));
}

__device__ __forceinline__ void consumer(const Params& p, const Tile& t, LAS char* lds, int wid, int lane) {
  char* const myws = t.myws;
  __builtin_amdgcn_s_setprio(2);
  const int wr = wid >> 1, wc = wid & 1, fr = lane & 15, fq = lane >> 4;
  const bool odd = fq & 1;
  const unsigned callv = __hip_atomic_load((unsigned*)(myws + WS_CTR + t.c * 128), __ATOMIC_RELAXED, __HIP_MEMORY_SCOPE_AGENT);

  const unsigned lb = (unsigned)(size_t)lds;
  unsigned a1[2], a2[2], baddr[4];
#pragma unroll
  for (int ks = 0; ks < 2; ++ks) { const unsigned u = lb + fr * 128 + 16 * ((4 * ks + fq) ^ ((fr >> 1) & 7)); a1[ks] = u + 16 * MF1 * wr * 128; a2[ks] = u + 16 * MF2 * wr * 128; }
  { const int q = (lane & 15) >> 2, pp = lane & 3, kr = 8 * fq + q, f = (kr & 1) | ((kr & 2) << 1) | (kr & 8);
#pragma unroll
    for (int gn = 0; gn < 4; ++gn) baddr[gn] = lb + kr * 256 + 16 * ((8 * wc + 4 * (gn >> 1) + pp) ^ f) + 8 * ((gn & 1) ^ (pp & 1)); }
  const auto rs1 = rsrc(p.out[t.peer], t.own ? 0 : OUT_BYTES);
  const unsigned o1 = (t.r * MLOC + t.row1 + 16 * MF1 * wr + (fr & 7)) * OPITCH + (t.pl1 * 128 + 64 * wc + 32 * (fr >> 3) + 8 * fq) * 2;
  const auto rs2 = rsrc(p.out[t.r], t.row2 < 0 ? 0 : OUT_BYTES);
  const unsigned o2 = (t.src2 * MLOC + t.row2 + fr) * OPITCH + (t.pl2 * 128 + 64 * wc + 8 * fq) * 2;

  Frag F0, F1;
  f32x4 acc1[MF1][4], acc2[MF2][4];
#pragma unroll
  for (int m = 0; m < MF1; ++m)
#pragma unroll
    for (int gn = 0; gn < 4; ++gn) { acc1[m][gn] = (f32x4){0.f, 0.f, 0.f, 0.f}; asm volatile("" : "+v"(acc1[m][gn])); }
  unsigned so = 0, sw, bo = t.own ? AMAX2 : AMAX, bc[4], ac[2];
#pragma unroll
  for (int gn = 0; gn < 4; ++gn) bc[gn] = baddr[gn] + bo;
  ac[0] = t.own ? a2[0] : a1[0]; ac[1] = t.own ? a2[1] : a1[1];
#pragma unroll
  for (int m = 0; m < MF2; ++m)
#pragma unroll
    for (int gn = 0; gn < 4; ++gn) { acc2[m][gn] = (f32x4){0.f, 0.f, 0.f, 0.f}; asm volatile("" : "+v"(acc2[m][gn])); }
  asm volatile("s_nop 1");
  BAR();
  static_for<0, 8 + MF1>([&](auto K) { rd1<K, 0>(F0, ac[0], bc); });
  GeluState gs; gs.ka = GELU_A; asm volatile("" : "+v"(gs.ka));
  auto half_a = [&](auto MF_, auto& acc, const unsigned (&an)[2], auto G0_) {
    constexpr int MF = MF_, NM = 4 * MF, NRD = 8 + MF, DBL = dbl_gaps(MF), LAST = NRD - DBL - 1, G0 = G0_;
    static_for<0, NM>([&](auto Z) {
      constexpr int z = Z;
      SCHED(); mm1<z, MF>(acc, F0); SCHED();
      if constexpr (z < DBL) { rd1<2 * z, 1>(F1, ac[1], bc); rd1<2 * z + 1, 1>(F1, ac[1], bc); }
      else if constexpr (z <= LAST) rd1<z + DBL, 1>(F1, ac[1], bc);
      if constexpr (z == LAST - 1) { if constexpr (MF == MF1) so = so + SLOT == RING ? 0 : so + SLOT; else so = so + SLOT2 == RING2 ? 0 : so + SLOT2; sw = so + bo; asm volatile("" : "+s"(so), "+s"(sw)); }
      if constexpr (z == LAST + 1) { bc[0] = baddr[0] + sw; bc[1] = baddr[1] + sw; asm volatile("" : "+v"(bc[0]), "+v"(bc[1])); }
      if constexpr (z == LAST + 2) { bc[2] = baddr[2] + sw; bc[3] = baddr[3] + sw; asm volatile("" : "+v"(bc[2]), "+v"(bc[3])); }
      if constexpr (z == LAST + 3) { ac[0] = an[0] + so; ac[1] = an[1] + so; asm volatile("" : "+v"(ac[0]), "+v"(ac[1])); }
      if constexpr (G0 >= 0 && z > LAST + 3) gelu_piece<G0 + z - LAST - 4>(gs);
    });
    static_assert(LAST + 3 < NM, "half step too short");
    SCHED();
  };
  auto half_b = [&](auto MF_, auto MFN_, auto& acc, auto G0_) {
    constexpr int MF = MF_, MFN = MFN_, NM = 4 * MF, NRD = 8 + MFN, DBL = dbl_gaps(MF), G0 = G0_;
    SCHED(); WAIT_L0(); BAR();
    static_for<0, NM>([&](auto Z) {
      constexpr int z = Z;
      SCHED(); mm1<z, MF>(acc, F1); SCHED();
      if constexpr (z < DBL) { rd1<2 * z, 0>(F0, ac[0], bc); rd1<2 * z + 1, 0>(F0, ac[0], bc); }
      else if constexpr (z + DBL < NRD) rd1<z + DBL, 0>(F0, ac[0], bc);
      else if constexpr (G0 >= 0) gelu_piece<G0 + z + DBL - NRD>(gs);
    });
    SCHED();
  };
  constexpr std::integral_constant<int, MF1> c1{}; constexpr std::integral_constant<int, MF2> c2{};
  constexpr std::integral_constant<int, -1> nog{};
  int it = 63;
  if (!t.own) {
#pragma unroll 1
    do {
      half_a(c1, acc1, a1, nog);
      half_b(c1, c1, acc1, nog);
      --it; asm volatile("" : "+s"(it));
    } while (it != 0);
    bo = AMAX2;
    half_a(c1, acc1, a2, nog);
    half_b(c1, c2, acc1, nog);
    asm volatile("s_nop 15" ::: "memory"); SCHED();
  }
  so = __builtin_amdgcn_readfirstlane(so); bo = AMAX2;
  constexpr int FA = 4 * MF2 - (8 + MF2 - dbl_gaps(MF2)) - 3, FB = 4 * MF2 - (8 + MF2 - dbl_gaps(MF2));
  static_assert(GL * (FA + FB) >= 42, "GELU pieces do not fit");
  constexpr std::integral_constant<int, GL * FB + (GL - 1) * FA> gtail{};
  const unsigned call = __builtin_amdgcn_readfirstlane(callv) + 1;
  static_for<0, G0N>([&](auto G) { constexpr int g = G;
    pick<g>(gs, acc1); __builtin_amdgcn_raw_buffer_store_b128(epi8(gs.cu, gs.cv, odd), rs1, o1 + g * 8 * OPITCH, 0, AUX_SYS); });
  unsigned oo = o1 + G0N * 8 * OPITCH;
  it = G0N;
#pragma unroll 1
  do {
    half_a(c2, acc2, a2, gtail);
    if (it != G0N) { __builtin_amdgcn_raw_buffer_store_b128((u32x4){gs.ov0, gs.ov1, gs.ov2, gs.ov3}, rs1, oo, 0, AUX_SYS); oo += 8 * OPITCH; }
#define PICK(g) case g: pick<g>(gs, acc1); break
    switch (it) { PICK(0); PICK(1); PICK(2); PICK(3); PICK(4); PICK(5); PICK(6); PICK(7); PICK(8); PICK(9); default: __builtin_unreachable(); }
    asm volatile("" : "+v"(gs.cu), "+v"(gs.cv));
    half_b(c2, c2, acc2, std::integral_constant<int, 0>{});
    static_for<1, GL>([&](auto K) { constexpr int k = K;
      half_a(c2, acc2, a2, std::integral_constant<int, FB + (k - 1) * (FA + FB)>{});
      half_b(c2, c2, acc2, std::integral_constant<int, k * (FA + FB)>{}); });
    ++it; asm volatile("" : "+s"(it));
  } while (it != 2 * MF1);
  half_a(c2, acc2, a2, gtail);
  __builtin_amdgcn_raw_buffer_store_b128((u32x4){gs.ov0, gs.ov1, gs.ov2, gs.ov3}, rs1, oo, 0, AUX_SYS);
  half_b(c2, c2, acc2, nog);
  constexpr int NTR = 63 - (2 * MF1 - G0N) * GL;
  static_assert(NTR >= 1, "all groups leave inside the loop");
  it = NTR;
#pragma unroll 1
  do {
    half_a(c2, acc2, a2, nog);
    if (it == (NTR > DLAG ? NTR - DLAG : 1)) {
      waitv<0>();
      if (!t.own && lane == 0) __hip_atomic_fetch_add((unsigned*)(p.ws[t.peer] + WS_DONE + (t.r * 4 + wid) * 128), 1u, __ATOMIC_RELAXED, __HIP_MEMORY_SCOPE_SYSTEM);
    }
    half_b(c2, c2, acc2, nog);
    --it; asm volatile("" : "+s"(it));
  } while (it != 0);
  asm volatile("s_nop 15" ::: "memory"); SCHED();
  const unsigned tmov = __hip_atomic_load((unsigned*)(myws + WS_TMO), __ATOMIC_RELAXED, __HIP_MEMORY_SCOPE_AGENT);
  SCHED();
  static_for<0, 2 * MF2>([&](auto G) { constexpr int g = G;
    __builtin_amdgcn_raw_buffer_store_b128(epi8(acc2[g >> 1][2 * (g & 1)], acc2[g >> 1][2 * (g & 1) + 1], odd), rs2, o2 + (g >> 1) * 16 * OPITCH + (g & 1) * 64, 0, AUX_SYS); });
  if (wid == 0 && lane == 0) __hip_atomic_store((unsigned*)(myws + WS_CTR + t.c * 128), call, __ATOMIC_RELAXED, __HIP_MEMORY_SCOPE_AGENT);
  bool bad = tmov == call;
  if (t.c == 0 && wid == 0) {
    unsigned* const tmo = (unsigned*)(myws + WS_TMO);
    unsigned* const fp = (unsigned*)(myws + WS_DONE + (((t.r + 1 + (lane < 28 ? lane >> 2 : 0)) & 7) * 4 + (lane & 3)) * 128);
    const unsigned need = 32u * call;
    const u64 t0 = __builtin_amdgcn_s_memrealtime();
    for (unsigned i = 1; __any((int)(__hip_atomic_load(fp, __ATOMIC_RELAXED, __HIP_MEMORY_SCOPE_SYSTEM) - need) < 0); ++i) {
      if ((i & 15) == 0 && (__hip_atomic_load(tmo, __ATOMIC_RELAXED, __HIP_MEMORY_SCOPE_AGENT) == call || __builtin_amdgcn_s_memrealtime() - t0 > DEADLINE)) {
        __hip_atomic_store(tmo, call, __ATOMIC_RELAXED, __HIP_MEMORY_SCOPE_AGENT);
        bad = true;
        break;
      }
      __builtin_amdgcn_s_sleep(1);
    }
  }
  if (__builtin_expect(bad, 0)) {
    waitv<0>();
    for (int g = 0; g < 2 * MF2; ++g)
      __builtin_amdgcn_raw_buffer_store_b128((u32x4){0x7fc07fc0u, 0x7fc07fc0u, 0x7fc07fc0u, 0x7fc07fc0u}, rs2, o2 + (g >> 1) * 16 * OPITCH + (g & 1) * 64, 0, AUX_SYS);
  }
}
}

__global__ __launch_bounds__(512, 2) void gemm_a2a(Params p) {
  extern __shared__ __attribute__((aligned(16))) char smem[];
  LAS char* lds = (LAS char*)smem;
  const int lane = threadIdx.x & 63, wid = __builtin_amdgcn_readfirstlane(threadIdx.x >> 6);
  Tile t;
  t.x = p.x; t.w = p.w; t.myws = p.myws; t.r = p.rank;
  asm volatile("" : "+s"(t.x), "+s"(t.w), "+s"(t.myws), "+s"(t.r));
  t.c = blockIdx.x;
  const int xcd = t.c & 7, j = ((t.c >> 3) + 28) & 31, e = j >> 2, hm = j & 1, pb = (j >> 1) & 1, h2 = xcd & 1;
  t.own = j >= 28;
  t.peer = (t.r + 1 + e) & 7;
  t.pl1 = xcd + 8 * pb;
  t.pl2 = 4 * (xcd >> 1) + (j & 3);
  t.row1 = 32 * MF1 * hm;
  if (((wid >> 1) & 1) == 0) { t.src2 = t.peer; t.row2 = t.own ? 128 * h2 : RROWS + 128 * h2; }
  else { t.src2 = h2 ? t.r : t.peer; t.row2 = t.own ? 128 * h2 + 64 : h2 ? (e < 4 ? 256 + 64 * e : -1) : RROWS + 64; }
    t.rot1 = 2 * hm + (j >> 1); t.rot2 = e + j;
  if (__builtin_expect(t.own, 0)) xpush(p, t, wid, lane);
  if (__builtin_expect(wid >= 4, 1)) producer(p, t, lds, wid - 4, lane); else consumer(p, t, lds, wid, lane);
}

extern "C" void kernel_launch(int rank, int nranks, ncclComm_t comm,
                              hipStream_t stream,
                              void* const* d_in, const int* in_sizes, int n_in,
                              void* d_out, int out_size,
                              const int* in_dtypes, int out_dtype,
                              void* const* d_out_all,
                              void* const* d_ws_all, size_t ws_bytes) {
  if (nranks != NR || (unsigned)rank >= (unsigned)NR || n_in != 2 || in_sizes[0] != MLOC * KD || in_sizes[1] != KD * NG ||
      out_size != NR * MLOC * NLOC || ws_bytes < WS_NEED) { fprintf(stderr, "gemm_a2a: unexpected problem shape\n"); return; }
  static bool attr_set[NR] = {};
  if (!attr_set[rank]) { HIP_CHECK(hipFuncSetAttribute((const void*)gemm_a2a, hipFuncAttributeMaxDynamicSharedMemorySize, LDS_BYTES)); attr_set[rank] = true; }
  Params p;
  p.x = (const char*)d_in[0]; p.w = (const char*)d_in[1]; p.rank = rank; p.pad = 0; p.myws = (char*)d_ws_all[rank];
  for (int i = 0; i < NR; ++i) { p.out[i] = (char*)d_out_all[i]; p.ws[i] = (char*)d_ws_all[i]; }
  hipLaunchKernelGGL(gemm_a2a, dim3(256), dim3(512), LDS_BYTES, stream, p);
  HIP_CHECK(hipGetLastError());
}
```

```cpp
#define DIAG 8
#define DIAGAVG 0
#include <hip/hip_runtime.h>
#include <rccl/rccl.h>
#include <cstdio>
#include <type_traits>

#define HIP_CHECK(expr) do { hipError_t _e = (expr); if (_e != hipSuccess) { \
    fprintf(stderr, "HIP error %s at %s:%d\n", hipGetErrorString(_e), __FILE__, __LINE__); \
    return; } } while (0)

namespace {
#define LAS __attribute__((address_space(3)))
#define GAS __attribute__((address_space(1)))
typedef short s16x4 __attribute__((ext_vector_type(4)));
typedef short s16x8 __attribute__((ext_vector_type(8)));
typedef __bf16 bf16x8 __attribute__((ext_vector_type(8)));
typedef float f32x4 __attribute__((ext_vector_type(4)));
typedef unsigned u32x4 __attribute__((ext_vector_type(4)));
typedef unsigned long long u64;

constexpr int NR = 8, MLOC = 512, KD = 4096, NG = 16384, NLOC = 2048;
constexpr int MF1 = 5, MF2 = 4;
constexpr int RROWS = 64 * MF1, FROWS = MLOC - RROWS;
#define NS2 5
constexpr int AMAX = 32 * MF1 * 128, SLOT = AMAX + 16384, RING = 4 * SLOT;
constexpr int AMAX2 = 32 * MF2 * 128, SLOT2 = AMAX2 + 16384, RING2 = NS2 * SLOT2, LDS_BYTES = RING > RING2 ? RING : RING2;
static_assert(FROWS == 192 && LDS_BYTES <= 163840, "tiles");
constexpr unsigned XPITCH = KD * 2, WPITCH = NG * 2, OPITCH = NLOC * 2;
constexpr unsigned OUT_BYTES = (unsigned)NR * MLOC * OPITCH;
constexpr unsigned XBLK = FROWS * XPITCH;
constexpr unsigned WS_CTR = 0;
constexpr unsigned WS_TMO = 32768;
constexpr unsigned WS_XFLAG = 36864;
constexpr unsigned WS_DONE = 49152;
constexpr unsigned WS_X = 1u << 20;
constexpr unsigned WS_NEED = WS_X + 2 * NR * XBLK;
constexpr u64 DEADLINE = 40000000ull;
#define AUXO 17
constexpr int AUX_SYS = AUXO;

#define GL 6
#define G0N 1
#define DLAG 3

struct Params { const char* x; const char* w; char* myws; int rank, pad; char* out[NR]; char* ws[NR]; };
struct Tile { const char* x; const char* w; char* myws; int c, r, peer, own, pl1, pl2, row1, src2, row2, rot1, rot2; unsigned t0; };

__device__ __forceinline__ unsigned pk_bf16(float lo, float hi) {
  typedef __bf16 bf16x2 __attribute__((ext_vector_type(2)));
  bf16x2 v = {(__bf16)lo, (__bf16)hi};
  return __builtin_bit_cast(unsigned, v);
}
constexpr float GELU_A = -2.0f * 0.7978845608028654f * 1.4426950408889634f, GELU_B = GELU_A * 0.044715f;
typedef float f32x2 __attribute__((ext_vector_type(2)));
__device__ __forceinline__ unsigned gelu2(const f32x2 y) {
  const f32x2 z = y * __builtin_elementwise_fma(y * y, (f32x2){GELU_B, GELU_B}, (f32x2){GELU_A, GELU_A});
  const f32x2 d = (f32x2){__builtin_amdgcn_exp2f(z.x), __builtin_amdgcn_exp2f(z.y)} + 1.0f;
  const f32x2 o = y * (f32x2){__builtin_amdgcn_rcpf(d.x), __builtin_amdgcn_rcpf(d.y)};
  return pk_bf16(o.x, o.y);
}
__device__ __forceinline__ u32x4 epi8(const f32x4 u, const f32x4 v, bool odd) {
  const unsigned a0 = gelu2(u.xy), a1 = gelu2(u.zw), b0 = gelu2(v.xy), b1 = gelu2(v.zw);
  u32x4 w; w.x = odd ? b0 : a0; w.y = odd ? b1 : a1; w.z = odd ? a0 : b0; w.w = odd ? a1 : b1;
  return w;
}
template <int N_> __device__ __forceinline__ void waitv() { asm volatile("s_waitcnt vmcnt(%0)" :: "i"(N_) : "memory"); }
#define WAIT_L0() __builtin_amdgcn_s_waitcnt(0xC07F)
#define BAR() __builtin_amdgcn_s_barrier()
#define SCHED() __builtin_amdgcn_sched_barrier(0)
template <int I, int N, class F> __device__ __forceinline__ void static_for(F&& f) { if constexpr (I < N) { f(std::integral_constant<int, I>{}); static_for<I + 1, N>(f); } }
__device__ __forceinline__ LAS char* lp(unsigned a) { return (LAS char*)(size_t)a; }
__device__ __forceinline__ auto rsrc(const void* base, unsigned bytes) { return __builtin_amdgcn_make_buffer_rsrc((void*)base, 0, (int)bytes, 0x00020000); }

struct Frag { s16x8 a[MF1]; s16x4 blo[4], bhi[4]; };
template <int K, int KS> __device__ __forceinline__ void rd1(Frag& f, unsigned aaddr, const unsigned (&baddr)[4]) {
  if constexpr (K == 2) f.a[0] = *(LAS s16x8*)lp(aaddr);
  else if constexpr (K >= 9) f.a[K - 8] = *(LAS s16x8*)(lp(aaddr) + (K - 8) * 2048);
  else {
    constexpr int q = K < 2 ? K : K - 1, gn = q >> 1;
    if constexpr (q & 1) f.bhi[gn] = __builtin_amdgcn_ds_read_tr16_b64_v4i16((LAS s16x4*)(lp(baddr[gn]) + KS * 8192 + 1024));
    else f.blo[gn] = __builtin_amdgcn_ds_read_tr16_b64_v4i16((LAS s16x4*)(lp(baddr[gn]) + KS * 8192));
  }
}
template <int Z, int MF> __device__ __forceinline__ void mm1(f32x4 (&acc)[MF][4], const Frag& f) {
  constexpr int m = Z >> 2, gn = Z & 3;
  const s16x8 b = __builtin_shufflevector(f.blo[gn], f.bhi[gn], 0, 1, 2, 3, 4, 5, 6, 7);
  asm volatile("v_mfma_f32_16x16x32_bf16 %0, %1, %2, %0" : "+v"(acc[m][gn]) : "v"(b), "v"(f.a[m]));
}
struct GeluState { f32x4 cu, cv; float ga, gb, ka; unsigned pk0, pk1, pk2, pk3, ov0, ov1, ov2, ov3; };
template <int F> __device__ __forceinline__ void gelu_piece(GeluState& g) {
  constexpr int pr = F / 10, st = F % 10;
  constexpr u64 ODDM = 0xffff0000ffff0000ull;
  if constexpr (F < 40) {
    const float ya = pr < 2 ? g.cu[2 * (pr & 1)] : g.cv[2 * (pr & 1)], yb = pr < 2 ? g.cu[2 * (pr & 1) + 1] : g.cv[2 * (pr & 1) + 1];
    if constexpr (st == 0) asm volatile("v_mul_f32 %0, %2, %2\n\tv_mul_f32 %1, %3, %3" : "=&v"(g.ga), "=v"(g.gb) : "v"(ya), "v"(yb));
    if constexpr (st == 1) asm volatile("v_fmamk_f32 %0, %0, %3, %2\n\tv_fmamk_f32 %1, %1, %3, %2" : "+v"(g.ga), "+v"(g.gb) : "v"(g.ka), "i"(__builtin_bit_cast(unsigned, GELU_B)));
    if constexpr (st == 2 || st == 8) asm volatile("v_mul_f32 %0, %2, %0\n\tv_mul_f32 %1, %3, %1" : "+v"(g.ga), "+v"(g.gb) : "v"(ya), "v"(yb));
    if constexpr (st == 3) asm volatile("v_exp_f32 %0, %0" : "+v"(g.ga));
    if constexpr (st == 4) asm volatile("v_exp_f32 %0, %0" : "+v"(g.gb));
    if constexpr (st == 5) asm volatile("v_add_f32 %0, 1.0, %0\n\tv_add_f32 %1, 1.0, %1" : "+v"(g.ga), "+v"(g.gb));
    if constexpr (st == 6) asm volatile("v_rcp_f32 %0, %0" : "+v"(g.ga));
    if constexpr (st == 7) asm volatile("v_rcp_f32 %0, %0" : "+v"(g.gb));
    if constexpr (st == 9) { unsigned& d = pr == 0 ? g.pk0 : pr == 1 ? g.pk1 : pr == 2 ? g.pk2 : g.pk3; asm volatile("v_cvt_pk_bf16_f32 %0, %1, %2" : "=v"(d) : "v"(g.ga), "v"(g.gb)); }
  } else if constexpr (F == 40) asm volatile("v_cndmask_b32_e64 %0, %2, %4, %6\n\tv_cndmask_b32_e64 %1, %3, %5, %6" : "=&v"(g.ov0), "=v"(g.ov1) : "v"(g.pk0), "v"(g.pk1), "v"(g.pk2), "v"(g.pk3), "s"(ODDM));
  else if constexpr (F == 41) asm volatile("v_cndmask_b32_e64 %0, %4, %2, %6\n\tv_cndmask_b32_e64 %1, %5, %3, %6" : "=&v"(g.ov2), "=v"(g.ov3) : "v"(g.pk0), "v"(g.pk1), "v"(g.pk2), "v"(g.pk3), "s"(ODDM));
}
template <int G> __device__ __forceinline__ void pick(GeluState& s, const f32x4 (&acc)[MF1][4]) {
  constexpr int m = G >> 1, h = G & 1;
#pragma unroll
  for (int i = 0; i < 4; ++i) {
    const float u0 = acc[m][2 * h][i], u1 = acc[m][2 - 2 * h][i], v0 = acc[m][2 * h + 1][i], v1 = acc[m][3 - 2 * h][i];
    s.cu[i] = __int_as_float(__builtin_amdgcn_update_dpp(__float_as_int(u0), __float_as_int(u1), 0x128, 0xf, h ? 0x3 : 0xc, false));
    s.cv[i] = __int_as_float(__builtin_amdgcn_update_dpp(__float_as_int(v0), __float_as_int(v1), 0x128, 0xf, h ? 0x3 : 0xc, false));
  }
}
constexpr int dbl_gaps(int mf) { return mf == 2 ? 5 : mf == 3 ? 4 : mf == 4 ? 1 : 0; }

#define STAMP(i)
#define NEED(n) (n)

#define AUXX 17
#define XFLY 3
__device__ __forceinline__ void xpush(const Params& p, const Tile& t, int wid, int lane) {
  const u64 wsl = (u64)p.ws[(t.r + 1 + (lane < 7 ? lane : 0)) & 7];
  const int w = t.c * 8 + wid;
  const unsigned par = (__builtin_amdgcn_readfirstlane(__hip_atomic_load((unsigned*)(t.myws + WS_CTR + t.c * 128), __ATOMIC_RELAXED, __HIP_MEMORY_SCOPE_AGENT)) + 1) & 1;
  u32x4 v[2][3]; unsigned po[3];
#pragma unroll
  for (int i = 0; i < 3; ++i) { const int q = 3 * w + i; po[i] = (q >> 2) * XPITCH + (q & 3) * 1024 + lane * 16;
    v[0][i] = *(const u32x4*)(t.x + (size_t)RROWS * XPITCH + po[i]); v[1][i] = *(const u32x4*)(t.x + (size_t)RROWS * XPITCH + po[i] + 4096); }
#pragma unroll
  for (int h = 0; h < 2; ++h) {
    int e = w % 7;
#pragma unroll 1
    for (int n = 0; n < 7; ++n) {
      char* base = (char*)(((u64)(unsigned)__builtin_amdgcn_readlane((unsigned)(wsl >> 32), e) << 32) | (unsigned)__builtin_amdgcn_readlane((unsigned)wsl, e));
      const auto rs = rsrc(base + WS_X + (par * NR + t.r) * XBLK, XBLK);
#pragma unroll
      for (int i = 0; i < 3; ++i) { __builtin_amdgcn_raw_buffer_store_b128(v[h][i], rs, po[i], h * 4096, AUXX); waitv<XFLY>(); }
      e = e == 6 ? 0 : e + 1;
    }
    waitv<0>(); BAR();
    if (wid == 0 && lane < 7) __hip_atomic_fetch_add((unsigned*)(GAS unsigned*)(wsl + WS_XFLAG + (t.r * 2 + h) * 128), 1u, __ATOMIC_RELAXED, __HIP_MEMORY_SCOPE_SYSTEM);
  }
}

__device__ __forceinline__ void producer(const Params& p, const Tile& t, LAS char* lds, int pw, int lane) {
  char* const myws = t.myws;
  const unsigned va0 = (lane >> 3) * XPITCH + (((lane & 7) ^ (lane >> 4)) << 4), va1 = va0 ^ 64;
  const unsigned vb0 = (lane >> 4) * WPITCH + (((lane & 15) ^ (((lane >> 4) & 1) | ((lane >> 5) << 2))) << 4), vb1 = vb0 ^ 128;
  const unsigned ve0 = va0 ^ ((pw & 1) << 6), ve1 = ve0 ^ 64;
  const auto ra1 = rsrc(t.x + (size_t)(t.row1 + 40 * pw) * XPITCH, 40 * XPITCH);
  const auto rb1 = rsrc(t.w + (size_t)(t.peer * 16 + t.pl1) * 256, (KD - 1) * WPITCH + 256);
  const int s2 = t.src2, r2 = t.row2 + 32 * (pw & 1);
  auto ra2 = rsrc(t.x + (size_t)r2 * XPITCH, t.row2 < 0 ? 0 : 32 * XPITCH);
  const auto rb2 = rsrc(t.w + (size_t)(t.r * 16 + t.pl2) * 256, (KD - 1) * WPITCH + 256);
  unsigned so = 0;
  auto issue1 = [&](int s) {
    const int kt = (s & 60) | ((s + t.rot1) & 3);
#pragma unroll
    for (int i = 0; i < MF1; ++i) __builtin_amdgcn_raw_ptr_buffer_load_lds(ra1, lds + so + (pw * MF1 + i) * 1024, 16, (i & 1) ? ve1 : ve0, kt * 128 + i * 8 * XPITCH, 0, 0);
#pragma unroll
    for (int i = 0; i < 4; ++i) __builtin_amdgcn_raw_ptr_buffer_load_lds(rb1, lds + so + AMAX + (pw * 4 + i) * 1024, 16, (i & 2) ? vb1 : vb0, (kt * 64 + pw * 16 + i * 4) * WPITCH, 0, 0);
    so = so + SLOT == RING ? 0 : so + SLOT;
  };
  auto issue2 = [&](int s) {
    const int kt = (s & 60) | ((s + t.rot2) & 3);
#pragma unroll
    for (int i = 0; i < MF2; ++i) __builtin_amdgcn_raw_ptr_buffer_load_lds(ra2, lds + so + (pw * MF2 + i) * 1024, 16, (i & 1) ? va1 : va0, kt * 128 + i * 8 * XPITCH, 0, 0);
#pragma unroll
    for (int i = 0; i < 4; ++i) __builtin_amdgcn_raw_ptr_buffer_load_lds(rb2, lds + so + AMAX2 + (pw * 4 + i) * 1024, 16, (i & 2) ? vb1 : vb0, (kt * 64 + pw * 16 + i * 4) * WPITCH, 0, 0);
    so = so + SLOT2 == RING2 ? 0 : so + SLOT2;
  };
  constexpr int L1 = MF1 + 4, L2 = MF2 + 4;
  unsigned callv = 0, fv = 0;
  const bool poller = (pw == 0) && !t.own;
  auto poll = [&](int q) {
    if (poller) { const char* fp = myws + WS_XFLAG + (t.peer * 2 + q) * 128; asm volatile("global_load_dword %0, %1, off sc0 sc1" : "+v"(fv) : "v"(fp) : "memory"); }
  };
  auto gate = [&](int q) {
    if (poller) {
      asm volatile("" : "+v"(fv), "+v"(callv));
      const unsigned call = __builtin_amdgcn_readfirstlane(callv) + 1, need = NEED(32u * call);
      if (__any((int)(fv - need) < 0)) {
        unsigned* const tmo = (unsigned*)(myws + WS_TMO);
        const u64 t0 = __builtin_amdgcn_s_memrealtime();
        while (__any((int)(__hip_atomic_load((unsigned*)(myws + WS_XFLAG + (t.peer * 2 + q) * 128), __ATOMIC_RELAXED, __HIP_MEMORY_SCOPE_SYSTEM) - need) < 0)) {
          if (__hip_atomic_load(tmo, __ATOMIC_RELAXED, __HIP_MEMORY_SCOPE_AGENT) == call || __builtin_amdgcn_s_memrealtime() - t0 > DEADLINE) {
            __hip_atomic_store(tmo, call, __ATOMIC_RELAXED, __HIP_MEMORY_SCOPE_AGENT);
            waitv<0>();
            break;
          }
          __builtin_amdgcn_s_sleep(2);
        }
      }
      asm volatile("buffer_inv sc0 sc1" ::: "memory");
    }
  };
  if (!t.own) {
    issue1(0); issue1(1); issue1(2); issue1(3);
    { const char* ctr = myws + WS_CTR + t.c * 128; asm volatile("global_load_dword %0, %1, off sc1" : "+v"(callv) : "v"(ctr) : "memory"); }
    waitv<3 * L1 + 1>(); BAR();
    waitv<2 * L1 + 1>(); BAR(); issue1(4);
#pragma unroll 1
    for (int i = 1; i < 60; ++i) {
      waitv<2 * L1>();
      if (i == 58) gate(0);
      BAR();
      if (i == 56) poll(0);
      issue1(i + 4);
    }
    asm volatile("" : "+v"(callv));
    if (s2 != t.r) ra2 = rsrc(myws + WS_X + (((__builtin_amdgcn_readfirstlane(callv) + 1) & 1) * NR + s2) * XBLK + (size_t)(r2 - RROWS) * XPITCH, 32 * XPITCH);
    waitv<2 * L1>(); BAR(); issue2(0);
    waitv<L1 + L2>(); BAR(); issue2(1);
    waitv<2 * L2>(); BAR(); issue2(2);
    waitv<2 * L2>(); BAR(); issue2(3);
    if (NS2 == 5) issue2(4);
  } else {
    waitv<0>();
#pragma unroll
    for (int n = 0; n < NS2; ++n) issue2(n);
    waitv<(NS2 - 1) * L2>(); BAR();
  }
  constexpr int W2 = (NS2 - 2) * L2, IG = 32 - NS2 - (NS2 - 2), IP = IG - (NS2 - 2);
#pragma unroll 1
  for (int i = 0; i < 64 - NS2; ++i) {
    waitv<W2>();
    if (i == IG) gate(1);
    BAR();
    if (i == IP) poll(1);
    issue2(i + NS2);
  }
  static_for<0, NS2 - 1>([&](auto Q) { waitv<(NS2 - 2 - Q) * L2>(); BAR(); });
  BAR();
  asm volatile("" :: "v"(callv), "v"(fv));
}

__device__ __forceinline__ void consumer(const Params& p, const Tile& t, LAS char* lds, int wid, int lane) {
  char* const myws = t.myws;
  __builtin_amdgcn_s_setprio(2);
  const int wr = wid >> 1, wc = wid & 1, fr = lane & 15, fq = lane >> 4;
  const bool odd = fq & 1;
  const unsigned callv = __hip_atomic_load((unsigned*)(myws + WS_CTR + t.c * 128), __ATOMIC_RELAXED, __HIP_MEMORY_SCOPE_AGENT);

  const unsigned lb = (unsigned)(size_t)lds;
  unsigned a1[2], a2[2], baddr[4];
#pragma unroll
  for (int ks = 0; ks < 2; ++ks) { const unsigned u = lb + fr * 128 + 16 * ((4 * ks + fq) ^ ((fr >> 1) & 7)); a1[ks] = u + 16 * MF1 * wr * 128; a2[ks] = u + 16 * MF2 * wr * 128; }
  { const int q = (lane & 15) >> 2, pp = lane & 3, kr = 8 * fq + q, f = (kr & 1) | ((kr & 2) << 1) | (kr & 8);
#pragma unroll
    for (int gn = 0; gn < 4; ++gn) baddr[gn] = lb + kr * 256 + 16 * ((8 * wc + 4 * (gn >> 1) + pp) ^ f) + 8 * ((gn & 1) ^ (pp & 1)); }
  const auto rs1 = rsrc(p.out[t.peer], t.own ? 0 : OUT_BYTES);
  const unsigned o1 = (t.r * MLOC + t.row1 + 16 * MF1 * wr + (fr & 7)) * OPITCH + (t.pl1 * 128 + 64 * wc + 32 * (fr >> 3) + 8 * fq) * 2;
#define OSTEP(g) (8 * OPITCH)
#define OADDR(g) ((g) * 8 * OPITCH)
  const auto rs2 = rsrc(p.out[t.r], t.row2 < 0 ? 0 : OUT_BYTES);
  const unsigned o2 = (t.src2 * MLOC + t.row2 + fr) * OPITCH + (t.pl2 * 128 + 64 * wc + 8 * fq) * 2;

  Frag F0, F1;
  f32x4 acc1[MF1][4], acc2[MF2][4];
#pragma unroll
  for (int m = 0; m < MF1; ++m)
#pragma unroll
    for (int gn = 0; gn < 4; ++gn) { acc1[m][gn] = (f32x4){0.f, 0.f, 0.f, 0.f}; asm volatile("" : "+v"(acc1[m][gn])); }
  unsigned so = 0, sw, bo = t.own ? AMAX2 : AMAX, bc[4], ac[2];
#pragma unroll
  for (int gn = 0; gn < 4; ++gn) bc[gn] = baddr[gn] + bo;
  ac[0] = t.own ? a2[0] : a1[0]; ac[1] = t.own ? a2[1] : a1[1];
#pragma unroll
  for (int m = 0; m < MF2; ++m)
#pragma unroll
    for (int gn = 0; gn < 4; ++gn) { acc2[m][gn] = (f32x4){0.f, 0.f, 0.f, 0.f}; asm volatile("" : "+v"(acc2[m][gn])); }
  asm volatile("s_nop 1");
  BAR();
  STAMP(1);
  static_for<0, 8 + MF1>([&](auto K) { rd1<K, 0>(F0, ac[0], bc); });
  GeluState gs; gs.ka = GELU_A; asm volatile("" : "+v"(gs.ka));
  auto half_a = [&](auto MF_, auto& acc, const unsigned (&an)[2], auto EV, auto G0_) {
    constexpr int MF = MF_, NM = 4 * MF, NRD = 8 + MF, DBL = dbl_gaps(MF), LAST = NRD - DBL - 1, G0 = G0_;
    static_for<0, NM>([&](auto Z) {
      constexpr int z = Z;
      SCHED(); mm1<z, MF>(acc, F0); SCHED();
      if constexpr (z < DBL) { rd1<2 * z, 1>(F1, ac[1], bc); rd1<2 * z + 1, 1>(F1, ac[1], bc); }
      else if constexpr (z <= LAST) rd1<z + DBL, 1>(F1, ac[1], bc);
      if constexpr (z == LAST - 1) { if constexpr (MF == MF1) so = so + SLOT == RING ? 0 : so + SLOT; else so = so + SLOT2 == RING2 ? 0 : so + SLOT2; sw = so + bo; asm volatile("" : "+s"(so), "+s"(sw)); }
      if constexpr (z == LAST + 1) { bc[0] = baddr[0] + sw; bc[1] = baddr[1] + sw; asm volatile("" : "+v"(bc[0]), "+v"(bc[1])); }
      if constexpr (z == LAST + 2) { bc[2] = baddr[2] + sw; bc[3] = baddr[3] + sw; asm volatile("" : "+v"(bc[2]), "+v"(bc[3])); }
      if constexpr (z == LAST + 3) { ac[0] = an[0] + so; ac[1] = an[1] + so; asm volatile("" : "+v"(ac[0]), "+v"(ac[1])); }
      if constexpr (G0 >= 0 && z > LAST + 3) gelu_piece<G0 + z - LAST - 4>(gs);
    });
    static_assert(LAST + 3 < NM, "half step too short");
    SCHED();
  };
  auto half_b = [&](auto MF_, auto MFN_, auto& acc, auto ZERO, auto G0_) {
    constexpr int MF = MF_, MFN = MFN_, NM = 4 * MF, NRD = 8 + MFN, DBL = dbl_gaps(MF), G0 = G0_;
    SCHED(); WAIT_L0(); BAR();
    static_for<0, NM>([&](auto Z) {
      constexpr int z = Z;
      SCHED(); mm1<z, MF>(acc, F1); SCHED();
      if constexpr (z < DBL) { rd1<2 * z, 0>(F0, ac[0], bc); rd1<2 * z + 1, 0>(F0, ac[0], bc); }
      else if constexpr (z + DBL < NRD) rd1<z + DBL, 0>(F0, ac[0], bc);
      else if constexpr (G0 >= 0) gelu_piece<G0 + z + DBL - NRD>(gs);
    });
    SCHED();
  };
  constexpr std::integral_constant<int, MF1> c1{}; constexpr std::integral_constant<int, MF2> c2{};
  constexpr std::integral_constant<int, -1> nog{}; constexpr std::true_type yes{}; constexpr std::false_type no{};
  int it = 63;
  if (!t.own) {
#pragma unroll 1
    do {
      half_a(c1, acc1, a1, no, nog);
      half_b(c1, c1, acc1, no, nog);
      --it; asm volatile("" : "+s"(it));
    } while (it != 0);
    bo = AMAX2;
    half_a(c1, acc1, a2, no, nog);
    half_b(c1, c2, acc1, no, nog);
    asm volatile("s_nop 15" ::: "memory"); SCHED();
  }
  so = __builtin_amdgcn_readfirstlane(so); bo = AMAX2;
  const unsigned dg2 = (unsigned)__builtin_amdgcn_s_memrealtime(); __builtin_amdgcn_s_waitcnt(0xC07F);
  STAMP(2);
  constexpr int FA = 4 * MF2 - (8 + MF2 - dbl_gaps(MF2)) - 3, FB = 4 * MF2 - (8 + MF2 - dbl_gaps(MF2));
  static_assert(GL * (FA + FB) >= 42, "GELU pieces do not fit");
  constexpr std::integral_constant<int, GL * FB + (GL - 1) * FA> gtail{};
  const unsigned call = __builtin_amdgcn_readfirstlane(callv) + 1;
  static_for<0, G0N>([&](auto G) { constexpr int g = G;
    pick<g>(gs, acc1); __builtin_amdgcn_raw_buffer_store_b128(epi8(gs.cu, gs.cv, odd), rs1, o1 + OADDR(g), 0, AUX_SYS); });
  unsigned oo = o1 + OADDR(G0N);
  it = G0N;
#pragma unroll 1
  do {
    half_a(c2, acc2, a2, no, gtail);
    if (it != G0N) { __builtin_amdgcn_raw_buffer_store_b128((u32x4){gs.ov0, gs.ov1, gs.ov2, gs.ov3}, rs1, oo, 0, AUX_SYS); oo += OSTEP(it); }
#define PICK(g) case g: pick<g>(gs, acc1); break
    switch (it) { PICK(0); PICK(1); PICK(2); PICK(3); PICK(4); PICK(5); PICK(6); PICK(7); PICK(8); PICK(9); default: __builtin_unreachable(); }
    asm volatile("" : "+v"(gs.cu), "+v"(gs.cv));
    half_b(c2, c2, acc2, no, std::integral_constant<int, 0>{});
    static_for<1, GL>([&](auto K) { constexpr int k = K;
      half_a(c2, acc2, a2, no, std::integral_constant<int, FB + (k - 1) * (FA + FB)>{});
      half_b(c2, c2, acc2, no, std::integral_constant<int, k * (FA + FB)>{}); });
    ++it; asm volatile("" : "+s"(it));
  } while (it != 2 * MF1);
  half_a(c2, acc2, a2, no, gtail);
  __builtin_amdgcn_raw_buffer_store_b128((u32x4){gs.ov0, gs.ov1, gs.ov2, gs.ov3}, rs1, oo, 0, AUX_SYS);
  half_b(c2, c2, acc2, no, nog);
  constexpr int NTR = 63 - (2 * MF1 - G0N) * GL;
  static_assert(NTR >= 1, "all groups leave inside the loop");
  it = NTR;
#pragma unroll 1
  do {
    half_a(c2, acc2, a2, no, nog);
    if (it == (NTR > DLAG ? NTR - DLAG : 1)) {
      waitv<0>();
      if (!t.own && lane == 0) __hip_atomic_fetch_add((unsigned*)(p.ws[t.peer] + WS_DONE + (t.r * 4 + wid) * 128), 1u, __ATOMIC_RELAXED, __HIP_MEMORY_SCOPE_SYSTEM);
      STAMP(4);
    }
    half_b(c2, c2, acc2, no, nog);
    --it; asm volatile("" : "+s"(it));
  } while (it != 0);
  asm volatile("s_nop 15" ::: "memory"); SCHED();
  const unsigned dg3 = (unsigned)__builtin_amdgcn_s_memrealtime(); __builtin_amdgcn_s_waitcnt(0xC07F);
  STAMP(3);
  const unsigned tmov = __hip_atomic_load((unsigned*)(myws + WS_TMO), __ATOMIC_RELAXED, __HIP_MEMORY_SCOPE_AGENT);
  SCHED();
#define AUXL AUX_SYS
  static_for<0, 2 * MF2>([&](auto G) { constexpr int g = G;
    __builtin_amdgcn_raw_buffer_store_b128(epi8(acc2[g >> 1][2 * (g & 1)], acc2[g >> 1][2 * (g & 1) + 1], odd), rs2, o2 + (g >> 1) * 16 * OPITCH + (g & 1) * 64, 0, AUXL); });
  STAMP(5);
  if (wid == 0 && lane == 0) __hip_atomic_store((unsigned*)(myws + WS_CTR + t.c * 128), call, __ATOMIC_RELAXED, __HIP_MEMORY_SCOPE_AGENT);
  unsigned* const dgw = (unsigned*)(myws + (40u << 20)) + (call & 4095) * 512;
  waitv<0>();
  { const unsigned dg5 = (unsigned)__builtin_amdgcn_s_memrealtime();
    const unsigned val = DIAG == 1 ? dg2 : DIAG == 3 ? dg3 : DIAG == 6 ? t.t0 + (dg3 - dg2) : DIAG == 7 ? (t.own ? dg2 : t.t0) : dg5;
    if (lane == 0 && wid == 0) { __hip_atomic_store(dgw + t.c * 2, t.t0, __ATOMIC_RELAXED, __HIP_MEMORY_SCOPE_AGENT); waitv<0>(); __hip_atomic_store(dgw + t.c * 2 + 1, val | 1u, __ATOMIC_RELAXED, __HIP_MEMORY_SCOPE_AGENT); } }
  bool bad = tmov == call;
  if (t.c == 0 && wid == 0) {
    unsigned* const tmo = (unsigned*)(myws + WS_TMO);
    unsigned* const fp = (unsigned*)(myws + WS_DONE + (((t.r + 1 + (lane < 28 ? lane >> 2 : 0)) & 7) * 4 + (lane & 3)) * 128);
    const unsigned need = NEED(32u * call);
    const u64 t0 = __builtin_amdgcn_s_memrealtime();
    for (unsigned i = 1; __any((int)(__hip_atomic_load(fp, __ATOMIC_RELAXED, __HIP_MEMORY_SCOPE_SYSTEM) - need) < 0); ++i) {
      if ((i & 15) == 0 && (__hip_atomic_load(tmo, __ATOMIC_RELAXED, __HIP_MEMORY_SCOPE_AGENT) == call || __builtin_amdgcn_s_memrealtime() - t0 > DEADLINE)) {
        __hip_atomic_store(tmo, call, __ATOMIC_RELAXED, __HIP_MEMORY_SCOPE_AGENT);
        bad = true;
        break;
      }
      __builtin_amdgcn_s_sleep(1);
    }
  }
  if (t.c == 0 && wid == 0) {
    const unsigned dg6 = (unsigned)__builtin_amdgcn_s_memrealtime();
    while ((unsigned)__builtin_amdgcn_s_memrealtime() - t.t0 < 14000u) __builtin_amdgcn_s_sleep(8);
    int so_ = 0, mo_ = -100000, sum_ = 0;
    for (int q = 0; q < 4; ++q) { const int a_ = (int)(__hip_atomic_load(dgw + (q * 64 + lane) * 2, __ATOMIC_RELAXED, __HIP_MEMORY_SCOPE_AGENT) - t.t0), b_ = (int)(__hip_atomic_load(dgw + (q * 64 + lane) * 2 + 1, __ATOMIC_RELAXED, __HIP_MEMORY_SCOPE_AGENT) - t.t0);
      so_ = min(so_, a_); mo_ = max(mo_, b_); sum_ += b_; }
    for (int o = 32; o >= 1; o >>= 1) { so_ = min(so_, __shfl_xor(so_, o)); mo_ = max(mo_, __shfl_xor(mo_, o)); sum_ += __shfl_xor(sum_, o); }
    const unsigned st = t.t0 + so_, mv = t.t0 + (DIAGAVG ? sum_ / 256 : mo_);
    const unsigned m = DIAG == 0 ? 0 : DIAG == 9 ? dg6 - st : DIAG == 8 ? (unsigned)((int)(dg6 - mv) + 2000) : mv - st;
    while ((unsigned)__builtin_amdgcn_s_memrealtime() - st < 15000u + (m < 20000u ? m : 20000u)) __builtin_amdgcn_s_sleep(1);
  }
  if (__builtin_expect(bad, 0)) {
    waitv<0>();
    for (int g = 0; g < 2 * MF2; ++g)
      __builtin_amdgcn_raw_buffer_store_b128((u32x4){0x7fc07fc0u, 0x7fc07fc0u, 0x7fc07fc0u, 0x7fc07fc0u}, rs2, o2 + (g >> 1) * 16 * OPITCH + (g & 1) * 64, 0, AUX_SYS);
  }
  STAMP(6);
}
}

__global__ __launch_bounds__(512, 2) void gemm_a2a(Params p) {
  extern __shared__ __attribute__((aligned(16))) char smem[];
  LAS char* lds = (LAS char*)smem;
  const int lane = threadIdx.x & 63, wid = __builtin_amdgcn_readfirstlane(threadIdx.x >> 6);
  Tile t;
  u64 ax = (u64)p.x, aw = (u64)p.w, as = (u64)p.myws; t.r = p.rank;
  asm volatile("" : "+s"(ax), "+s"(aw), "+s"(as), "+s"(t.r));
  t.x = (const char*)(const GAS char*)ax; t.w = (const char*)(const GAS char*)aw; t.myws = (char*)(GAS char*)as;
  t.t0 = (unsigned)__builtin_amdgcn_s_memrealtime();
  t.c = blockIdx.x;
  const int xcd = t.c & 7, j = ((t.c >> 3) + 28) & 31, e = j >> 2, hm = j & 1, pb = (j >> 1) & 1, h2 = xcd & 1;
  t.own = j >= 28;
  t.peer = (t.r + 1 + e) & 7;
  t.pl1 = xcd + 8 * pb;
  t.pl2 = 4 * (xcd >> 1) + (j & 3);
  t.row1 = 32 * MF1 * hm;
  if (((wid >> 1) & 1) == 0) { t.src2 = t.peer; t.row2 = t.own ? 128 * h2 : RROWS + 128 * h2; }
  else { t.src2 = h2 ? t.r : t.peer; t.row2 = t.own ? 128 * h2 + 64 : h2 ? (e < 4 ? 256 + 64 * e : -1) : RROWS + 64; }
#define R1M 2
#define R1J 1
#define R2M 1
  t.rot1 = R1M * hm + R1J * (j >> 1); t.rot2 = R2M * (e + j);
  STAMP(0);
  if (__builtin_expect(t.own, 0)) xpush(p, t, wid, lane);
  if (__builtin_expect(wid >= 4, 1)) producer(p, t, lds, wid - 4, lane); else consumer(p, t, lds, wid, lane);
}

extern "C" void kernel_launch(int rank, int nranks, ncclComm_t comm,
                              hipStream_t stream,
                              void* const* d_in, const int* in_sizes, int n_in,
                              void* d_out, int out_size,
                              const int* in_dtypes, int out_dtype,
                              void* const* d_out_all,
                              void* const* d_ws_all, size_t ws_bytes) {
  if (nranks != NR || (unsigned)rank >= (unsigned)NR || n_in != 2 || in_sizes[0] != MLOC * KD || in_sizes[1] != KD * NG ||
      out_size != NR * MLOC * NLOC || ws_bytes < WS_NEED) { fprintf(stderr, "gemm_a2a: unexpected problem shape\n"); return; }
  static bool attr_set[NR] = {};
  if (!attr_set[rank]) { HIP_CHECK(hipFuncSetAttribute((const void*)gemm_a2a, hipFuncAttributeMaxDynamicSharedMemorySize, LDS_BYTES)); attr_set[rank] = true; }
  Params p;
  p.x = (const char*)d_in[0]; p.w = (const char*)d_in[1]; p.rank = rank; p.pad = 0; p.myws = (char*)d_ws_all[rank];
  for (int i = 0; i < NR; ++i) { p.out[i] = (char*)d_out_all[i]; p.ws[i] = (char*)d_ws_all[i]; }
  hipLaunchKernelGGL(gemm_a2a, dim3(256), dim3(512), LDS_BYTES, stream, p);
  HIP_CHECK(hipGetLastError());
}
```

```cpp
#include <hip/hip_runtime.h>
#include <rccl/rccl.h>
#include <cstdio>
#include <type_traits>

#define HIP_CHECK(expr) do { hipError_t _e = (expr); if (_e != hipSuccess) { \
    fprintf(stderr, "HIP error %s at %s:%d\n", hipGetErrorString(_e), __FILE__, __LINE__); \
    return; } } while (0)

namespace {
#define LAS __attribute__((address_space(3)))
#define GAS __attribute__((address_space(1)))
typedef short s16x4 __attribute__((ext_vector_type(4)));
typedef short s16x8 __attribute__((ext_vector_type(8)));
typedef __bf16 bf16x8 __attribute__((ext_vector_type(8)));
typedef float f32x4 __attribute__((ext_vector_type(4)));
typedef unsigned u32x4 __attribute__((ext_vector_type(4)));
typedef unsigned long long u64;

constexpr int NR = 8, MLOC = 512, KD = 4096, NG = 16384, NLOC = 2048;
constexpr int MF1 = 5, MF2 = 4;
constexpr int RROWS = 64 * MF1, FROWS = MLOC - RROWS;
constexpr int NS2 = 5;
constexpr int AMAX = 32 * MF1 * 128, SLOT = AMAX + 16384, RING = 4 * SLOT;
constexpr int AMAX2 = 32 * MF2 * 128, SLOT2 = AMAX2 + 16384, RING2 = NS2 * SLOT2, LDS_BYTES = RING > RING2 ? RING : RING2;
static_assert(FROWS == 192 && LDS_BYTES <= 163840, "tiles");
constexpr unsigned XPITCH = KD * 2, WPITCH = NG * 2, OPITCH = NLOC * 2;
constexpr unsigned OUT_BYTES = (unsigned)NR * MLOC * OPITCH;
constexpr unsigned XBLK = FROWS * XPITCH;
constexpr unsigned WS_CTR = 0;
constexpr unsigned WS_TMO = 32768;
constexpr unsigned WS_XFLAG = 36864;
constexpr unsigned WS_DONE = 49152;
constexpr unsigned WS_X = 1u << 20;
constexpr unsigned WS_NEED = WS_X + 2 * NR * XBLK;
constexpr u64 DEADLINE = 40000000ull;
constexpr int AUX_SYS = 17;

constexpr int GL = 6;
constexpr int G0N = 1;
constexpr int DLAG = 3;

struct Params { const char* x; const char* w; char* myws; int rank, pad; char* out[NR]; char* ws[NR]; };
struct Tile { const char* x; const char* w; char* myws; int c, r, peer, own, pl1, pl2, row1, src2, row2, rot1, rot2; };

__device__ __forceinline__ unsigned pk_bf16(float lo, float hi) {
  typedef __bf16 bf16x2 __attribute__((ext_vector_type(2)));
  bf16x2 v = {(__bf16)lo, (__bf16)hi};
  return __builtin_bit_cast(unsigned, v);
}
constexpr float GELU_A = -2.0f * 0.7978845608028654f * 1.4426950408889634f, GELU_B = GELU_A * 0.044715f;
typedef float f32x2 __attribute__((ext_vector_type(2)));
__device__ __forceinline__ unsigned gelu2(const f32x2 y) {
  const f32x2 z = y * __builtin_elementwise_fma(y * y, (f32x2){GELU_B, GELU_B}, (f32x2){GELU_A, GELU_A});
  const f32x2 d = (f32x2){__builtin_amdgcn_exp2f(z.x), __builtin_amdgcn_exp2f(z.y)} + 1.0f;
  const f32x2 o = y * (f32x2){__builtin_amdgcn_rcpf(d.x), __builtin_amdgcn_rcpf(d.y)};
  return pk_bf16(o.x, o.y);
}
__device__ __forceinline__ u32x4 epi8(const f32x4 u, const f32x4 v, bool odd) {
  const unsigned a0 = gelu2(u.xy), a1 = gelu2(u.zw), b0 = gelu2(v.xy), b1 = gelu2(v.zw);
  u32x4 w; w.x = odd ? b0 : a0; w.y = odd ? b1 : a1; w.z = odd ? a0 : b0; w.w = odd ? a1 : b1;
  return w;
}
template <int N_> __device__ __forceinline__ void waitv() { asm volatile("s_waitcnt vmcnt(%0)" :: "i"(N_) : "memory"); }
#define WAIT_L0() __builtin_amdgcn_s_waitcnt(0xC07F)
#define BAR() __builtin_amdgcn_s_barrier()
#define SCHED() __builtin_amdgcn_sched_barrier(0)
template <int I, int N, class F> __device__ __forceinline__ void static_for(F&& f) { if constexpr (I < N) { f(std::integral_constant<int, I>{}); static_for<I + 1, N>(f); } }
__device__ __forceinline__ LAS char* lp(unsigned a) { return (LAS char*)(size_t)a; }
__device__ __forceinline__ auto rsrc(const void* base, unsigned bytes) { return __builtin_amdgcn_make_buffer_rsrc((void*)base, 0, (int)bytes, 0x00020000); }

struct Frag { s16x8 a[MF1]; s16x4 blo[4], bhi[4]; };
template <int K, int KS> __device__ __forceinline__ void rd1(Frag& f, unsigned aaddr, const unsigned (&baddr)[4]) {
  if constexpr (K == 2) f.a[0] = *(LAS s16x8*)lp(aaddr);
  else if constexpr (K >= 9) f.a[K - 8] = *(LAS s16x8*)(lp(aaddr) + (K - 8) * 2048);
  else {
    constexpr int q = K < 2 ? K : K - 1, gn = q >> 1;
    if constexpr (q & 1) f.bhi[gn] = __builtin_amdgcn_ds_read_tr16_b64_v4i16((LAS s16x4*)(lp(baddr[gn]) + KS * 8192 + 1024));
    else f.blo[gn] = __builtin_amdgcn_ds_read_tr16_b64_v4i16((LAS s16x4*)(lp(baddr[gn]) + KS * 8192));
  }
}
template <int Z, int MF> __device__ __forceinline__ void mm1(f32x4 (&acc)[MF][4], const Frag& f) {
  constexpr int m = Z >> 2, gn = Z & 3;
  const s16x8 b = __builtin_shufflevector(f.blo[gn], f.bhi[gn], 0, 1, 2, 3, 4, 5, 6, 7);
  asm volatile("v_mfma_f32_16x16x32_bf16 %0, %1, %2, %0" : "+v"(acc[m][gn]) : "v"(b), "v"(f.a[m]));
}
struct GeluState { f32x4 cu, cv; float ga, gb, ka; unsigned pk0, pk1, pk2, pk3, ov0, ov1, ov2, ov3; };
template <int F> __device__ __forceinline__ void gelu_piece(GeluState& g) {
  constexpr int pr = F / 10, st = F % 10;
  constexpr u64 ODDM = 0xffff0000ffff0000ull;
  if constexpr (F < 40) {
    const float ya = pr < 2 ? g.cu[2 * (pr & 1)] : g.cv[2 * (pr & 1)], yb = pr < 2 ? g.cu[2 * (pr & 1) + 1] : g.cv[2 * (pr & 1) + 1];
    if constexpr (st == 0) asm volatile("v_mul_f32 %0, %2, %2\n\tv_mul_f32 %1, %3, %3" : "=&v"(g.ga), "=v"(g.gb) : "v"(ya), "v"(yb));
    if constexpr (st == 1) asm volatile("v_fmamk_f32 %0, %0, %3, %2\n\tv_fmamk_f32 %1, %1, %3, %2" : "+v"(g.ga), "+v"(g.gb) : "v"(g.ka), "i"(__builtin_bit_cast(unsigned, GELU_B)));
    if constexpr (st == 2 || st == 8) asm volatile("v_mul_f32 %0, %2, %0\n\tv_mul_f32 %1, %3, %1" : "+v"(g.ga), "+v"(g.gb) : "v"(ya), "v"(yb));
    if constexpr (st == 3) asm volatile("v_exp_f32 %0, %0" : "+v"(g.ga));
    if constexpr (st == 4) asm volatile("v_exp_f32 %0, %0" : "+v"(g.gb));
    if constexpr (st == 5) asm volatile("v_add_f32 %0, 1.0, %0\n\tv_add_f32 %1, 1.0, %1" : "+v"(g.ga), "+v"(g.gb));
    if constexpr (st == 6) asm volatile("v_rcp_f32 %0, %0" : "+v"(g.ga));
    if constexpr (st == 7) asm volatile("v_rcp_f32 %0, %0" : "+v"(g.gb));
    if constexpr (st == 9) { unsigned& d = pr == 0 ? g.pk0 : pr == 1 ? g.pk1 : pr == 2 ? g.pk2 : g.pk3; asm volatile("v_cvt_pk_bf16_f32 %0, %1, %2" : "=v"(d) : "v"(g.ga), "v"(g.gb)); }
  } else if constexpr (F == 40) asm volatile("v_cndmask_b32_e64 %0, %2, %4, %6\n\tv_cndmask_b32_e64 %1, %3, %5, %6" : "=&v"(g.ov0), "=v"(g.ov1) : "v"(g.pk0), "v"(g.pk1), "v"(g.pk2), "v"(g.pk3), "s"(ODDM));
  else if constexpr (F == 41) asm volatile("v_cndmask_b32_e64 %0, %4, %2, %6\n\tv_cndmask_b32_e64 %1, %5, %3, %6" : "=&v"(g.ov2), "=v"(g.ov3) : "v"(g.pk0), "v"(g.pk1), "v"(g.pk2), "v"(g.pk3), "s"(ODDM));
}
template <int G> __device__ __forceinline__ void pick(GeluState& s, const f32x4 (&acc)[MF1][4]) {
  constexpr int m = G >> 1, h = G & 1;
#pragma unroll
  for (int i = 0; i < 4; ++i) {
    const float u0 = acc[m][2 * h][i], u1 = acc[m][2 - 2 * h][i], v0 = acc[m][2 * h + 1][i], v1 = acc[m][3 - 2 * h][i];
    s.cu[i] = __int_as_float(__builtin_amdgcn_update_dpp(__float_as_int(u0), __float_as_int(u1), 0x128, 0xf, h ? 0x3 : 0xc, false));
    s.cv[i] = __int_as_float(__builtin_amdgcn_update_dpp(__float_as_int(v0), __float_as_int(v1), 0x128, 0xf, h ? 0x3 : 0xc, false));
  }
}
constexpr int dbl_gaps(int mf) { return mf == 4 ? 1 : 0; }


constexpr int XFLY = 3;
__device__ __forceinline__ void xpush(const Params& p, const Tile& t, int wid, int lane) {
  const u64 wsl = (u64)p.ws[(t.r + 1 + (lane < 7 ? lane : 0)) & 7];
  const int w = t.c * 8 + wid;
  const unsigned par = (__builtin_amdgcn_readfirstlane(__hip_atomic_load((unsigned*)(t.myws + WS_CTR + t.c * 128), __ATOMIC_RELAXED, __HIP_MEMORY_SCOPE_AGENT)) + 1) & 1;
  u32x4 v[2][3]; unsigned po[3];
#pragma unroll
  for (int i = 0; i < 3; ++i) { const int q = 3 * w + i; po[i] = (q >> 2) * XPITCH + (q & 3) * 1024 + lane * 16;
    v[0][i] = *(const u32x4*)(t.x + (size_t)RROWS * XPITCH + po[i]); v[1][i] = *(const u32x4*)(t.x + (size_t)RROWS * XPITCH + po[i] + 4096); }
#pragma unroll
  for (int h = 0; h < 2; ++h) {
    int e = w % 7;
#pragma unroll 1
    for (int n = 0; n < 7; ++n) {
      char* base = (char*)(((u64)(unsigned)__builtin_amdgcn_readlane((unsigned)(wsl >> 32), e) << 32) | (unsigned)__builtin_amdgcn_readlane((unsigned)wsl, e));
      const auto rs = rsrc(base + WS_X + (par * NR + t.r) * XBLK, XBLK);
#pragma unroll
      for (int i = 0; i < 3; ++i) { __builtin_amdgcn_raw_buffer_store_b128(v[h][i], rs, po[i], h * 4096, AUX_SYS); waitv<XFLY>(); }
      e = e == 6 ? 0 : e + 1;
    }
    waitv<0>(); BAR();
    if (wid == 0 && lane < 7) __hip_atomic_fetch_add((unsigned*)(GAS unsigned*)(wsl + WS_XFLAG + (t.r * 2 + h) * 128), 1u, __ATOMIC_RELAXED, __HIP_MEMORY_SCOPE_SYSTEM);
  }
}

__device__ __forceinline__ void producer(const Params& p, const Tile& t, LAS char* lds, int pw, int lane) {
  char* const myws = t.myws;
  const unsigned va0 = (lane >> 3) * XPITCH + (((lane & 7) ^ (lane >> 4)) << 4), va1 = va0 ^ 64;
  const unsigned vb0 = (lane >> 4) * WPITCH + (((lane & 15) ^ (((lane >> 4) & 1) | ((lane >> 5) << 2))) << 4), vb1 = vb0 ^ 128;
  const unsigned ve0 = va0 ^ ((pw & 1) << 6), ve1 = ve0 ^ 64;
  const auto ra1 = rsrc(t.x + (size_t)(t.row1 + 40 * pw) * XPITCH, 40 * XPITCH);
  const auto rb1 = rsrc(t.w + (size_t)(t.peer * 16 + t.pl1) * 256, (KD - 1) * WPITCH + 256);
  const int s2 = t.src2, r2 = t.row2 + 32 * (pw & 1);
  auto ra2 = rsrc(t.x + (size_t)r2 * XPITCH, t.row2 < 0 ? 0 : 32 * XPITCH);
  const auto rb2 = rsrc(t.w + (size_t)(t.r * 16 + t.pl2) * 256, (KD - 1) * WPITCH + 256);
  unsigned so = 0;
  auto issue1 = [&](int s) {
    const int kt = (s & 60) | ((s + t.rot1) & 3);
#pragma unroll
    for (int i = 0; i < MF1; ++i) __builtin_amdgcn_raw_ptr_buffer_load_lds(ra1, lds + so + (pw * MF1 + i) * 1024, 16, (i & 1) ? ve1 : ve0, kt * 128 + i * 8 * XPITCH, 0, 0);
#pragma unroll
    for (int i = 0; i < 4; ++i) __builtin_amdgcn_raw_ptr_buffer_load_lds(rb1, lds + so + AMAX + (pw * 4 + i) * 1024, 16, (i & 2) ? vb1 : vb0, (kt * 64 + pw * 16 + i * 4) * WPITCH, 0, 0);
    so = so + SLOT == RING ? 0 : so + SLOT;
  };
  auto issue2 = [&](int s) {
    const int kt = (s & 60) | ((s + t.rot2) & 3);
#pragma unroll
    for (int i = 0; i < MF2; ++i) __builtin_amdgcn_raw_ptr_buffer_load_lds(ra2, lds + so + (pw * MF2 + i) * 1024, 16, (i & 1) ? va1 : va0, kt * 128 + i * 8 * XPITCH, 0, 0);
#pragma unroll
    for (int i = 0; i < 4; ++i) __builtin_amdgcn_raw_ptr_buffer_load_lds(rb2, lds + so + AMAX2 + (pw * 4 + i) * 1024, 16, (i & 2) ? vb1 : vb0, (kt * 64 + pw * 16 + i * 4) * WPITCH, 0, 0);
    so = so + SLOT2 == RING2 ? 0 : so + SLOT2;
  };
  constexpr int L1 = MF1 + 4, L2 = MF2 + 4;
  unsigned callv = 0, fv = 0;
  const bool poller = (pw == 0) && !t.own;
  auto poll = [&](int q) {
    if (poller) { const char* fp = myws + WS_XFLAG + (t.peer * 2 + q) * 128; asm volatile("global_load_dword %0, %1, off sc0 sc1" : "+v"(fv) : "v"(fp) : "memory"); }
  };
  auto gate = [&](int q) {
    if (poller) {
      asm volatile("" : "+v"(fv), "+v"(callv));
      const unsigned call = __builtin_amdgcn_readfirstlane(callv) + 1, need = 32u * call;
      if (__any((int)(fv - need) < 0)) {
        unsigned* const tmo = (unsigned*)(myws + WS_TMO);
        const u64 t0 = __builtin_amdgcn_s_memrealtime();
        while (__any((int)(__hip_atomic_load((unsigned*)(myws + WS_XFLAG + (t.peer * 2 + q) * 128), __ATOMIC_RELAXED, __HIP_MEMORY_SCOPE_SYSTEM) - need) < 0)) {
          if (__hip_atomic_load(tmo, __ATOMIC_RELAXED, __HIP_MEMORY_SCOPE_AGENT) == call || __builtin_amdgcn_s_memrealtime() - t0 > DEADLINE) {
            __hip_atomic_store(tmo, call, __ATOMIC_RELAXED, __HIP_MEMORY_SCOPE_AGENT);
            waitv<0>();
            break;
          }
          __builtin_amdgcn_s_sleep(2);
        }
      }
      asm volatile("buffer_inv sc0 sc1" ::: "memory");
    }
  };
  if (!t.own) {
    issue1(0); issue1(1); issue1(2); issue1(3);
    { const char* ctr = myws + WS_CTR + t.c * 128; asm volatile("global_load_dword %0, %1, off sc1" : "+v"(callv) : "v"(ctr) : "memory"); }
    waitv<3 * L1 + 1>(); BAR();
    waitv<2 * L1 + 1>(); BAR(); issue1(4);
#pragma unroll 1
    for (int i = 1; i < 60; ++i) {
      waitv<2 * L1>();
      if (i == 58) gate(0);
      BAR();
      if (i == 56) poll(0);
      issue1(i + 4);
    }
    asm volatile("" : "+v"(callv));
    if (s2 != t.r) ra2 = rsrc(myws + WS_X + (((__builtin_amdgcn_readfirstlane(callv) + 1) & 1) * NR + s2) * XBLK + (size_t)(r2 - RROWS) * XPITCH, 32 * XPITCH);
    waitv<2 * L1>(); BAR(); issue2(0);
    waitv<L1 + L2>(); BAR(); issue2(1);
    waitv<2 * L2>(); BAR(); issue2(2);
    waitv<2 * L2>(); BAR(); issue2(3);
    if (NS2 == 5) issue2(4);
  } else {
    waitv<0>();
#pragma unroll
    for (int n = 0; n < NS2; ++n) issue2(n);
    waitv<(NS2 - 1) * L2>(); BAR();
  }
  constexpr int W2 = (NS2 - 2) * L2, IG = 32 - NS2 - (NS2 - 2), IP = IG - (NS2 - 2);
#pragma unroll 1
  for (int i = 0; i < 64 - NS2; ++i) {
    waitv<W2>();
    if (i == IG) gate(1);
    BAR();
    if (i == IP) poll(1);
    issue2(i + NS2);
  }
  static_for<0, NS2 - 1>([&](auto Q) { waitv<(NS2 - 2 - Q) * L2>(); BAR(); });
  BAR();
  asm volatile("" :: "v"(callv), "v"(fv));
}

__device__ __forceinline__ void consumer(const Params& p, const Tile& t, LAS char* lds, int wid, int lane) {
  char* const myws = t.myws;
  __builtin_amdgcn_s_setprio(2);
  const int wr = wid >> 1, wc = wid & 1, fr = lane & 15, fq = lane >> 4;
  const bool odd = fq & 1;
  const unsigned callv = __hip_atomic_load((unsigned*)(myws + WS_CTR + t.c * 128), __ATOMIC_RELAXED, __HIP_MEMORY_SCOPE_AGENT);

  const unsigned lb = (unsigned)(size_t)lds;
  unsigned a1[2], a2[2], baddr[4];
#pragma unroll
  for (int ks = 0; ks < 2; ++ks) { const unsigned u = lb + fr * 128 + 16 * ((4 * ks + fq) ^ ((fr >> 1) & 7)); a1[ks] = u + 16 * MF1 * wr * 128; a2[ks] = u + 16 * MF2 * wr * 128; }
  { const int q = (lane & 15) >> 2, pp = lane & 3, kr = 8 * fq + q, f = (kr & 1) | ((kr & 2) << 1) | (kr & 8);
#pragma unroll
    for (int gn = 0; gn < 4; ++gn) baddr[gn] = lb + kr * 256 + 16 * ((8 * wc + 4 * (gn >> 1) + pp) ^ f) + 8 * ((gn & 1) ^ (pp & 1)); }
  const auto rs1 = rsrc(p.out[t.peer], t.own ? 0 : OUT_BYTES);
  const unsigned o1 = (t.r * MLOC + t.row1 + 16 * MF1 * wr + (fr & 7)) * OPITCH + (t.pl1 * 128 + 64 * wc + 32 * (fr >> 3) + 8 * fq) * 2;
  const auto rs2 = rsrc(p.out[t.r], t.row2 < 0 ? 0 : OUT_BYTES);
  const unsigned o2 = (t.src2 * MLOC + t.row2 + fr) * OPITCH + (t.pl2 * 128 + 64 * wc + 8 * fq) * 2;

  Frag F0, F1;
  f32x4 acc1[MF1][4], acc2[MF2][4];
#pragma unroll
  for (int m = 0; m < MF1; ++m)
#pragma unroll
    for (int gn = 0; gn < 4; ++gn) { acc1[m][gn] = (f32x4){0.f, 0.f, 0.f, 0.f}; asm volatile("" : "+v"(acc1[m][gn])); }
  unsigned so = 0, sw, bo = t.own ? AMAX2 : AMAX, bc[4], ac[2];
#pragma unroll
  for (int gn = 0; gn < 4; ++gn) bc[gn] = baddr[gn] + bo;
  ac[0] = t.own ? a2[0] : a1[0]; ac[1] = t.own ? a2[1] : a1[1];
#pragma unroll
  for (int m = 0; m < MF2; ++m)
#pragma unroll
    for (int gn = 0; gn < 4; ++gn) { acc2[m][gn] = (f32x4){0.f, 0.f, 0.f, 0.f}; asm volatile("" : "+v"(acc2[m][gn])); }
  asm volatile("s_nop 1");
  BAR();
  static_for<0, 8 + MF1>([&](auto K) { rd1<K, 0>(F0, ac[0], bc); });
  GeluState gs; gs.ka = GELU_A; asm volatile("" : "+v"(gs.ka));
  auto half_a = [&](auto MF_, auto& acc, const unsigned (&an)[2], auto G0_) {
    constexpr int MF = MF_, NM = 4 * MF, NRD = 8 + MF, DBL = dbl_gaps(MF), LAST = NRD - DBL - 1, G0 = G0_;
    static_for<0, NM>([&](auto Z) {
      constexpr int z = Z;
      SCHED(); mm1<z, MF>(acc, F0); SCHED();
      if constexpr (z < DBL) { rd1<2 * z, 1>(F1, ac[1], bc); rd1<2 * z + 1, 1>(F1, ac[1], bc); }
      else if constexpr (z <= LAST) rd1<z + DBL, 1>(F1, ac[1], bc);
      if constexpr (z == LAST - 1) { if constexpr (MF == MF1) so = so + SLOT == RING ? 0 : so + SLOT; else so = so + SLOT2 == RING2 ? 0 : so + SLOT2; sw = so + bo; asm volatile("" : "+s"(so), "+s"(sw)); }
      if constexpr (z == LAST + 1) { bc[0] = baddr[0] + sw; bc[1] = baddr[1] + sw; asm volatile("" : "+v"(bc[0]), "+v"(bc[1])); }
      if constexpr (z == LAST + 2) { bc[2] = baddr[2] + sw; bc[3] = baddr[3] + sw; asm volatile("" : "+v"(bc[2]), "+v"(bc[3])); }
      if constexpr (z == LAST + 3) { ac[0] = an[0] + so; ac[1] = an[1] + so; asm volatile("" : "+v"(ac[0]), "+v"(ac[1])); }
      if constexpr (G0 >= 0 && z > LAST + 3) gelu_piece<G0 + z - LAST - 4>(gs);
    });
    static_assert(LAST + 3 < NM, "half step too short");
    SCHED();
  };
  auto half_b = [&](auto MF_, auto MFN_, auto& acc, auto G0_) {
    constexpr int MF = MF_, MFN = MFN_, NM = 4 * MF, NRD = 8 + MFN, DBL = dbl_gaps(MF), G0 = G0_;
    SCHED(); WAIT_L0(); BAR();
    static_for<0, NM>([&](auto Z) {
      constexpr int z = Z;
      SCHED(); mm1<z, MF>(acc, F1); SCHED();
      if constexpr (z < DBL) { rd1<2 * z, 0>(F0, ac[0], bc); rd1<2 * z + 1, 0>(F0, ac[0], bc); }
      else if constexpr (z + DBL < NRD) rd1<z + DBL, 0>(F0, ac[0], bc);
      else if constexpr (G0 >= 0) gelu_piece<G0 + z + DBL - NRD>(gs);
    });
    SCHED();
  };
  constexpr std::integral_constant<int, MF1> c1{}; constexpr std::integral_constant<int, MF2> c2{};
  constexpr std::integral_constant<int, -1> nog{};
  int it = 63;
  if (!t.own) {
#pragma unroll 1
    do {
      half_a(c1, acc1, a1, nog);
      half_b(c1, c1, acc1, nog);
      --it; asm volatile("" : "+s"(it));
    } while (it != 0);
    bo = AMAX2;
    half_a(c1, acc1, a2, nog);
    half_b(c1, c2, acc1, nog);
    asm volatile("s_nop 15" ::: "memory"); SCHED();
  }
  so = __builtin_amdgcn_readfirstlane(so); bo = AMAX2;
  constexpr int FA = 4 * MF2 - (8 + MF2 - dbl_gaps(MF2)) - 3, FB = 4 * MF2 - (8 + MF2 - dbl_gaps(MF2));
  static_assert(GL * (FA + FB) >= 42, "GELU pieces do not fit");
  constexpr std::integral_constant<int, GL * FB + (GL - 1) * FA> gtail{};
  const unsigned call = __builtin_amdgcn_readfirstlane(callv) + 1;
  static_for<0, G0N>([&](auto G) { constexpr int g = G;
    pick<g>(gs, acc1); __builtin_amdgcn_raw_buffer_store_b128(epi8(gs.cu, gs.cv, odd), rs1, o1 + g * 8 * OPITCH, 0, AUX_SYS); });
  unsigned oo = o1 + G0N * 8 * OPITCH;
  it = G0N;
#pragma unroll 1
  do {
    half_a(c2, acc2, a2, gtail);
    if (it != G0N) { __builtin_amdgcn_raw_buffer_store_b128((u32x4){gs.ov0, gs.ov1, gs.ov2, gs.ov3}, rs1, oo, 0, AUX_SYS); oo += 8 * OPITCH; }
#define PICK(g) case g: pick<g>(gs, acc1); break
    switch (it) { PICK(0); PICK(1); PICK(2); PICK(3); PICK(4); PICK(5); PICK(6); PICK(7); PICK(8); PICK(9); default: __builtin_unreachable(); }
    asm volatile("" : "+v"(gs.cu), "+v"(gs.cv));
    half_b(c2, c2, acc2, std::integral_constant<int, 0>{});
    static_for<1, GL>([&](auto K) { constexpr int k = K;
      half_a(c2, acc2, a2, std::integral_constant<int, FB + (k - 1) * (FA + FB)>{});
      half_b(c2, c2, acc2, std::integral_constant<int, k * (FA + FB)>{}); });
    ++it; asm volatile("" : "+s"(it));
  } while (it != 2 * MF1);
  half_a(c2, acc2, a2, gtail);
  __builtin_amdgcn_raw_buffer_store_b128((u32x4){gs.ov0, gs.ov1, gs.ov2, gs.ov3}, rs1, oo, 0, AUX_SYS);
  half_b(c2, c2, acc2, nog);
  constexpr int NTR = 63 - (2 * MF1 - G0N) * GL;
  static_assert(NTR >= 1, "all groups leave inside the loop");
  it = NTR;
#pragma unroll 1
  do {
    half_a(c2, acc2, a2, nog);
    if (it == (NTR > DLAG ? NTR - DLAG : 1)) {
      waitv<0>();
      if (!t.own && lane == 0) __hip_atomic_fetch_add((unsigned*)(p.ws[t.peer] + WS_DONE + (t.r * 4 + wid) * 128), 1u, __ATOMIC_RELAXED, __HIP_MEMORY_SCOPE_SYSTEM);
    }
    half_b(c2, c2, acc2, nog);
    --it; asm volatile("" : "+s"(it));
  } while (it != 0);
  asm volatile("s_nop 15" ::: "memory"); SCHED();
  const unsigned tmov = __hip_atomic_load((unsigned*)(myws + WS_TMO), __ATOMIC_RELAXED, __HIP_MEMORY_SCOPE_AGENT);
  SCHED();
  static_for<0, 2 * MF2>([&](auto G) { constexpr int g = G;
    __builtin_amdgcn_raw_buffer_store_b128(epi8(acc2[g >> 1][2 * (g & 1)], acc2[g >> 1][2 * (g & 1) + 1], odd), rs2, o2 + (g >> 1) * 16 * OPITCH + (g & 1) * 64, 0, AUX_SYS); });
  if (wid == 0 && lane == 0) __hip_atomic_store((unsigned*)(myws + WS_CTR + t.c * 128), call, __ATOMIC_RELAXED, __HIP_MEMORY_SCOPE_AGENT);
  bool bad = tmov == call;
  if (t.c == 0 && wid == 0) {
    unsigned* const tmo = (unsigned*)(myws + WS_TMO);
    unsigned* const fp = (unsigned*)(myws + WS_DONE + (((t.r + 1 + (lane < 28 ? lane >> 2 : 0)) & 7) * 4 + (lane & 3)) * 128);
    const unsigned need = 32u * call;
    const u64 t0 = __builtin_amdgcn_s_memrealtime();
    for (unsigned i = 1; __any((int)(__hip_atomic_load(fp, __ATOMIC_RELAXED, __HIP_MEMORY_SCOPE_SYSTEM) - need) < 0); ++i) {
      if ((i & 15) == 0 && (__hip_atomic_load(tmo, __ATOMIC_RELAXED, __HIP_MEMORY_SCOPE_AGENT) == call || __builtin_amdgcn_s_memrealtime() - t0 > DEADLINE)) {
        __hip_atomic_store(tmo, call, __ATOMIC_RELAXED, __HIP_MEMORY_SCOPE_AGENT);
        bad = true;
        break;
      }
      __builtin_amdgcn_s_sleep(1);
    }
  }
  if (__builtin_expect(bad, 0)) {
    waitv<0>();
    for (int g = 0; g < 2 * MF2; ++g)
      __builtin_amdgcn_raw_buffer_store_b128((u32x4){0x7fc07fc0u, 0x7fc07fc0u, 0x7fc07fc0u, 0x7fc07fc0u}, rs2, o2 + (g >> 1) * 16 * OPITCH + (g & 1) * 64, 0, AUX_SYS);
  }
}
}

__global__ __launch_bounds__(512, 2) void gemm_a2a(Params p) {
  extern __shared__ __attribute__((aligned(16))) char smem[];
  LAS char* lds = (LAS char*)smem;
  const int lane = threadIdx.x & 63, wid = __builtin_amdgcn_readfirstlane(threadIdx.x >> 6);
  Tile t;
  u64 ax = (u64)p.x, aw = (u64)p.w, as = (u64)p.myws; t.r = p.rank;
  asm volatile("" : "+s"(ax), "+s"(aw), "+s"(as), "+s"(t.r));
  t.x = (const char*)(const GAS char*)ax; t.w = (const char*)(const GAS char*)aw; t.myws = (char*)(GAS char*)as;
  t.c = blockIdx.x;
  const int xcd = t.c & 7, j = ((t.c >> 3) + 28) & 31, e = j >> 2, hm = j & 1, pb = (j >> 1) & 1, h2 = xcd & 1;
  t.own = j >= 28;
  t.peer = (t.r + 1 + e) & 7;
  t.pl1 = xcd + 8 * pb;
  t.pl2 = 4 * (xcd >> 1) + (j & 3);
  t.row1 = 32 * MF1 * hm;
  if (((wid >> 1) & 1) == 0) { t.src2 = t.peer; t.row2 = t.own ? 128 * h2 : RROWS + 128 * h2; }
  else { t.src2 = h2 ? t.r : t.peer; t.row2 = t.own ? 128 * h2 + 64 : h2 ? (e < 4 ? 256 + 64 * e : -1) : RROWS + 64; }
    t.rot1 = 2 * hm + (j >> 1); t.rot2 = e + j;
  if (__builtin_expect(t.own, 0)) xpush(p, t, wid, lane);
  if (__builtin_expect(wid >= 4, 1)) producer(p, t, lds, wid - 4, lane); else consumer(p, t, lds, wid, lane);
}

extern "C" void kernel_launch(int rank, int nranks, ncclComm_t comm,
                              hipStream_t stream,
                              void* const* d_in, const int* in_sizes, int n_in,
                              void* d_out, int out_size,
                              const int* in_dtypes, int out_dtype,
                              void* const* d_out_all,
                              void* const* d_ws_all, size_t ws_bytes) {
  if (nranks != NR || (unsigned)rank >= (unsigned)NR || n_in != 2 || in_sizes[0] != MLOC * KD || in_sizes[1] != KD * NG ||
      out_size != NR * MLOC * NLOC || ws_bytes < WS_NEED) { fprintf(stderr, "gemm_a2a: unexpected problem shape\n"); return; }
  static bool attr_set[NR] = {};
  if (!attr_set[rank]) { HIP_CHECK(hipFuncSetAttribute((const void*)gemm_a2a, hipFuncAttributeMaxDynamicSharedMemorySize, LDS_BYTES)); attr_set[rank] = true; }
  Params p;
  p.x = (const char*)d_in[0]; p.w = (const char*)d_in[1]; p.rank = rank; p.pad = 0; p.myws = (char*)d_ws_all[rank];
  for (int i = 0; i < NR; ++i) { p.out[i] = (char*)d_out_all[i]; p.ws[i] = (char*)d_ws_all[i]; }
  hipLaunchKernelGGL(gemm_a2a, dim3(256), dim3(512), LDS_BYTES, stream, p);
  HIP_CHECK(hipGetLastError());
}
```
